```python
import numpy as np

kernel_name = "gpu_gemm_ar_m4096_k4096_n4096_bf16_none"


ABSMAX_RTOL = 0.008

COMPUTE_DTYPE = "bfloat16"
EPILOGUE = "none"
IN_QUANT = None
OUT_QUANT = None
IN_DTYPES = {"x": "bfloat16", "w_mat": "bfloat16"}
WORLD_SIZE = 8

def _to_bf16(x):
    u = np.ascontiguousarray(x, dtype=np.float32).view(np.uint32)
    return ((u + 0x7FFF + ((u >> 16) & 1)) & 0xFFFF0000).view(np.float32)

def setup_inputs():

    x = _to_bf16(np.random.randn(4096, 4096).astype(np.float32))
    w_mat = _to_bf16((np.random.randn(4096, 4096) / np.sqrt(4096)).astype(np.float32))
    return {"x": x, "w_mat": w_mat}

def _epilogue(y):
    return y

def reference(x, w_mat):
    y = (x @ w_mat).astype(np.float32)
    return _epilogue(y)
```

```cpp
#include <hip/hip_runtime.h>
#include <rccl/rccl.h>
#include <cstdio>
#include <type_traits>

typedef unsigned u32;
typedef unsigned long long u64;
typedef unsigned u32x4 __attribute__((ext_vector_type(4)));
typedef unsigned u32x2 __attribute__((ext_vector_type(2)));
typedef unsigned short u16x2 __attribute__((ext_vector_type(2)));
#define LAS __attribute__((address_space(3)))
typedef LAS unsigned char lds_u8;

constexpr int CD_V = 0;
constexpr int CD_R2 = 1040;
constexpr int CD_R3 = CD_R2 + 64 * 68;
constexpr int CD_R4 = CD_R3 + 64 * 36;
constexpr int CD_BYTES = CD_R4 + 64 * 20;
constexpr int CD_DEC_R2 = 1040, CD_DEC_R3 = CD_DEC_R2 + 64 * 36, CD_DEC_BYTES = CD_DEC_R3 + 64 * 20;
constexpr int CD_TAB_BYTES = 2048;
static_assert(CD_BYTES % 16 == 0 && CD_DEC_BYTES % 16 == 0, "align");

__device__ __forceinline__ u32 perm(u32 hi, u32 lo, u32 sel) { return __builtin_amdgcn_perm(hi, lo, sel); }
__device__ __forceinline__ u32 dot4(u32 a, u32 b, u32 c) { return __builtin_amdgcn_udot4(a, b, c, false); }
__device__ __forceinline__ u32 pk_sub(u32 a, u32 b) { return __builtin_bit_cast(u32, (u16x2)(__builtin_bit_cast(u16x2, a) - __builtin_bit_cast(u16x2, b))); }
__device__ __forceinline__ u32 pk_min(u32 a, u32 b) { return __builtin_bit_cast(u32, __builtin_elementwise_min(__builtin_bit_cast(u16x2, a), __builtin_bit_cast(u16x2, b))); }
__device__ __forceinline__ u32 wave_incl_scan(u32 x) {
    x += (u32)__builtin_amdgcn_update_dpp(0, (int)x, 0x111, 0xf, 0xf, false);
    x += (u32)__builtin_amdgcn_update_dpp(0, (int)x, 0x112, 0xf, 0xf, false);
    x += (u32)__builtin_amdgcn_update_dpp(0, (int)x, 0x114, 0xf, 0xf, false);
    x += (u32)__builtin_amdgcn_update_dpp(0, (int)x, 0x118, 0xf, 0xf, false);
    x += (u32)__builtin_amdgcn_update_dpp(0, (int)x, 0x142, 0xa, 0xf, false);
    x += (u32)__builtin_amdgcn_update_dpp(0, (int)x, 0x143, 0xc, 0xf, false);
    return x;
}
template <class T> __device__ __forceinline__ LAS T* lp(lds_u8* p) { return (LAS T*)p; }
#define CFENCE() asm volatile("" ::: "memory")

__device__ __forceinline__ void codec_build_tables(lds_u8* tab, int i) {
    u32 dep = 0, cmp = 0x0c0c0c0cu; int n = 0;
#pragma unroll
    for (int k = 0; k < 4; ++k) {
        const u32 c = (i >> (2 * k)) & 3;
        if (c == 3) { dep |= (u32)n << (8 * k); cmp = (cmp & ~(0xffu << (8 * n))) | ((u32)k << (8 * n)); ++n; }
        else dep |= (4u + c) << (8 * k);
    }
    lp<u32>(tab)[i] = dep; lp<u32>(tab)[256 + i] = cmp;
}

__device__ __forceinline__ void stream_append(LAS u32* r, u32 pk, u32& p, u32 pnew, u32& acc) {
    const u64 w = (u64)pk << ((p & 3u) * 8u);
    const u32 lo = (u32)w | acc;
    r[p >> 2] = lo;
    acc = ((p ^ pnew) & ~3u) ? (u32)(w >> 32) : lo;
    p = pnew;
}

__device__ __forceinline__ u32 codec_encode(const u32 (&x)[32], u32 (&A)[16], u32 (&L1)[4], lds_u8* sc, const lds_u8* tab, int lane) {
    const LAS u32* CMP = (const LAS u32*)(tab + 1024);
    LAS u32* r2 = (LAS u32*)(sc + CD_R2) + lane * 17; LAS u32* r3 = (LAS u32*)(sc + CD_R3) + lane * 9; LAS u32* r4 = (LAS u32*)(sc + CD_R4) + lane * 5;
    const u32x4 z4 = {0, 0, 0, 0};
#pragma unroll
    for (int k = 0; k < 8; ++k) r2[k] = 0;
#pragma unroll
    for (int k = 0; k < 4; ++k) r3[k] = 0;
    r4[0] = 0; r4[1] = 0;
    *lp<u32x4>(sc + CD_V + lane * 16) = z4; if (lane == 0) *lp<u32x4>(sc + CD_V + 1024) = z4;
    u32 B;
    { const u32 y0 = __builtin_amdgcn_alignbit(x[0], x[0], 31), y1 = __builtin_amdgcn_alignbit(x[1], x[1], 31);
      const u32 s = wave_incl_scan(dot4(perm(y1, y0, 0x07050301u), 0x01010101u, 0u));
      B = ((u32)__builtin_amdgcn_readlane((int)s, 63) + 724u) >> 8; B = B > 255u ? 255u : B; }
    const u32 BK = (B << 8 | 0xffu) * 0x00010001u;
    u32 tb[16], bb[16], pn[16];
    u32 pc = 0;
#pragma unroll
    for (int g = 0; g < 16; ++g) {
        const u32 y0 = __builtin_amdgcn_alignbit(x[2 * g], x[2 * g], 31), y1 = __builtin_amdgcn_alignbit(x[2 * g + 1], x[2 * g + 1], 31);
        A[g] = perm(y1, y0, 0x06040200u);
        const u32 d0 = pk_sub(BK, y0), d1 = pk_sub(BK, y1);
        const u32 m0 = pk_min(d0, 0x07ff07ffu), m1 = pk_min(d1, 0x07ff07ffu);
        const u32 tc = perm(m1, m0, 0x07050301u);
        tb[g] = perm(d1, d0, 0x07050301u) ^ 0x04040404u;
        const u32 c1 = perm(0x03030303u, 0x02010003u, tc), ef = perm(0x01010101u, 0x00000001u, tc);
        bb[g] = dot4(c1, 0x40100401u, 0u);
        pc = dot4(ef, 0x01010101u, pc); pn[g] = pc;
        if ((g & 3) == 0) L1[g >> 2] = bb[g]; else L1[g >> 2] |= bb[g] << (8 * (g & 3));
    }
    u32 sel[16];
#pragma unroll
    for (int g = 0; g < 16; ++g) sel[g] = CMP[bb[g]];
    CFENCE();
    u32 p2 = 0, acc = 0;
#pragma unroll
    for (int g = 0; g < 16; ++g) stream_append(r2, perm(0u, tb[g], sel[g]), p2, pn[g], acc);
    r2[p2 >> 2] = acc;
    CFENCE();
    u32 C2[2], p3 = 0;
    { u32 u[8], c[8], b[8], s2[8];
#pragma unroll
      for (int k = 0; k < 8; ++k) u[k] = r2[k];
#pragma unroll
      for (int k = 0; k < 8; ++k) {
          const u32 big = perm(0u, 0u, ((u[k] & 0xf8f8f8f8u) >> 1) + 0x0c0c0c0cu);
          c[k] = perm(0x03030302u, 0x03030100u, u[k] & 0x07070707u) | (big & 0x03030303u);
          b[k] = dot4(c[k], 0x40100401u, 0u);
          if ((k & 3) == 0) C2[k >> 2] = b[k]; else C2[k >> 2] |= b[k] << (8 * (k & 3));
      }
#pragma unroll
      for (int k = 0; k < 8; ++k) s2[k] = CMP[b[k]];
      CFENCE();
      acc = 0;
#pragma unroll
      for (int k = 0; k < 8; ++k)
          stream_append(r3, perm(0u, u[k] ^ 0x02020202u, s2[k]), p3, dot4(perm(0u, 0x01000000u, c[k]), 0x01010101u, p3), acc);
      r3[p3 >> 2] = acc;
      CFENCE(); }
    u32 C3 = 0, p4 = 0;
    { u32 u[4], c[4], b[4], s3[4];
#pragma unroll
      for (int k = 0; k < 4; ++k) u[k] = r3[k];
#pragma unroll
      for (int k = 0; k < 4; ++k) {
          const u32 big = perm(0u, 0u, ((u[k] & 0xfefefefeu) >> 1) + 0x0c0c0c0cu);
          const u32 n14 = perm(0u, 0u, (((u[k] ^ 0x0e0e0e0eu) & 0x7f7f7f7fu) + 0x0c0c0c0cu) | (u[k] & 0x80808080u));
          c[k] = (u[k] & 0x01010101u & ~big) | (big & (0x02020202u | (n14 & 0x01010101u)));
          b[k] = dot4(c[k], 0x40100401u, 0u);
          C3 |= b[k] << (8 * k);
      }
#pragma unroll
      for (int k = 0; k < 4; ++k) s3[k] = CMP[b[k]];
      CFENCE();
      acc = 0;
#pragma unroll
      for (int k = 0; k < 4; ++k) {
          const u32 pk = perm(0u, u[k] ^ 0x06060606u, s3[k]);
          stream_append(r4, pk, p4, dot4(perm(0u, 0x01000000u, c[k]), 0x01010101u, p4), acc);
      }
      r4[p4 >> 2] = acc;
      CFENCE(); }
    const bool over = p2 > 31u || p3 > 15u || p4 > 8u;
    if (__any(over)) return 0x80000000u;
    const u32 q = p2 | p3 << 11 | p4 << 21, qi = wave_incl_scan(q), qe = qi - q;
    const u32 tot = (u32)__builtin_amdgcn_readlane((int)qi, 63);
    const u32 o3 = ((2 * (tot & 0x7ffu) + 31u) >> 5) * 4u, o4 = o3 + ((2 * ((tot >> 11) & 0x3ffu) + 31u) >> 5) * 4u, vlen = o4 + (tot >> 21);
    if (vlen > 1024u) return 0x80000000u;
    LAS u32* V = (LAS u32*)(sc + CD_V);
    { const u32 bit = 2 * (qe & 0x7ffu), sh = bit & 31u; LAS u32* a = V + (bit >> 5);
      const u64 w = ((u64)C2[1] << 32 | C2[0]) << sh;
      __hip_atomic_fetch_or(a, (u32)w, __ATOMIC_RELAXED, __HIP_MEMORY_SCOPE_WAVEFRONT);
      __hip_atomic_fetch_or(a + 1, (u32)(w >> 32), __ATOMIC_RELAXED, __HIP_MEMORY_SCOPE_WAVEFRONT);
      __hip_atomic_fetch_or(a + 2, (C2[1] >> 1) >> (31u - sh), __ATOMIC_RELAXED, __HIP_MEMORY_SCOPE_WAVEFRONT); }
    { const u32 bit = 8 * o3 + 2 * ((qe >> 11) & 0x3ffu), sh = bit & 31u; LAS u32* a = V + (bit >> 5);
      const u64 w = (u64)C3 << sh;
      __hip_atomic_fetch_or(a, (u32)w, __ATOMIC_RELAXED, __HIP_MEMORY_SCOPE_WAVEFRONT);
      __hip_atomic_fetch_or(a + 1, (u32)(w >> 32), __ATOMIC_RELAXED, __HIP_MEMORY_SCOPE_WAVEFRONT); }
    { const u32 r0 = r4[0], r1 = r4[1];
      const u32 bit = 8 * (o4 + (qe >> 21)), sh = bit & 31u; LAS u32* a = V + (bit >> 5);
      const u64 w = ((u64)r1 << 32 | r0) << sh;
      __hip_atomic_fetch_or(a, (u32)w, __ATOMIC_RELAXED, __HIP_MEMORY_SCOPE_WAVEFRONT);
      __hip_atomic_fetch_or(a + 1, (u32)(w >> 32), __ATOMIC_RELAXED, __HIP_MEMORY_SCOPE_WAVEFRONT);
      __hip_atomic_fetch_or(a + 2, (r1 >> 1) >> (31u - sh), __ATOMIC_RELAXED, __HIP_MEMORY_SCOPE_WAVEFRONT); }
    CFENCE();
    return B | vlen << 8;
}

__device__ __forceinline__ void codec_decode(u32 (&x)[32], const u32 (&A)[16], const u32 (&L1)[4], u32 B, const LAS u32* V, lds_u8* sc, const lds_u8* tab, int lane) {
    const LAS u32* DEP = (const LAS u32*)tab;
    LAS u32* r2 = (LAS u32*)(sc + CD_DEC_R2) + lane * 9; LAS u32* r3 = (LAS u32*)(sc + CD_DEC_R3) + lane * 5;
    const u32 T1 = ((B - 1u) & 255u) | ((B - 2u) & 255u) << 8 | ((B - 3u) & 255u) << 16;
    const u32 T2 = ((B - 4u) & 255u) | ((B - 5u) & 255u) << 8 | (B & 255u) << 16;
    const u32 T3 = ((B - 6u) & 255u) | ((B - 7u) & 255u) << 8 | ((B - 8u) & 255u) << 16;
    u32 e1[4], n2 = 0;
#pragma unroll
    for (int k = 0; k < 4; ++k) { e1[k] = L1[k] & (L1[k] >> 1) & 0x55555555u; n2 += __builtin_popcount(e1[k]); }
    const u32 s2i = wave_incl_scan(n2), s2 = s2i - n2, tot2 = (u32)__builtin_amdgcn_readlane((int)s2i, 63);
    u32 d1[16];
#pragma unroll
    for (int g = 0; g < 16; ++g) d1[g] = DEP[(L1[g >> 2] >> (8 * (g & 3))) & 255u];
    u32 C2[2];
    { const u32 bit = 2 * s2, sh = bit & 31u; const LAS u32* a = V + (bit >> 5);
      const u32 d0 = a[0], dd1 = a[1], d2 = a[2];
      const u64 m = (1ull << (2 * n2)) - 1ull;
      C2[0] = __builtin_amdgcn_alignbit(dd1, d0, sh) & (u32)m; C2[1] = __builtin_amdgcn_alignbit(d2, dd1, sh) & (u32)(m >> 32); }
    u32 e2[2]; e2[0] = C2[0] & (C2[0] >> 1) & 0x55555555u; e2[1] = C2[1] & (C2[1] >> 1) & 0x55555555u;
    const u32 n3 = __builtin_popcount(e2[0]) + __builtin_popcount(e2[1]);
    const u32 s3i = wave_incl_scan(n3), s3 = s3i - n3, tot3 = (u32)__builtin_amdgcn_readlane((int)s3i, 63);
    u32 dp2[8];
#pragma unroll
    for (int k = 0; k < 8; ++k) dp2[k] = DEP[(C2[k >> 2] >> (8 * (k & 3))) & 255u];
    const u32 o3 = ((2 * tot2 + 31u) >> 5) * 4u, o4 = o3 + ((2 * tot3 + 31u) >> 5) * 4u;
    u32 C3;
    { const u32 bit = 8 * o3 + 2 * s3, sh = bit & 31u; const LAS u32* a = V + (bit >> 5);
      C3 = __builtin_amdgcn_alignbit(a[1], a[0], sh) & ((1u << (2 * n3)) - 1u); }
    const u32 e3 = C3 & (C3 >> 1) & 0x55555555u, n4 = __builtin_popcount(e3);
    const u32 s4 = wave_incl_scan(n4) - n4;
    u32 dp3[4];
#pragma unroll
    for (int k = 0; k < 4; ++k) dp3[k] = DEP[(C3 >> (8 * k)) & 255u];
    u64 R4;
    { const u32 byte = o4 + s4, sh = (byte & 3u) * 8u; const LAS u32* a = V + (byte >> 2);
      const u32 d0 = a[0], dd1 = a[1], d2 = a[2];
      u32 t0 = __builtin_amdgcn_alignbit(dd1, d0, sh), t1 = __builtin_amdgcn_alignbit(d2, dd1, sh);
      const u32 B4 = B * 0x01010101u;
      t0 = ((B4 | 0x80808080u) - (t0 & 0x7f7f7f7fu)) ^ ((B4 ^ ~t0) & 0x80808080u);
      t1 = ((B4 | 0x80808080u) - (t1 & 0x7f7f7f7fu)) ^ ((B4 ^ ~t1) & 0x80808080u);
      R4 = (u64)t1 << 32 | t0; }
#pragma unroll
    for (int k = 0; k < 4; ++k) {
        const u32 p = k == 0 ? 0u : (u32)__builtin_popcount(e3 & ((1u << (8 * k)) - 1u));
        r3[k] = perm(T3, (u32)(R4 >> (8 * p)), dp3[k]);
    }
    CFENCE();
    { u32 ov[8];
#pragma unroll
      for (int k = 0; k < 8; ++k) {
          u32 p = (k & 3) == 0 ? 0u : (u32)__builtin_popcount(e2[k >> 2] & ((1u << (8 * (k & 3))) - 1u));
          if (k >= 4) p += __builtin_popcount(e2[0]);
          const LAS u32* a = r3 + (p >> 2);
          ov[k] = perm(T2, __builtin_amdgcn_alignbyte(a[1], a[0], p), dp2[k]);
      }
      CFENCE();
#pragma unroll
      for (int k = 0; k < 8; ++k) r2[k] = ov[k]; }
    CFENCE();
    u32 pb = 0;
#pragma unroll
    for (int g = 0; g < 16; ++g) {
        const u32 p = pb + ((g & 3) == 0 ? 0u : (u32)__builtin_popcount(e1[g >> 2] & ((1u << (8 * (g & 3))) - 1u)));
        const LAS u32* a = r2 + (p >> 2);
        const u32 E = perm(T1, __builtin_amdgcn_alignbyte(a[1], a[0], p), d1[g]);
        const u32 y0 = perm(E, A[g], 0x05010400u), y1 = perm(E, A[g], 0x07030602u);
        x[2 * g] = __builtin_amdgcn_alignbit(y0, y0, 1); x[2 * g + 1] = __builtin_amdgcn_alignbit(y1, y1, 1);
        if ((g & 3) == 3) pb += __builtin_popcount(e1[g >> 2]);
    }
    CFENCE();
}

constexpr int NR = 8;
constexpr int MM = 4096, NN = 4096, KSH = 512;
constexpr int NWG = 256, NTH = 512;
constexpr int PIECE_B = 8192;
constexpr int NINA = 224, NIN = NINA + 128, NOUT = 512, NQ = NIN + NOUT;
#define ND 8
#define WIN 2
constexpr int NSLOT = 14;
constexpr int NPW = (NQ + ND - 1) / ND;
#define RAWT 12
#define PFD 0
constexpr int NST = 6;
constexpr int STAGE_B = 24576;
constexpr size_t OFF_CALL = 0;
constexpr size_t OFF_TMO = 32768;
constexpr size_t OFF_CNT = 65536;
constexpr size_t OFF_PROF = 131072;
constexpr size_t OFF_FLAG = 1u << 20;
constexpr size_t OFF_OBI = 2u << 20;
constexpr size_t OFF_OBO = OFF_OBI + 1024ull * PIECE_B;
constexpr size_t OFF_XS = OFF_OBO + 512ull * PIECE_B;
constexpr size_t OFF_WS = OFF_XS + 64ull * 32 * PIECE_B;
constexpr size_t WS_NEED = OFF_WS + 64ull * 16 * PIECE_B;
static_assert(OFF_FLAG + (size_t)NR * ND * 256 * 8 <= OFF_OBI && NPW <= 256, "flags");
__host__ __device__ constexpr size_t flag_off(int src, int idx) { return OFF_FLAG + ((size_t)(src * ND + idx % ND) * 256 + idx / ND) * 8; }
constexpr int LDS_TAB = 0, LDS_ROLE = 2048;
constexpr int LDS_MISC = LDS_ROLE + NST * STAGE_B;
constexpr int LDS_BYTES = LDS_MISC + 512;
constexpr int DEC_SCR = CD_DEC_BYTES - 1040;
constexpr int LDS_SLOTS = LDS_ROLE, LDS_DSCR = LDS_SLOTS + NSLOT * PIECE_B, LDS_CTL = LDS_DSCR + 6 * DEC_SCR;
static_assert(LDS_CTL + 1024 + 64 + 64 + 16 <= LDS_MISC, "decoder layout");
static_assert(LDS_ROLE + 8 * CD_BYTES <= LDS_MISC && ND == 8, "phase 0 scratch; 32 pushers per XCD");
static_assert(16384 + 2 * CD_BYTES <= NST * STAGE_B, "epilogue scratch");
constexpr u64 DEADLINE = 40000000ull;

struct Params {
    const unsigned short* x;
    const unsigned short* w;
    unsigned short* out;
    unsigned char* ws[NR];
    int rank;
    int pad_;
};

typedef short bf16x8 __attribute__((ext_vector_type(8)));
typedef float f32x4 __attribute__((ext_vector_type(4)));
typedef short s16x4 __attribute__((ext_vector_type(4)));
#define GAS __attribute__((address_space(1)))

__device__ __forceinline__ void st_sys(void* p, u32x4 v) { asm volatile("global_store_dwordx4 %0, %1, off sc0 sc1\n\ts_nop 1" :: "v"(p), "v"(v) : "memory"); }
__device__ __forceinline__ void st_agent(void* p, u32x4 v) { asm volatile("global_store_dwordx4 %0, %1, off sc1\n\ts_nop 1" :: "v"(p), "v"(v) : "memory"); }
constexpr int wcnt(int vm, int lgkm) { return (vm & 15) | 0x70 | (lgkm << 8) | ((vm >> 4) << 14); }
__device__ __forceinline__ void wait_vm0() { asm volatile("s_waitcnt vmcnt(0)" ::: "memory"); }
__device__ __forceinline__ u32 rfl(u32 v) { return (u32)__builtin_amdgcn_readfirstlane((int)v); }
__device__ __forceinline__ u64 now() { return __builtin_amdgcn_s_memrealtime(); }

#define STAMP(slot) do { } while (0)

__device__ __forceinline__ void in_entry(int idx, int& c, bool& isw, int& bl) {
    if (idx < NINA) { c = 1 + (idx >> 5); isw = (idx & 31) < 16; bl = idx & 15; }
    else { const int e = idx - NINA; c = (e >> 3) & 7; isw = false; bl = 16 + ((e >> 6) << 3) + (e & 7); }
}
__device__ __forceinline__ bool wait_cnt2(const unsigned char* c0, u32 t0v, const unsigned char* c1, u32 t1v, u32 call, u32* tmo) {
    const u64 t0 = now();
    for (;;) {
        const u32 a = __hip_atomic_load((const GAS u32*)c0, __ATOMIC_RELAXED, __HIP_MEMORY_SCOPE_SYSTEM);
        const u32 b = __hip_atomic_load((const GAS u32*)c1, __ATOMIC_RELAXED, __HIP_MEMORY_SCOPE_SYSTEM);
        if (rfl(a) >= t0v && rfl(b) >= t1v) return true;
        if (rfl(__hip_atomic_load((const GAS u32*)tmo, __ATOMIC_RELAXED, __HIP_MEMORY_SCOPE_AGENT)) == call || now() - t0 > DEADLINE) {
            __hip_atomic_store((GAS u32*)tmo, call, __ATOMIC_RELAXED, __HIP_MEMORY_SCOPE_AGENT);
            return false;
        }
        __builtin_amdgcn_s_sleep(2);
    }
}

__device__ __forceinline__ u32 encode_store(const u32 (&x)[32], unsigned char* slot, lds_u8* sc, const lds_u8* tab, int lane) {
    u32 A[16], L1[4];
    const u32 inf = codec_encode(x, A, L1, sc, tab, lane);
    unsigned char* o = slot + lane * 16;
    if (inf >> 31) {
#pragma unroll
        for (int j = 0; j < 8; ++j) st_sys(o + j * 1024, (u32x4){x[4 * j], x[4 * j + 1], x[4 * j + 2], x[4 * j + 3]});
    } else {
#pragma unroll
        for (int j = 0; j < 4; ++j) st_sys(o + j * 1024, (u32x4){A[4 * j], A[4 * j + 1], A[4 * j + 2], A[4 * j + 3]});
        st_sys(o + 4096, (u32x4){L1[0], L1[1], L1[2], L1[3]});
        const u32x4 v = *lp<u32x4>(sc + CD_V + lane * 16);
        if ((u32)lane * 16u < ((inf >> 8) & 0x7ffu)) st_sys(o + 5120, v);
    }
    return inf;
}

__device__ __forceinline__ u32 pk_bf16(float lo, float hi) { u32 r; asm("v_cvt_pk_bf16_f32 %0, %1, %2" : "=v"(r) : "v"(lo), "v"(hi)); return r; }

__global__ __launch_bounds__(NTH, 2) void gemm_ar_kernel(Params P) {
    extern __shared__ __attribute__((aligned(16))) unsigned char smem[];
    lds_u8* sm = (lds_u8*)smem;
    const int tid = threadIdx.x, lane = tid & 63, wid = (int)rfl(tid >> 6);
    const int b = blockIdx.x, xcd = b & 7, q = b >> 3;
    const int rank = P.rank, ri = rank >> 2, rj = rank & 3;
    unsigned char* my = P.ws[rank];
    u32* tmo = (u32*)(my + OFF_TMO);
    const lds_u8* tab = sm + LDS_TAB;
    STAMP(0);
    u32 call_v = 0;
    if (lane == 0) call_v = __hip_atomic_fetch_add((GAS u32*)(my + OFF_CALL + b * 128 + wid * 4), 1u, __ATOMIC_RELAXED, __HIP_MEMORY_SCOPE_AGENT) + 1u;

    unsigned char* wsl = P.ws[lane & 7];

    const bool is_dec = q >= 16 && q < 16 + ND && xcd < NR - 1;
    const int peer_r = (rank + 1 + xcd) & 7;
    const bool pusher = wid >= 4 && is_dec;
    const int pi = (q - 16) * 4 + (wid - 4);
    const int pid = pusher ? (pi < 16 ? 512 + (16 * (peer_r & 3) + pi) * 8 : (32 * (peer_r >> 2) + pi - 16) * 8) : b * 4 + wid;
    const bool enc = wid < 4 || pusher;
    const bool isw = pid >= 512;
    const int blk = (pid & 511) >> 3, ch = pid & 7;
    u32x4 raw[8];
    if (enc) {
        if (!isw) {
            const unsigned short* s = P.x + (size_t)(blk * 64 + (lane & 15)) * KSH + ch * 64 + (lane >> 4) * 8;
#pragma unroll
            for (int j = 0; j < 8; ++j) raw[j] = *(const u32x4*)(s + (size_t)(j >> 1) * 16 * KSH + (j & 1) * 32);
        } else {
            const unsigned short* s = P.w + (size_t)(ch * 64 + (lane >> 3)) * NN + blk * 64 + (lane & 7) * 8;
#pragma unroll
            for (int j = 0; j < 8; ++j) raw[j] = *(const u32x4*)(s + (size_t)j * 8 * NN);
        }
    }
    if (tid < 256) codec_build_tables(sm + LDS_TAB, tid);
    __syncthreads();
    const u32 call = rfl(call_v);
    if (enc) {
        lds_u8* sc = sm + LDS_ROLE + wid * CD_BYTES;
        u32 x[32];
        if (!isw) {
#pragma unroll
            for (int j = 0; j < 8; ++j) { x[4 * j] = raw[j].x; x[4 * j + 1] = raw[j].y; x[4 * j + 2] = raw[j].z; x[4 * j + 3] = raw[j].w; }
        } else {
#pragma unroll
            for (int j = 0; j < 8; ++j) { const int k = 8 * j + (lane >> 3); *lp<u32x4>(sc + k * 128 + (((lane & 7) ^ (k & 7)) << 4)) = raw[j]; }
            const int g = lane >> 4, qq = (lane & 15) >> 2, pp = lane & 3;
#pragma unroll
            for (int j = 0; j < 8; ++j) {
                const int ni = j >> 1, ks = j & 1;
#pragma unroll
                for (int t = 0; t < 2; ++t) {
                    const int k = 32 * ks + 8 * g + 4 * t + qq, chunk = 2 * ni + (pp >> 1);
                    const s16x4 v = __builtin_amdgcn_ds_read_tr16_b64_v4i16((LAS s16x4*)(sc + k * 128 + ((chunk ^ (k & 7)) << 4) + (pp & 1) * 8));
                    const u32x2 u = __builtin_bit_cast(u32x2, v);
                    x[4 * j + 2 * t] = u.x; x[4 * j + 2 * t + 1] = u.y;
                }
            }
            asm volatile("s_waitcnt lgkmcnt(0)" ::: "memory");
        }
        STAMP(1);
        const int rb = blk & 31, step = rank * 8 + ch;
        const size_t stg = isw ? OFF_WS + (size_t)(step * 16 + (blk & 15)) * PIECE_B : OFF_XS + (size_t)(step * 32 + rb) * PIECE_B;
        const size_t cnt = OFF_CNT + (size_t)(isw ? 128 + (blk & 15) * 8 + ch : (rb >> 1) * 8 + ch) * 128;
        const bool cons = lane < NR && (isw ? (lane & 3) == (blk >> 4) : (lane >> 2) == (blk >> 5));
        const bool mine = isw ? ((blk >> 4) == rj) : ((blk >> 5) == ri);
        if (pusher) {
            unsigned char* d = P.ws[peer_r] + stg + lane * 16;
#pragma unroll
            for (int j = 0; j < 8; ++j) st_sys(d + j * 1024, (u32x4){x[4 * j], x[4 * j + 1], x[4 * j + 2], x[4 * j + 3]});
            wait_vm0();
            STAMP(2);
            if (lane == peer_r) __hip_atomic_fetch_add((GAS u32*)(wsl + cnt), 1u, __ATOMIC_RELAXED, __HIP_MEMORY_SCOPE_SYSTEM);
        } else if (ch == 0 && (isw || rb < 16)) {
            if (mine) {
                unsigned char* d = my + stg + lane * 16;
#pragma unroll
                for (int j = 0; j < 8; ++j) st_sys(d + j * 1024, (u32x4){x[4 * j], x[4 * j + 1], x[4 * j + 2], x[4 * j + 3]});
                wait_vm0();
                if (lane == 0) __hip_atomic_fetch_add((GAS u32*)(my + cnt), 1u, __ATOMIC_RELAXED, __HIP_MEMORY_SCOPE_SYSTEM);
            }
        } else {
            const u32 inf = encode_store(x, my + OFF_OBI + (size_t)pid * PIECE_B, sc, tab, lane);
            wait_vm0();
            STAMP(2);
            const int idx = isw ? (ch - 1) * 32 + (blk & 15) : (rb < 16 ? (ch - 1) * 32 + 16 + rb : NINA + ((rb - 16) >> 3) * 64 + ch * 8 + (rb & 7));
            if (cons && lane != rank) __hip_atomic_store((GAS u64*)(wsl + flag_off(rank, idx)), (u64)call << 32 | inf, __ATOMIC_RELAXED, __HIP_MEMORY_SCOPE_SYSTEM);
            if (mine) {
                unsigned char* d = my + stg + lane * 16;
#pragma unroll
                for (int j = 0; j < 8; ++j) st_agent(d + j * 1024, (u32x4){x[4 * j], x[4 * j + 1], x[4 * j + 2], x[4 * j + 3]});
                wait_vm0();
                if (lane == 0) __hip_atomic_fetch_add((GAS u32*)(my + cnt), 1u, __ATOMIC_RELAXED, __HIP_MEMORY_SCOPE_AGENT);
            }
        }
        STAMP(3);
    }

    if (q < 16 || q >= 24) {
        __syncthreads();
        lds_u8* ring = sm + LDS_ROLE;
        const int wr = wid >> 1, wc = wid & 1;
        const int cb = 4 * (xcd & 3) + (q & 3);
        bool bad = false;
        const int ntile = q < 8 ? 2 : 1;
        for (int t = 0; t < ntile; ++t) {
            const int a = q >= 24 ? 8 + 2 * (xcd >> 2) + ((q >> 2) & 1) : (t == 0 ? 4 * (xcd >> 2) + (q >> 2) : 12 + 2 * (xcd >> 2) + (q >> 2));
            const int g = a >> 3;
            const unsigned char* gx = my + OFF_XS + (size_t)(2 * a) * PIECE_B;
            const unsigned char* gw = my + OFF_WS + (size_t)cb * PIECE_B;
            const bool ldA = wid < 4;
            const unsigned char* gl = (ldA ? gx + wid * 4096 : gw + (wid - 4) * 4096) + lane * 16;
            const size_t lstep = ldA ? (size_t)32 * PIECE_B : (size_t)16 * PIECE_B;
            f32x4 acc[2][2] = {};
            auto issue = [&](int i) {
                const int step = (i & 7) * 8 + (i >> 3);
                lds_u8* d = ring + (i % NST) * STAGE_B + wid * 4096;
                const unsigned char* sa = gl + (size_t)step * lstep;
#pragma unroll
                for (int j = 0; j < 4; ++j) __builtin_amdgcn_global_load_lds((const GAS u32*)(sa + j * 1024), (LAS u32*)(d + j * 1024), 16, 0, 17);
            };
            const unsigned char* cx = my + OFF_CNT + (size_t)(a * 8) * 128; const unsigned char* cw = my + OFF_CNT + (size_t)(128 + cb * 8) * 128;
            volatile LAS u32* rdyw = (volatile LAS u32*)(sm + LDS_MISC + 256);
            int rdy = 0;
            u32 pa = 0, pb = 0; bool pend = false;
            auto poll_issue = [&]() { pa = __hip_atomic_load((const GAS u32*)(cx + rdy * 128), __ATOMIC_RELAXED, __HIP_MEMORY_SCOPE_SYSTEM); pb = __hip_atomic_load((const GAS u32*)(cw + rdy * 128), __ATOMIC_RELAXED, __HIP_MEMORY_SCOPE_SYSTEM); pend = true; };
            auto poll_take = [&]() { if (pend) { pend = false; if (rfl(pa) >= 16u * call && rfl(pb) >= 8u * call) { ++rdy; if (lane == 0) *rdyw = rdy; } } };
            auto need = [&](int c) {
                poll_take();
                while (rdy <= c) {
                    if (!wait_cnt2(cx + rdy * 128, 16u * call, cw + rdy * 128, 8u * call, call, tmo)) bad = true;
                    ++rdy; if (lane == 0) *rdyw = rdy;
                }
            };
            int tp = NST;
            const unsigned char* gt = lane < 32 ? gx + (q & 3) * 4096 + lane * 128 : gw + ((q >> 2) & 3) * 2048 + (lane & 15) * 128;
            const size_t tstep = lane < 32 ? (size_t)32 * PIECE_B : (size_t)16 * PIECE_B;
            auto warm = [&](int i) {
                const int lim = min(min(64, (int)rfl(*rdyw) * 8), i + PFD);
#pragma unroll 1
                for (int k = 0; k < 3 && tp < lim; ++k, ++tp)
                    __builtin_amdgcn_global_load_lds((const GAS u32*)(gt + (size_t)((tp & 7) * 8 + (tp >> 3)) * tstep), (LAS u32*)(sm + LDS_MISC), 4, 0, 17);
            };
            auto rd = [&](int i, bf16x8 (&af)[2][2], bf16x8 (&bfr)[2][2]) {
                lds_u8* st = ring + (i % NST) * STAGE_B;
#pragma unroll
                for (int m = 0; m < 2; ++m)
#pragma unroll
                    for (int ks = 0; ks < 2; ++ks) af[m][ks] = *lp<bf16x8>(st + (wr >> 1) * 8192 + (((wr & 1) * 2 + m) * 2 + ks) * 1024 + lane * 16);
#pragma unroll
                for (int n = 0; n < 2; ++n)
#pragma unroll
                    for (int ks = 0; ks < 2; ++ks) bfr[n][ks] = *lp<bf16x8>(st + 16384 + ((wc * 2 + n) * 2 + ks) * 1024 + lane * 16);
            };
            auto mm = [&](bf16x8 (&af)[2][2], bf16x8 (&bfr)[2][2]) {
#pragma unroll
                for (int ks = 0; ks < 2; ++ks)
#pragma unroll
                    for (int m = 0; m < 2; ++m)
#pragma unroll
                        for (int n = 0; n < 2; ++n) acc[m][n] = __builtin_amdgcn_mfma_f32_16x16x32_bf16(bfr[n][ks], af[m][ks], acc[m][n], 0, 0, 0);
            };
            auto kloop = [&](auto role_c) {
                constexpr int ROLE = decltype(role_c)::value;
                bf16x8 a0[2][2], b0[2][2], a1[2][2], b1[2][2];
                if constexpr (ROLE == 1) { if (lane == 0) *rdyw = 0; need(0); }
                __builtin_amdgcn_s_barrier();
                if (g == 0) STAMP(4); else STAMP(12);
                if constexpr (ROLE == 0) {
#pragma unroll
                    for (int i = 0; i < NST - 1; ++i) issue(i);
                    __builtin_amdgcn_s_waitcnt(wcnt(4 * (NST - 2), 15));
                }
                __builtin_amdgcn_s_barrier();
                rd(0, a0, b0);
                auto pair = [&](int i, auto tail_c) {
                    constexpr int TAIL = decltype(tail_c)::value;
                    { const int nx = i + NST - 1;
                      if constexpr (ROLE == 0) __builtin_amdgcn_s_waitcnt(wcnt(TAIL == 0 || TAIL + 4 < 64 ? 4 * (NST - 3) : 4 * (63 - TAIL - 1 > 0 ? 63 - TAIL - 1 : 0), 0));
                      else __builtin_amdgcn_s_waitcnt(wcnt(63, 0));
                      __builtin_amdgcn_s_barrier();
                      if constexpr (ROLE == 0 && (TAIL == 0 || TAIL + NST - 1 < 64)) issue(nx);
                      rd(i + 1, a1, b1);
                      __builtin_amdgcn_sched_barrier(0);
                      mm(a0, b0);
                      __builtin_amdgcn_sched_barrier(0);
                      if constexpr (ROLE == 1 && TAIL == 0) { if ((i & 2) == 0 && rdy < 8) { poll_take(); if (rdy < 8) poll_issue(); } }
                      if constexpr (ROLE == 2 && TAIL == 0) warm(i); }
                    { const int nx = i + NST;
                      if constexpr (ROLE == 1 && TAIL == 0) { if ((nx & 7) == 0) need(nx >> 3); }
                      if constexpr (ROLE == 0) __builtin_amdgcn_s_waitcnt(wcnt(TAIL == 0 ? 4 * (NST - 3) : 4 * (63 - TAIL - 2 > 0 ? 63 - TAIL - 2 : 0), 0));
                      else __builtin_amdgcn_s_waitcnt(wcnt(63, 0));
                      __builtin_amdgcn_s_barrier();
                      if constexpr (ROLE == 0 && TAIL == 0) issue(nx);
                      if constexpr (TAIL != 62) rd(i + 2, a0, b0);
                      __builtin_amdgcn_sched_barrier(0);
                      mm(a1, b1);
                      __builtin_amdgcn_sched_barrier(0);
                      if constexpr (ROLE == 2 && TAIL == 0) warm(i + 1); }
                };
#pragma unroll 1
                for (int i = 0; i < 58; i += 2) pair(i, std::integral_constant<int, 0>{});
                pair(58, std::integral_constant<int, 58>{});
                pair(60, std::integral_constant<int, 60>{});
                pair(62, std::integral_constant<int, 62>{});
            };
            if (wid < 6) kloop(std::integral_constant<int, 0>{});
            else if (wid == 6) kloop(std::integral_constant<int, 1>{});
            else kloop(std::integral_constant<int, 2>{});
            wait_vm0();
            if (g == 0) STAMP(5); else STAMP(13);
            __syncthreads();
#pragma unroll
            for (int m = 0; m < 2; ++m)
#pragma unroll
                for (int n = 0; n < 2; ++n) {
                    const int row = 32 * wr + 16 * m + (lane & 15), col = 32 * wc + 16 * n + 4 * (lane >> 4);
                    const u32x2 v = {pk_bf16(acc[m][n][0], acc[m][n][1]), pk_bf16(acc[m][n][2], acc[m][n][3])};
                    *lp<u32x2>(ring + row * 128 + (((col >> 3) ^ (row & 7)) << 4) + (col & 4) * 2) = v;
                }
            if (tid == 6 * 64) *lp<u32>(sm + LDS_MISC + 272) = (bad || __hip_atomic_load((const GAS u32*)tmo, __ATOMIC_RELAXED, __HIP_MEMORY_SCOPE_AGENT) == call) ? 1u : 0u;
            __syncthreads();
            if (wid < 4) {
                const int h = wid & 1;
                const bool poison = *lp<u32>(sm + LDS_MISC + 272) != 0u;
                u32 x[32];
#pragma unroll
                for (int j = 0; j < 8; ++j) {
                    const int row = 64 * h + 8 * j + (lane >> 3);
                    u32x4 v = *lp<u32x4>(ring + row * 128 + (((lane & 7) ^ (row & 7)) << 4));
                    if (poison) v = (u32x4){0x7fc07fc0u, 0x7fc07fc0u, 0x7fc07fc0u, 0x7fc07fc0u};
                    x[4 * j] = v.x; x[4 * j + 1] = v.y; x[4 * j + 2] = v.z; x[4 * j + 3] = v.w;
                }
                const int a8 = a & 7, o = g * 256 + (a8 * 16 + cb) * 2 + h;
                if (wid < 2) {
                    u32 inf;
                    if (o >= NOUT - RAWT) {
                        inf = 0x80000000u;
                        unsigned char* ob = my + OFF_OBO + (size_t)o * PIECE_B + lane * 16;
#pragma unroll
                        for (int j = 0; j < 8; ++j) st_sys(ob + j * 1024, (u32x4){x[4 * j], x[4 * j + 1], x[4 * j + 2], x[4 * j + 3]});
                    } else inf = encode_store(x, my + OFF_OBO + (size_t)o * PIECE_B, ring + 16384 + wid * CD_BYTES, tab, lane);
                    wait_vm0();
                    if (lane < NR && lane != rank)
                        __hip_atomic_store((GAS u64*)(wsl + flag_off(rank, NIN + o)), (u64)call << 32 | inf, __ATOMIC_RELAXED, __HIP_MEMORY_SCOPE_SYSTEM);
                } else {
                    unsigned short* d = P.out + (size_t)(2048 * ri + 128 * a + 64 * h + (lane >> 3)) * NN + 1024 * rj + 64 * cb + (lane & 7) * 8;
#pragma unroll
                    for (int j = 0; j < 8; ++j) st_agent(d + (size_t)j * 8 * NN, (u32x4){x[4 * j], x[4 * j + 1], x[4 * j + 2], x[4 * j + 3]});
                }
            }
            if (g == 0) STAMP(6); else STAMP(14);
            __syncthreads();
        }
    } else if (is_dec) {
        const int src = peer_r, si = src >> 2, sj = src & 3;
        const unsigned char* peer = P.ws[src];
        const int dj = q - 16;
        const int np = (NQ - dj + ND - 1) / ND;
        volatile LAS u32* c_info = (volatile LAS u32*)(sm + LDS_CTL);
        volatile LAS u32* c_full = (volatile LAS u32*)(sm + LDS_CTL + 1024);
        volatile LAS u32* c_done = (volatile LAS u32*)(sm + LDS_CTL + 1088);
        LAS u32* c_ticket = (LAS u32*)(sm + LDS_CTL + 1152);
        __syncthreads();
        if (tid < 256) c_info[tid] = 0;
        if (tid < 16) { c_full[tid] = 0; c_done[tid] = 0; }
        if (tid == 0) *c_ticket = 0;
        __syncthreads();
        auto src_slot = [&](int idx) -> const unsigned char* {
            if (idx >= NIN) return peer + OFF_OBO + (size_t)(idx - NIN) * PIECE_B;
            int c, bl; bool isw2; in_entry(idx, c, isw2, bl);
            return peer + OFF_OBI + (size_t)(isw2 ? 512 + (16 * rj + bl) * 8 + c : (32 * ri + bl) * 8 + c) * PIECE_B;
        };
        if (wid == 0) {
            const unsigned char* fb = my + OFF_FLAG + (size_t)(src * ND + dj) * 256 * 8;
            const u64 t0 = now();
            for (int base = 0; base < np; base += 64) {
                const int n = base + lane;
                bool posted = n >= np;
                for (;;) {
                    if (!posted) {
                        const u64 v = __hip_atomic_load((const GAS u64*)(fb + (size_t)n * 8), __ATOMIC_RELAXED, __HIP_MEMORY_SCOPE_SYSTEM);
                        if ((u32)(v >> 32) >= call) { c_info[n] = (u32)v | 0x40000000u; posted = true; }
                    }
                    if (__all(posted)) break;
                    if (rfl(__hip_atomic_load((const GAS u32*)tmo, __ATOMIC_RELAXED, __HIP_MEMORY_SCOPE_AGENT)) == call || now() - t0 > DEADLINE) {
                        __hip_atomic_store((GAS u32*)tmo, call, __ATOMIC_RELAXED, __HIP_MEMORY_SCOPE_AGENT);
                        if (!posted) c_info[n] = 0x60000000u;
                        posted = true;
                        for (int m = base + 64 + lane; m < np; m += 64) c_info[m] = 0x60000000u;
                        base = np;
                        break;
                    }
                    __builtin_amdgcn_s_sleep(2);
                }
            }
        } else if (wid == 1) {
            int pub = 0;
            auto publish_upto = [&](int e) { for (; pub < e; ++pub) if (lane == 0) c_full[pub % NSLOT] = pub + 1; };
            for (int n = 0; n < np; ++n) {
                u32 inf = c_info[n];
                const bool slot_free = n < NSLOT || c_done[n % NSLOT] == (u32)(n - NSLOT + 1);
                if (rfl(inf) == 0u || !rfl(slot_free)) {
                    wait_vm0(); publish_upto(n);
                    while (rfl(inf = c_info[n]) == 0u) __builtin_amdgcn_s_sleep(1);
                    if (n >= NSLOT) while (rfl(c_done[n % NSLOT]) != (u32)(n - NSLOT + 1)) __builtin_amdgcn_s_sleep(1);
                }
                inf = rfl(inf);
                lds_u8* d = sm + LDS_SLOTS + (n % NSLOT) * PIECE_B;
                const unsigned char* s0 = src_slot(dj + n * ND);
                const unsigned char* s = s0 + lane * 16;
                if (inf & 0x20000000u) { wait_vm0(); publish_upto(n + 1); continue; }
                if (inf >> 31) {
                    wait_vm0(); publish_upto(n);
#pragma unroll
                    for (int j = 0; j < 8; ++j) __builtin_amdgcn_global_load_lds((const GAS u32*)(s + j * 1024), (LAS u32*)(d + j * 1024), 16, 0, 17);
                    wait_vm0(); publish_upto(n + 1);
                } else {
#pragma unroll
                    for (int j = 0; j < 5; ++j) __builtin_amdgcn_global_load_lds((const GAS u32*)(s + j * 1024), (LAS u32*)(d + j * 1024), 16, 0, 17);
                    const unsigned char* sv = ((u32)lane * 16u < ((inf >> 8) & 0x7ffu)) ? s + 5120 : s0 + 5120;
                    __builtin_amdgcn_global_load_lds((const GAS u32*)sv, (LAS u32*)(d + 5120), 16, 0, 17);
                    asm volatile("s_waitcnt vmcnt(%0)" :: "n"(6 * (WIN - 1)) : "memory");
                    publish_upto(n + 2 - WIN);
                }
            }
            wait_vm0(); publish_upto(np);
            STAMP(9);
        } else {
            lds_u8* sc = sm + LDS_DSCR + (wid - 2) * DEC_SCR - 1040;
            int nstamp = 0;
            for (;;) {
                u32 nn = 0;
                if (lane == 0) nn = __hip_atomic_fetch_add(c_ticket, 1u, __ATOMIC_RELAXED, __HIP_MEMORY_SCOPE_WORKGROUP);
                const int n = (int)rfl(nn);
                if (n >= np) break;
                while (rfl(c_full[n % NSLOT]) != (u32)(n + 1)) __builtin_amdgcn_s_sleep(1);
                asm volatile("" ::: "memory");
                const u32 inf = rfl(c_info[n]);
                const bool ok = !(inf & 0x20000000u);
                lds_u8* sl = sm + LDS_SLOTS + (n % NSLOT) * PIECE_B;
                u32 x[32];
                if (!ok) {
#pragma unroll
                    for (int j = 0; j < 32; ++j) x[j] = 0x7fc07fc0u;
                } else if (inf >> 31) {
#pragma unroll
                    for (int j = 0; j < 8; ++j) { const u32x4 v = *lp<u32x4>(sl + j * 1024 + lane * 16); x[4 * j] = v.x; x[4 * j + 1] = v.y; x[4 * j + 2] = v.z; x[4 * j + 3] = v.w; }
                } else {
                    u32 A[16], L1[4];
#pragma unroll
                    for (int j = 0; j < 4; ++j) { const u32x4 v = *lp<u32x4>(sl + j * 1024 + lane * 16); A[4 * j] = v.x; A[4 * j + 1] = v.y; A[4 * j + 2] = v.z; A[4 * j + 3] = v.w; }
                    { const u32x4 v = *lp<u32x4>(sl + 4096 + lane * 16); L1[0] = v.x; L1[1] = v.y; L1[2] = v.z; L1[3] = v.w; }
                    codec_decode(x, A, L1, inf & 255u, (const LAS u32*)(sl + 5120), sc, tab, lane);
                }
                asm volatile("s_waitcnt lgkmcnt(0)" ::: "memory");
                if (lane == 0) c_done[n % NSLOT] = n + 1;
                const int idx = dj + n * ND;
                if (idx < NIN) {
                    int c, bl; bool isw2; in_entry(idx, c, isw2, bl);
                    const int step = src * 8 + c;
                    unsigned char* d = (isw2 ? my + OFF_WS + (size_t)(step * 16 + bl) * PIECE_B : my + OFF_XS + (size_t)(step * 32 + bl) * PIECE_B) + lane * 16;
                    if (ok) {
#pragma unroll
                        for (int j = 0; j < 8; ++j) st_agent(d + j * 1024, (u32x4){x[4 * j], x[4 * j + 1], x[4 * j + 2], x[4 * j + 3]});
                        wait_vm0();
                        if (lane == 0) __hip_atomic_fetch_add((GAS u32*)(my + OFF_CNT + (size_t)(isw2 ? 128 + bl * 8 + c : (bl >> 1) * 8 + c) * 128), 1u, __ATOMIC_RELAXED, __HIP_MEMORY_SCOPE_AGENT);
                    }
                } else {
                    const int o = idx - NIN, g = o >> 8, tile = (o & 255) >> 1, h = o & 1;
                    unsigned short* d = P.out + (size_t)(2048 * si + 1024 * g + 128 * (tile >> 4) + 64 * h + (lane >> 3)) * NN + 1024 * sj + 64 * (tile & 15) + (lane & 7) * 8;
#pragma unroll
                    for (int j = 0; j < 8; ++j) st_agent(d + (size_t)j * 8 * NN, (u32x4){x[4 * j], x[4 * j + 1], x[4 * j + 2], x[4 * j + 3]});
                }
            }
            (void)nstamp;
        }
    }
    STAMP(31);
}

extern "C" void kernel_launch(int rank, int nranks, ncclComm_t comm, hipStream_t stream,
                              void* const* d_in, const int* in_sizes, int n_in,
                              void* d_out, int out_size, const int* in_dtypes, int out_dtype,
                              void* const* d_out_all, void* const* d_ws_all, size_t ws_bytes) {
    (void)comm; (void)in_dtypes; (void)out_dtype; (void)d_out_all;
    if (nranks != NR || n_in != 2 || rank < 0 || rank >= NR || in_sizes[0] != MM * KSH || in_sizes[1] != KSH * NN || out_size != MM * NN || ws_bytes < WS_NEED) {
        fprintf(stderr, "kernel_launch: unexpected problem shape\n");
        return;
    }
    static bool first = true;
    if (first) {
        hipError_t e = hipFuncSetAttribute((const void*)gemm_ar_kernel, hipFuncAttributeMaxDynamicSharedMemorySize, LDS_BYTES);
        if (e != hipSuccess) { fprintf(stderr, "hipFuncSetAttribute: %s\n", hipGetErrorString(e)); return; }
        first = false;
    }
    Params P;
    P.x = (const unsigned short*)d_in[0];
    P.w = (const unsigned short*)d_in[1];
    P.out = (unsigned short*)d_out;
    for (int p = 0; p < NR; ++p) P.ws[p] = (unsigned char*)d_ws_all[p];
    P.rank = rank;
    P.pad_ = 0;
    hipLaunchKernelGGL(gemm_ar_kernel, dim3(NWG), dim3(NTH), LDS_BYTES, stream, P);
    hipError_t e = hipGetLastError();
    if (e != hipSuccess) fprintf(stderr, "launch: %s\n", hipGetErrorString(e));
}
```

```cpp
#include <hip/hip_runtime.h>
#include <rccl/rccl.h>
#include <cstdio>
#include <type_traits>

typedef unsigned u32;
typedef unsigned long long u64;
typedef unsigned u32x4 __attribute__((ext_vector_type(4)));
typedef unsigned u32x2 __attribute__((ext_vector_type(2)));
typedef unsigned short u16x2 __attribute__((ext_vector_type(2)));
#define LAS __attribute__((address_space(3)))
typedef LAS unsigned char lds_u8;

constexpr int CD_V = 0;
constexpr int CD_R2 = 1040;
constexpr int CD_R3 = CD_R2 + 64 * 68;
constexpr int CD_R4 = CD_R3 + 64 * 36;
constexpr int CD_BYTES = CD_R4 + 64 * 20;
constexpr int CD_DEC_R2 = 1040, CD_DEC_R3 = CD_DEC_R2 + 64 * 36, CD_DEC_BYTES = CD_DEC_R3 + 64 * 20;
constexpr int CD_TAB_BYTES = 2048;
static_assert(CD_BYTES % 16 == 0 && CD_DEC_BYTES % 16 == 0, "align");

__device__ __forceinline__ u32 perm(u32 hi, u32 lo, u32 sel) { return __builtin_amdgcn_perm(hi, lo, sel); }
__device__ __forceinline__ u32 dot4(u32 a, u32 b, u32 c) { return __builtin_amdgcn_udot4(a, b, c, false); }
__device__ __forceinline__ u32 pk_sub(u32 a, u32 b) { return __builtin_bit_cast(u32, (u16x2)(__builtin_bit_cast(u16x2, a) - __builtin_bit_cast(u16x2, b))); }
__device__ __forceinline__ u32 pk_min(u32 a, u32 b) { return __builtin_bit_cast(u32, __builtin_elementwise_min(__builtin_bit_cast(u16x2, a), __builtin_bit_cast(u16x2, b))); }
__device__ __forceinline__ u32 wave_incl_scan(u32 x) {
    x += (u32)__builtin_amdgcn_update_dpp(0, (int)x, 0x111, 0xf, 0xf, false);
    x += (u32)__builtin_amdgcn_update_dpp(0, (int)x, 0x112, 0xf, 0xf, false);
    x += (u32)__builtin_amdgcn_update_dpp(0, (int)x, 0x114, 0xf, 0xf, false);
    x += (u32)__builtin_amdgcn_update_dpp(0, (int)x, 0x118, 0xf, 0xf, false);
    x += (u32)__builtin_amdgcn_update_dpp(0, (int)x, 0x142, 0xa, 0xf, false);
    x += (u32)__builtin_amdgcn_update_dpp(0, (int)x, 0x143, 0xc, 0xf, false);
    return x;
}
template <class T> __device__ __forceinline__ LAS T* lp(lds_u8* p) { return (LAS T*)p; }
#define CFENCE() asm volatile("" ::: "memory")

__device__ __forceinline__ void codec_build_tables(lds_u8* tab, int i) {
    u32 dep = 0, cmp = 0x0c0c0c0cu; int n = 0;
#pragma unroll
    for (int k = 0; k < 4; ++k) {
        const u32 c = (i >> (2 * k)) & 3;
        if (c == 3) { dep |= (u32)n << (8 * k); cmp = (cmp & ~(0xffu << (8 * n))) | ((u32)k << (8 * n)); ++n; }
        else dep |= (4u + c) << (8 * k);
    }
    lp<u32>(tab)[i] = dep; lp<u32>(tab)[256 + i] = cmp;
}

__device__ __forceinline__ void stream_append(LAS u32* r, u32 pk, u32& p, u32 pnew, u32& acc) {
    const u64 w = (u64)pk << ((p & 3u) * 8u);
    const u32 lo = (u32)w | acc;
    r[p >> 2] = lo;
    acc = ((p ^ pnew) & ~3u) ? (u32)(w >> 32) : lo;
    p = pnew;
}

__device__ __forceinline__ u32 codec_encode(const u32 (&x)[32], u32 (&A)[16], u32 (&L1)[4], lds_u8* sc, const lds_u8* tab, int lane) {
    const LAS u32* CMP = (const LAS u32*)(tab + 1024);
    LAS u32* r2 = (LAS u32*)(sc + CD_R2) + lane * 17; LAS u32* r3 = (LAS u32*)(sc + CD_R3) + lane * 9; LAS u32* r4 = (LAS u32*)(sc + CD_R4) + lane * 5;
    const u32x4 z4 = {0, 0, 0, 0};
#pragma unroll
    for (int k = 0; k < 8; ++k) r2[k] = 0;
#pragma unroll
    for (int k = 0; k < 4; ++k) r3[k] = 0;
    r4[0] = 0; r4[1] = 0;
    *lp<u32x4>(sc + CD_V + lane * 16) = z4; if (lane == 0) *lp<u32x4>(sc + CD_V + 1024) = z4;
    u32 B;
    { const u32 y0 = __builtin_amdgcn_alignbit(x[0], x[0], 31), y1 = __builtin_amdgcn_alignbit(x[1], x[1], 31);
      const u32 s = wave_incl_scan(dot4(perm(y1, y0, 0x07050301u), 0x01010101u, 0u));
      B = ((u32)__builtin_amdgcn_readlane((int)s, 63) + 724u) >> 8; B = B > 255u ? 255u : B; }
    const u32 BK = (B << 8 | 0xffu) * 0x00010001u;
    u32 tb[16], bb[16], pn[16];
    u32 pc = 0;
#pragma unroll
    for (int g = 0; g < 16; ++g) {
        const u32 y0 = __builtin_amdgcn_alignbit(x[2 * g], x[2 * g], 31), y1 = __builtin_amdgcn_alignbit(x[2 * g + 1], x[2 * g + 1], 31);
        A[g] = perm(y1, y0, 0x06040200u);
        const u32 d0 = pk_sub(BK, y0), d1 = pk_sub(BK, y1);
        const u32 m0 = pk_min(d0, 0x07ff07ffu), m1 = pk_min(d1, 0x07ff07ffu);
        const u32 tc = perm(m1, m0, 0x07050301u);
        tb[g] = perm(d1, d0, 0x07050301u) ^ 0x04040404u;
        const u32 c1 = perm(0x03030303u, 0x02010003u, tc), ef = perm(0x01010101u, 0x00000001u, tc);
        bb[g] = dot4(c1, 0x40100401u, 0u);
        pc = dot4(ef, 0x01010101u, pc); pn[g] = pc;
        if ((g & 3) == 0) L1[g >> 2] = bb[g]; else L1[g >> 2] |= bb[g] << (8 * (g & 3));
    }
    u32 sel[16];
#pragma unroll
    for (int g = 0; g < 16; ++g) sel[g] = CMP[bb[g]];
    CFENCE();
    u32 p2 = 0, acc = 0;
#pragma unroll
    for (int g = 0; g < 16; ++g) stream_append(r2, perm(0u, tb[g], sel[g]), p2, pn[g], acc);
    r2[p2 >> 2] = acc;
    CFENCE();
    u32 C2[2], p3 = 0;
    { u32 u[8], c[8], b[8], s2[8];
#pragma unroll
      for (int k = 0; k < 8; ++k) u[k] = r2[k];
#pragma unroll
      for (int k = 0; k < 8; ++k) {
          const u32 big = perm(0u, 0u, ((u[k] & 0xf8f8f8f8u) >> 1) + 0x0c0c0c0cu);
          c[k] = perm(0x03030302u, 0x03030100u, u[k] & 0x07070707u) | (big & 0x03030303u);
          b[k] = dot4(c[k], 0x40100401u, 0u);
          if ((k & 3) == 0) C2[k >> 2] = b[k]; else C2[k >> 2] |= b[k] << (8 * (k & 3));
      }
#pragma unroll
      for (int k = 0; k < 8; ++k) s2[k] = CMP[b[k]];
      CFENCE();
      acc = 0;
#pragma unroll
      for (int k = 0; k < 8; ++k)
          stream_append(r3, perm(0u, u[k] ^ 0x02020202u, s2[k]), p3, dot4(perm(0u, 0x01000000u, c[k]), 0x01010101u, p3), acc);
      r3[p3 >> 2] = acc;
      CFENCE(); }
    u32 C3 = 0, p4 = 0;
    { u32 u[4], c[4], b[4], s3[4];
#pragma unroll
      for (int k = 0; k < 4; ++k) u[k] = r3[k];
#pragma unroll
      for (int k = 0; k < 4; ++k) {
          const u32 big = perm(0u, 0u, ((u[k] & 0xfefefefeu) >> 1) + 0x0c0c0c0cu);
          const u32 n14 = perm(0u, 0u, (((u[k] ^ 0x0e0e0e0eu) & 0x7f7f7f7fu) + 0x0c0c0c0cu) | (u[k] & 0x80808080u));
          c[k] = (u[k] & 0x01010101u & ~big) | (big & (0x02020202u | (n14 & 0x01010101u)));
          b[k] = dot4(c[k], 0x40100401u, 0u);
          C3 |= b[k] << (8 * k);
      }
#pragma unroll
      for (int k = 0; k < 4; ++k) s3[k] = CMP[b[k]];
      CFENCE();
      acc = 0;
#pragma unroll
      for (int k = 0; k < 4; ++k) {
          const u32 pk = perm(0u, u[k] ^ 0x06060606u, s3[k]);
          stream_append(r4, pk, p4, dot4(perm(0u, 0x01000000u, c[k]), 0x01010101u, p4), acc);
      }
      r4[p4 >> 2] = acc;
      CFENCE(); }
    const bool over = p2 > 31u || p3 > 15u || p4 > 8u;
    if (__any(over)) return 0x80000000u;
    const u32 q = p2 | p3 << 11 | p4 << 21, qi = wave_incl_scan(q), qe = qi - q;
    const u32 tot = (u32)__builtin_amdgcn_readlane((int)qi, 63);
    const u32 o3 = ((2 * (tot & 0x7ffu) + 31u) >> 5) * 4u, o4 = o3 + ((2 * ((tot >> 11) & 0x3ffu) + 31u) >> 5) * 4u, vlen = o4 + (tot >> 21);
    if (vlen > 1024u) return 0x80000000u;
    LAS u32* V = (LAS u32*)(sc + CD_V);
    { const u32 bit = 2 * (qe & 0x7ffu), sh = bit & 31u; LAS u32* a = V + (bit >> 5);
      const u64 w = ((u64)C2[1] << 32 | C2[0]) << sh;
      __hip_atomic_fetch_or(a, (u32)w, __ATOMIC_RELAXED, __HIP_MEMORY_SCOPE_WAVEFRONT);
      __hip_atomic_fetch_or(a + 1, (u32)(w >> 32), __ATOMIC_RELAXED, __HIP_MEMORY_SCOPE_WAVEFRONT);
      __hip_atomic_fetch_or(a + 2, (C2[1] >> 1) >> (31u - sh), __ATOMIC_RELAXED, __HIP_MEMORY_SCOPE_WAVEFRONT); }
    { const u32 bit = 8 * o3 + 2 * ((qe >> 11) & 0x3ffu), sh = bit & 31u; LAS u32* a = V + (bit >> 5);
      const u64 w = (u64)C3 << sh;
      __hip_atomic_fetch_or(a, (u32)w, __ATOMIC_RELAXED, __HIP_MEMORY_SCOPE_WAVEFRONT);
      __hip_atomic_fetch_or(a + 1, (u32)(w >> 32), __ATOMIC_RELAXED, __HIP_MEMORY_SCOPE_WAVEFRONT); }
    { const u32 r0 = r4[0], r1 = r4[1];
      const u32 bit = 8 * (o4 + (qe >> 21)), sh = bit & 31u; LAS u32* a = V + (bit >> 5);
      const u64 w = ((u64)r1 << 32 | r0) << sh;
      __hip_atomic_fetch_or(a, (u32)w, __ATOMIC_RELAXED, __HIP_MEMORY_SCOPE_WAVEFRONT);
      __hip_atomic_fetch_or(a + 1, (u32)(w >> 32), __ATOMIC_RELAXED, __HIP_MEMORY_SCOPE_WAVEFRONT);
      __hip_atomic_fetch_or(a + 2, (r1 >> 1) >> (31u - sh), __ATOMIC_RELAXED, __HIP_MEMORY_SCOPE_WAVEFRONT); }
    CFENCE();
    return B | vlen << 8;
}

__device__ __forceinline__ void codec_decode(u32 (&x)[32], const u32 (&A)[16], const u32 (&L1)[4], u32 B, const LAS u32* V, lds_u8* sc, const lds_u8* tab, int lane) {
    const LAS u32* DEP = (const LAS u32*)tab;
    LAS u32* r2 = (LAS u32*)(sc + CD_DEC_R2) + lane * 9; LAS u32* r3 = (LAS u32*)(sc + CD_DEC_R3) + lane * 5;
    const u32 T1 = ((B - 1u) & 255u) | ((B - 2u) & 255u) << 8 | ((B - 3u) & 255u) << 16;
    const u32 T2 = ((B - 4u) & 255u) | ((B - 5u) & 255u) << 8 | (B & 255u) << 16;
    const u32 T3 = ((B - 6u) & 255u) | ((B - 7u) & 255u) << 8 | ((B - 8u) & 255u) << 16;
    u32 e1[4], n2 = 0;
#pragma unroll
    for (int k = 0; k < 4; ++k) { e1[k] = L1[k] & (L1[k] >> 1) & 0x55555555u; n2 += __builtin_popcount(e1[k]); }
    const u32 s2i = wave_incl_scan(n2), s2 = s2i - n2, tot2 = (u32)__builtin_amdgcn_readlane((int)s2i, 63);
    u32 d1[16];
#pragma unroll
    for (int g = 0; g < 16; ++g) d1[g] = DEP[(L1[g >> 2] >> (8 * (g & 3))) & 255u];
    u32 C2[2];
    { const u32 bit = 2 * s2, sh = bit & 31u; const LAS u32* a = V + (bit >> 5);
      const u32 d0 = a[0], dd1 = a[1], d2 = a[2];
      const u64 m = (1ull << (2 * n2)) - 1ull;
      C2[0] = __builtin_amdgcn_alignbit(dd1, d0, sh) & (u32)m; C2[1] = __builtin_amdgcn_alignbit(d2, dd1, sh) & (u32)(m >> 32); }
    u32 e2[2]; e2[0] = C2[0] & (C2[0] >> 1) & 0x55555555u; e2[1] = C2[1] & (C2[1] >> 1) & 0x55555555u;
    const u32 n3 = __builtin_popcount(e2[0]) + __builtin_popcount(e2[1]);
    const u32 s3i = wave_incl_scan(n3), s3 = s3i - n3, tot3 = (u32)__builtin_amdgcn_readlane((int)s3i, 63);
    u32 dp2[8];
#pragma unroll
    for (int k = 0; k < 8; ++k) dp2[k] = DEP[(C2[k >> 2] >> (8 * (k & 3))) & 255u];
    const u32 o3 = ((2 * tot2 + 31u) >> 5) * 4u, o4 = o3 + ((2 * tot3 + 31u) >> 5) * 4u;
    u32 C3;
    { const u32 bit = 8 * o3 + 2 * s3, sh = bit & 31u; const LAS u32* a = V + (bit >> 5);
      C3 = __builtin_amdgcn_alignbit(a[1], a[0], sh) & ((1u << (2 * n3)) - 1u); }
    const u32 e3 = C3 & (C3 >> 1) & 0x55555555u, n4 = __builtin_popcount(e3);
    const u32 s4 = wave_incl_scan(n4) - n4;
    u32 dp3[4];
#pragma unroll
    for (int k = 0; k < 4; ++k) dp3[k] = DEP[(C3 >> (8 * k)) & 255u];
    u64 R4;
    { const u32 byte = o4 + s4, sh = (byte & 3u) * 8u; const LAS u32* a = V + (byte >> 2);
      const u32 d0 = a[0], dd1 = a[1], d2 = a[2];
      u32 t0 = __builtin_amdgcn_alignbit(dd1, d0, sh), t1 = __builtin_amdgcn_alignbit(d2, dd1, sh);
      const u32 B4 = B * 0x01010101u;
      t0 = ((B4 | 0x80808080u) - (t0 & 0x7f7f7f7fu)) ^ ((B4 ^ ~t0) & 0x80808080u);
      t1 = ((B4 | 0x80808080u) - (t1 & 0x7f7f7f7fu)) ^ ((B4 ^ ~t1) & 0x80808080u);
      R4 = (u64)t1 << 32 | t0; }
#pragma unroll
    for (int k = 0; k < 4; ++k) {
        const u32 p = k == 0 ? 0u : (u32)__builtin_popcount(e3 & ((1u << (8 * k)) - 1u));
        r3[k] = perm(T3, (u32)(R4 >> (8 * p)), dp3[k]);
    }
    CFENCE();
    { u32 ov[8];
#pragma unroll
      for (int k = 0; k < 8; ++k) {
          u32 p = (k & 3) == 0 ? 0u : (u32)__builtin_popcount(e2[k >> 2] & ((1u << (8 * (k & 3))) - 1u));
          if (k >= 4) p += __builtin_popcount(e2[0]);
          const LAS u32* a = r3 + (p >> 2);
          ov[k] = perm(T2, __builtin_amdgcn_alignbyte(a[1], a[0], p), dp2[k]);
      }
      CFENCE();
#pragma unroll
      for (int k = 0; k < 8; ++k) r2[k] = ov[k]; }
    CFENCE();
    u32 pb = 0;
#pragma unroll
    for (int g = 0; g < 16; ++g) {
        const u32 p = pb + ((g & 3) == 0 ? 0u : (u32)__builtin_popcount(e1[g >> 2] & ((1u << (8 * (g & 3))) - 1u)));
        const LAS u32* a = r2 + (p >> 2);
        const u32 E = perm(T1, __builtin_amdgcn_alignbyte(a[1], a[0], p), d1[g]);
        const u32 y0 = perm(E, A[g], 0x05010400u), y1 = perm(E, A[g], 0x07030602u);
        x[2 * g] = __builtin_amdgcn_alignbit(y0, y0, 1); x[2 * g + 1] = __builtin_amdgcn_alignbit(y1, y1, 1);
        if ((g & 3) == 3) pb += __builtin_popcount(e1[g >> 2]);
    }
    CFENCE();
}

constexpr int NR = 8;
constexpr int MM = 4096, NN = 4096, KSH = 512;
constexpr int NWG = 256, NTH = 512;
constexpr int PIECE_B = 8192;
#define NPUSH 16
constexpr int GA = 12;
constexpr int CHP = 16 + GA;
constexpr int NINA = 8 * CHP - NPUSH, NIN = NINA + (32 - GA) * 8, NOUT = 512, NQ = NIN + NOUT;
#define ND 8
#define NFL 9
constexpr int NSLOT = 14;
constexpr int NPW = (NQ + ND - 1) / ND;
#define PTHR 0
#define RAWT 12
#define PFD 0
constexpr int NST = 6;
constexpr int STAGE_B = 24576;
constexpr size_t OFF_CALL = 0;
constexpr size_t OFF_TMO = 32768;
constexpr size_t OFF_CNT = 65536;
constexpr size_t OFF_PROF = 131072;
constexpr size_t OFF_FLAG = 1u << 20;
constexpr size_t OFF_OBI = 2u << 20;
constexpr size_t OFF_OBO = OFF_OBI + 1024ull * PIECE_B;
constexpr size_t OFF_XS = OFF_OBO + 512ull * PIECE_B;
constexpr size_t OFF_WS = OFF_XS + 64ull * 32 * PIECE_B;
constexpr size_t WS_NEED = OFF_WS + 64ull * 16 * PIECE_B;
static_assert(OFF_FLAG + (size_t)NR * ND * 256 * 8 <= OFF_OBI && NPW <= 256, "flags");
__host__ __device__ constexpr size_t flag_off(int src, int idx) { return OFF_FLAG + ((size_t)(src * ND + idx % ND) * 256 + idx / ND) * 8; }
constexpr int LDS_TAB = 0, LDS_ROLE = 2048;
constexpr int LDS_MISC = LDS_ROLE + NST * STAGE_B;
constexpr int LDS_BYTES = LDS_MISC + 512;
constexpr int DEC_SCR = CD_DEC_BYTES - 1040;
constexpr int LDS_SLOTS = LDS_ROLE, LDS_DSCR = LDS_SLOTS + NSLOT * PIECE_B, LDS_CTL = LDS_DSCR + 6 * DEC_SCR;
static_assert(LDS_CTL + 1024 + 64 + 64 + 64 + 16 <= LDS_MISC && NPW <= 128, "decoder layout");
static_assert(LDS_ROLE + 8 * CD_BYTES <= LDS_MISC && ND == 8, "phase 0 scratch; 32 pushers per XCD");
static_assert(16384 + 2 * CD_BYTES <= NST * STAGE_B, "epilogue scratch");
constexpr u64 DEADLINE = 40000000ull;

struct Params {
    const unsigned short* x;
    const unsigned short* w;
    unsigned short* out;
    unsigned char* ws[NR];
    int rank;
    int pad_;
};

typedef short bf16x8 __attribute__((ext_vector_type(8)));
typedef float f32x4 __attribute__((ext_vector_type(4)));
typedef short s16x4 __attribute__((ext_vector_type(4)));
#define GAS __attribute__((address_space(1)))

__device__ __forceinline__ void st_sys(void* p, u32x4 v) { asm volatile("global_store_dwordx4 %0, %1, off sc0 sc1\n\ts_nop 1" :: "v"(p), "v"(v) : "memory"); }
__device__ __forceinline__ void st_agent(void* p, u32x4 v) { asm volatile("global_store_dwordx4 %0, %1, off sc1\n\ts_nop 1" :: "v"(p), "v"(v) : "memory"); }
constexpr int wcnt(int vm, int lgkm) { return (vm & 15) | 0x70 | (lgkm << 8) | ((vm >> 4) << 14); }
__device__ __forceinline__ void wait_vm0() { asm volatile("s_waitcnt vmcnt(0)" ::: "memory"); }
__device__ __forceinline__ u32 rfl(u32 v) { return (u32)__builtin_amdgcn_readfirstlane((int)v); }
__device__ __forceinline__ u64 now() { return __builtin_amdgcn_s_memrealtime(); }

#define STAMP(slot) do { } while (0)

__device__ __forceinline__ void in_entry(int idx, int& c, bool& isw, int& bl) {
    if (idx < NINA) { const int e = idx + NPUSH; c = e / CHP; const int r = e - c * CHP; isw = r < 16; bl = isw ? r : r - 16; }
    else { const int e = idx - NINA; c = (e >> 1) & 7; isw = false; bl = GA + ((e >> 4) << 1) + (e & 1); }
}
__device__ __forceinline__ bool wait_cnt2(const unsigned char* c0, u32 t0v, const unsigned char* c1, u32 t1v, u32 call, u32* tmo) {
    const u64 t0 = now();
    for (;;) {
        const u32 a = __hip_atomic_load((const GAS u32*)c0, __ATOMIC_RELAXED, __HIP_MEMORY_SCOPE_SYSTEM);
        const u32 b = __hip_atomic_load((const GAS u32*)c1, __ATOMIC_RELAXED, __HIP_MEMORY_SCOPE_SYSTEM);
        if (rfl(a) >= t0v && rfl(b) >= t1v) return true;
        if (rfl(__hip_atomic_load((const GAS u32*)tmo, __ATOMIC_RELAXED, __HIP_MEMORY_SCOPE_AGENT)) == call || now() - t0 > DEADLINE) {
            __hip_atomic_store((GAS u32*)tmo, call, __ATOMIC_RELAXED, __HIP_MEMORY_SCOPE_AGENT);
            return false;
        }
        __builtin_amdgcn_s_sleep(2);
    }
}

__device__ __forceinline__ u32 encode_store(const u32 (&x)[32], unsigned char* slot, lds_u8* sc, const lds_u8* tab, int lane) {
    u32 A[16], L1[4];
    const u32 inf = codec_encode(x, A, L1, sc, tab, lane);
    unsigned char* o = slot + lane * 16;
    if (inf >> 31) {
#pragma unroll
        for (int j = 0; j < 8; ++j) st_sys(o + j * 1024, (u32x4){x[4 * j], x[4 * j + 1], x[4 * j + 2], x[4 * j + 3]});
    } else {
#pragma unroll
        for (int j = 0; j < 4; ++j) st_sys(o + j * 1024, (u32x4){A[4 * j], A[4 * j + 1], A[4 * j + 2], A[4 * j + 3]});
        st_sys(o + 4096, (u32x4){L1[0], L1[1], L1[2], L1[3]});
        const u32x4 v = *lp<u32x4>(sc + CD_V + lane * 16);
        if ((u32)lane * 16u < ((inf >> 8) & 0x7ffu)) st_sys(o + 5120, v);
    }
    return inf;
}

__device__ __forceinline__ u32 pk_bf16(float lo, float hi) { u32 r; asm("v_cvt_pk_bf16_f32 %0, %1, %2" : "=v"(r) : "v"(lo), "v"(hi)); return r; }

__global__ __launch_bounds__(NTH, 2) void gemm_ar_kernel(Params P) {
    extern __shared__ __attribute__((aligned(16))) unsigned char smem[];
    lds_u8* sm = (lds_u8*)smem;
    const int tid = threadIdx.x, lane = tid & 63, wid = (int)rfl(tid >> 6);
    const int b = blockIdx.x, xcd = b & 7, q = b >> 3;
    const int rank = P.rank, ri = rank >> 2, rj = rank & 3;
    unsigned char* my = P.ws[rank];
    u32* tmo = (u32*)(my + OFF_TMO);
    const lds_u8* tab = sm + LDS_TAB;
    STAMP(0);
    u32 call_v = 0;
    if (lane == 0) call_v = __hip_atomic_fetch_add((GAS u32*)(my + OFF_CALL + b * 128 + wid * 4), 1u, __ATOMIC_RELAXED, __HIP_MEMORY_SCOPE_AGENT) + 1u;

    unsigned char* wsl = P.ws[lane & 7];

    const bool is_dec = q >= 16 && q < 16 + ND && xcd < NR - 1;
    const int peer_r = (rank + 1 + xcd) & 7;
    const bool pusher = wid >= 4 && is_dec && (q - 16) * 4 + (wid - 4) < NPUSH;
    const int pi = (q - 16) * 4 + (wid - 4);
    const int pid = pusher ? (pi < 16 ? 512 + (16 * (peer_r & 3) + pi) * 8 : (32 * (peer_r >> 2) + pi - 16) * 8) : b * 4 + wid;
    const bool enc = wid < 4 || pusher;
    const bool isw = pid >= 512;
    const int blk = (pid & 511) >> 3, ch = pid & 7;
    u32x4 raw[8];
    if (enc) {
        if (!isw) {
            const unsigned short* s = P.x + (size_t)(blk * 64 + (lane & 15)) * KSH + ch * 64 + (lane >> 4) * 8;
#pragma unroll
            for (int j = 0; j < 8; ++j) raw[j] = *(const u32x4*)(s + (size_t)(j >> 1) * 16 * KSH + (j & 1) * 32);
        } else {
            const unsigned short* s = P.w + (size_t)(ch * 64 + (lane >> 3)) * NN + blk * 64 + (lane & 7) * 8;
#pragma unroll
            for (int j = 0; j < 8; ++j) raw[j] = *(const u32x4*)(s + (size_t)j * 8 * NN);
        }
    }
    if (tid < 256) codec_build_tables(sm + LDS_TAB, tid);
    __syncthreads();
    const u32 call = rfl(call_v);
    if (enc) {
        lds_u8* sc = sm + LDS_ROLE + wid * CD_BYTES;
        u32 x[32];
        if (!isw) {
#pragma unroll
            for (int j = 0; j < 8; ++j) { x[4 * j] = raw[j].x; x[4 * j + 1] = raw[j].y; x[4 * j + 2] = raw[j].z; x[4 * j + 3] = raw[j].w; }
        } else {
#pragma unroll
            for (int j = 0; j < 8; ++j) { const int k = 8 * j + (lane >> 3); *lp<u32x4>(sc + k * 128 + (((lane & 7) ^ (k & 7)) << 4)) = raw[j]; }
            const int g = lane >> 4, qq = (lane & 15) >> 2, pp = lane & 3;
#pragma unroll
            for (int j = 0; j < 8; ++j) {
                const int ni = j >> 1, ks = j & 1;
#pragma unroll
                for (int t = 0; t < 2; ++t) {
                    const int k = 32 * ks + 8 * g + 4 * t + qq, chunk = 2 * ni + (pp >> 1);
                    const s16x4 v = __builtin_amdgcn_ds_read_tr16_b64_v4i16((LAS s16x4*)(sc + k * 128 + ((chunk ^ (k & 7)) << 4) + (pp & 1) * 8));
                    const u32x2 u = __builtin_bit_cast(u32x2, v);
                    x[4 * j + 2 * t] = u.x; x[4 * j + 2 * t + 1] = u.y;
                }
            }
            asm volatile("s_waitcnt lgkmcnt(0)" ::: "memory");
        }
        STAMP(1);
        const int rb = blk & 31, step = rank * 8 + ch;
        const size_t stg = isw ? OFF_WS + (size_t)(step * 16 + (blk & 15)) * PIECE_B : OFF_XS + (size_t)(step * 32 + rb) * PIECE_B;
        const size_t cnt = OFF_CNT + (size_t)(isw ? 128 + (blk & 15) * 8 + ch : (rb >> 1) * 8 + ch) * 128;
        const bool cons = lane < NR && (isw ? (lane & 3) == (blk >> 4) : (lane >> 2) == (blk >> 5));
        const bool mine = isw ? ((blk >> 4) == rj) : ((blk >> 5) == ri);
        if (pusher) {
            unsigned char* d = P.ws[peer_r] + stg + lane * 16;
#pragma unroll
            for (int j = 0; j < 8; ++j) {
                st_sys(d + j * 1024, (u32x4){x[4 * j], x[4 * j + 1], x[4 * j + 2], x[4 * j + 3]});
                if (PTHR > 0 && (j % PTHR) == PTHR - 1) wait_vm0();
            }
            wait_vm0();
            STAMP(2);
            if (lane == peer_r) __hip_atomic_fetch_add((GAS u32*)(wsl + cnt), 1u, __ATOMIC_RELAXED, __HIP_MEMORY_SCOPE_SYSTEM);
        } else if (ch == 0 && (isw ? NPUSH >= 16 : (rb < GA && NPUSH == CHP))) {
            if (mine) {
                unsigned char* d = my + stg + lane * 16;
#pragma unroll
                for (int j = 0; j < 8; ++j) st_sys(d + j * 1024, (u32x4){x[4 * j], x[4 * j + 1], x[4 * j + 2], x[4 * j + 3]});
                wait_vm0();
                if (lane == 0) __hip_atomic_fetch_add((GAS u32*)(my + cnt), 1u, __ATOMIC_RELAXED, __HIP_MEMORY_SCOPE_SYSTEM);
            }
        } else {
            const u32 inf = encode_store(x, my + OFF_OBI + (size_t)pid * PIECE_B, sc, tab, lane);
            wait_vm0();
            STAMP(2);
            const int idx = isw ? ch * CHP + (blk & 15) - NPUSH : (rb < GA ? ch * CHP + 16 + rb - NPUSH : NINA + ((rb - GA) >> 1) * 16 + ch * 2 + (rb & 1));
            if (cons && lane != rank) __hip_atomic_store((GAS u64*)(wsl + flag_off(rank, idx)), (u64)call << 32 | inf, __ATOMIC_RELAXED, __HIP_MEMORY_SCOPE_SYSTEM);
            if (mine) {
                unsigned char* d = my + stg + lane * 16;
#pragma unroll
                for (int j = 0; j < 8; ++j) st_agent(d + j * 1024, (u32x4){x[4 * j], x[4 * j + 1], x[4 * j + 2], x[4 * j + 3]});
                wait_vm0();
                if (lane == 0) __hip_atomic_fetch_add((GAS u32*)(my + cnt), 1u, __ATOMIC_RELAXED, __HIP_MEMORY_SCOPE_AGENT);
            }
        }
        STAMP(3);
    }

    if (q < 16 || q >= 24) {
        __syncthreads();
        lds_u8* ring = sm + LDS_ROLE;
        const int wr = wid >> 1, wc = wid & 1;
        const int cb = 4 * (xcd & 3) + (q & 3);
        bool bad = false;
        const int ntile = q < 8 ? 2 : 1;
        const int q2 = q < 16 ? q : q - 8;
        for (int t = 0; t < ntile; ++t) {
            const int a = t == 1 ? 12 + 2 * (xcd >> 2) + (q >> 2) : (q2 < 12 ? 3 * (xcd >> 2) + (q2 >> 2) : 6 + 3 * (xcd >> 2) + ((q2 - 12) >> 2));
            const int g = t;
            const unsigned char* gx = my + OFF_XS + (size_t)(2 * a) * PIECE_B;
            const unsigned char* gw = my + OFF_WS + (size_t)cb * PIECE_B;
            const bool ldA = wid < 4;
            const unsigned char* gl = (ldA ? gx + wid * 4096 : gw + (wid - 4) * 4096) + lane * 16;
            const size_t lstep = ldA ? (size_t)32 * PIECE_B : (size_t)16 * PIECE_B;
            f32x4 acc[2][2] = {};
            auto issue = [&](int i) {
                const int step = (i & 7) * 8 + (i >> 3);
                lds_u8* d = ring + (i % NST) * STAGE_B + wid * 4096;
                const unsigned char* sa = gl + (size_t)step * lstep;
#pragma unroll
                for (int j = 0; j < 4; ++j) __builtin_amdgcn_global_load_lds((const GAS u32*)(sa + j * 1024), (LAS u32*)(d + j * 1024), 16, 0, 17);
            };
            const unsigned char* cx = my + OFF_CNT + (size_t)(a * 8) * 128; const unsigned char* cw = my + OFF_CNT + (size_t)(128 + cb * 8) * 128;
            volatile LAS u32* rdyw = (volatile LAS u32*)(sm + LDS_MISC + 256);
            int rdy = 0;
            u32 pa = 0, pb = 0; bool pend = false;
            auto poll_issue = [&]() { pa = __hip_atomic_load((const GAS u32*)(cx + rdy * 128), __ATOMIC_RELAXED, __HIP_MEMORY_SCOPE_SYSTEM); pb = __hip_atomic_load((const GAS u32*)(cw + rdy * 128), __ATOMIC_RELAXED, __HIP_MEMORY_SCOPE_SYSTEM); pend = true; };
            auto poll_take = [&]() { if (pend) { pend = false; if (rfl(pa) >= 16u * call && rfl(pb) >= 8u * call) { ++rdy; if (lane == 0) *rdyw = rdy; } } };
            auto need = [&](int c) {
                poll_take();
                while (rdy <= c) {
                    if (!wait_cnt2(cx + rdy * 128, 16u * call, cw + rdy * 128, 8u * call, call, tmo)) bad = true;
                    ++rdy; if (lane == 0) *rdyw = rdy;
                }
            };
            int tp = NST;
            const unsigned char* gt = lane < 32 ? gx + (q & 3) * 4096 + lane * 128 : gw + ((q >> 2) & 3) * 2048 + (lane & 15) * 128;
            const size_t tstep = lane < 32 ? (size_t)32 * PIECE_B : (size_t)16 * PIECE_B;
            auto warm = [&](int i) {
                const int lim = min(min(64, (int)rfl(*rdyw) * 8), i + PFD);
#pragma unroll 1
                for (int k = 0; k < 3 && tp < lim; ++k, ++tp)
                    __builtin_amdgcn_global_load_lds((const GAS u32*)(gt + (size_t)((tp & 7) * 8 + (tp >> 3)) * tstep), (LAS u32*)(sm + LDS_MISC), 4, 0, 17);
            };
            auto rd = [&](int i, bf16x8 (&af)[2][2], bf16x8 (&bfr)[2][2]) {
                lds_u8* st = ring + (i % NST) * STAGE_B;
#pragma unroll
                for (int m = 0; m < 2; ++m)
#pragma unroll
                    for (int ks = 0; ks < 2; ++ks) af[m][ks] = *lp<bf16x8>(st + (wr >> 1) * 8192 + (((wr & 1) * 2 + m) * 2 + ks) * 1024 + lane * 16);
#pragma unroll
                for (int n = 0; n < 2; ++n)
#pragma unroll
                    for (int ks = 0; ks < 2; ++ks) bfr[n][ks] = *lp<bf16x8>(st + 16384 + ((wc * 2 + n) * 2 + ks) * 1024 + lane * 16);
            };
            auto mm = [&](bf16x8 (&af)[2][2], bf16x8 (&bfr)[2][2]) {
#pragma unroll
                for (int ks = 0; ks < 2; ++ks)
#pragma unroll
                    for (int m = 0; m < 2; ++m)
#pragma unroll
                        for (int n = 0; n < 2; ++n) acc[m][n] = __builtin_amdgcn_mfma_f32_16x16x32_bf16(bfr[n][ks], af[m][ks], acc[m][n], 0, 0, 0);
            };
            auto kloop = [&](auto role_c) {
                constexpr int ROLE = decltype(role_c)::value;
                bf16x8 a0[2][2], b0[2][2], a1[2][2], b1[2][2];
                if constexpr (ROLE == 1) { if (lane == 0) *rdyw = 0; need(0); }
                __builtin_amdgcn_s_barrier();
                if (g == 0) STAMP(4); else STAMP(12);
                if constexpr (ROLE == 0) {
#pragma unroll
                    for (int i = 0; i < NST - 1; ++i) issue(i);
                    __builtin_amdgcn_s_waitcnt(wcnt(4 * (NST - 2), 15));
                }
                __builtin_amdgcn_s_barrier();
                rd(0, a0, b0);
                auto pair = [&](int i, auto tail_c) {
                    constexpr int TAIL = decltype(tail_c)::value;
                    { const int nx = i + NST - 1;
                      if constexpr (ROLE == 0) __builtin_amdgcn_s_waitcnt(wcnt(TAIL == 0 || TAIL + 4 < 64 ? 4 * (NST - 3) : 4 * (63 - TAIL - 1 > 0 ? 63 - TAIL - 1 : 0), 0));
                      else __builtin_amdgcn_s_waitcnt(wcnt(63, 0));
                      __builtin_amdgcn_s_barrier();
                      if constexpr (ROLE == 0 && (TAIL == 0 || TAIL + NST - 1 < 64)) issue(nx);
                      rd(i + 1, a1, b1);
                      __builtin_amdgcn_sched_barrier(0);
                      mm(a0, b0);
                      __builtin_amdgcn_sched_barrier(0);
                      if constexpr (ROLE == 1 && TAIL == 0) { if ((i & 2) == 0 && rdy < 8) { poll_take(); if (rdy < 8) poll_issue(); } }
                      if constexpr (ROLE == 2 && TAIL == 0) warm(i); }
                    { const int nx = i + NST;
                      if constexpr (ROLE == 1 && TAIL == 0) { if ((nx & 7) == 0) need(nx >> 3); }
                      if constexpr (ROLE == 0) __builtin_amdgcn_s_waitcnt(wcnt(TAIL == 0 ? 4 * (NST - 3) : 4 * (63 - TAIL - 2 > 0 ? 63 - TAIL - 2 : 0), 0));
                      else __builtin_amdgcn_s_waitcnt(wcnt(63, 0));
                      __builtin_amdgcn_s_barrier();
                      if constexpr (ROLE == 0 && TAIL == 0) issue(nx);
                      if constexpr (TAIL != 62) rd(i + 2, a0, b0);
                      __builtin_amdgcn_sched_barrier(0);
                      mm(a1, b1);
                      __builtin_amdgcn_sched_barrier(0);
                      if constexpr (ROLE == 2 && TAIL == 0) warm(i + 1); }
                };
#pragma unroll 1
                for (int i = 0; i < 58; i += 2) pair(i, std::integral_constant<int, 0>{});
                pair(58, std::integral_constant<int, 58>{});
                pair(60, std::integral_constant<int, 60>{});
                pair(62, std::integral_constant<int, 62>{});
            };
            if (wid < 6) kloop(std::integral_constant<int, 0>{});
            else if (wid == 6) kloop(std::integral_constant<int, 1>{});
            else kloop(std::integral_constant<int, 2>{});
            wait_vm0();
            if (g == 0) STAMP(5); else STAMP(13);
            __syncthreads();
#pragma unroll
            for (int m = 0; m < 2; ++m)
#pragma unroll
                for (int n = 0; n < 2; ++n) {
                    const int row = 32 * wr + 16 * m + (lane & 15), col = 32 * wc + 16 * n + 4 * (lane >> 4);
                    const u32x2 v = {pk_bf16(acc[m][n][0], acc[m][n][1]), pk_bf16(acc[m][n][2], acc[m][n][3])};
                    *lp<u32x2>(ring + row * 128 + (((col >> 3) ^ (row & 7)) << 4) + (col & 4) * 2) = v;
                }
            if (tid == 6 * 64) *lp<u32>(sm + LDS_MISC + 272) = (bad || __hip_atomic_load((const GAS u32*)tmo, __ATOMIC_RELAXED, __HIP_MEMORY_SCOPE_AGENT) == call) ? 1u : 0u;
            __syncthreads();
            if (wid < 4) {
                const int h = wid & 1;
                const bool poison = *lp<u32>(sm + LDS_MISC + 272) != 0u;
                u32 x[32];
#pragma unroll
                for (int j = 0; j < 8; ++j) {
                    const int row = 64 * h + 8 * j + (lane >> 3);
                    u32x4 v = *lp<u32x4>(ring + row * 128 + (((lane & 7) ^ (row & 7)) << 4));
                    if (poison) v = (u32x4){0x7fc07fc0u, 0x7fc07fc0u, 0x7fc07fc0u, 0x7fc07fc0u};
                    x[4 * j] = v.x; x[4 * j + 1] = v.y; x[4 * j + 2] = v.z; x[4 * j + 3] = v.w;
                }
                const int o = (a * 16 + cb) * 2 + h;
                if (wid < 2) {
                    u32 inf;
                    if (o >= NOUT - RAWT) {
                        inf = 0x80000000u;
                        unsigned char* ob = my + OFF_OBO + (size_t)o * PIECE_B + lane * 16;
#pragma unroll
                        for (int j = 0; j < 8; ++j) st_sys(ob + j * 1024, (u32x4){x[4 * j], x[4 * j + 1], x[4 * j + 2], x[4 * j + 3]});
                    } else inf = encode_store(x, my + OFF_OBO + (size_t)o * PIECE_B, ring + 16384 + wid * CD_BYTES, tab, lane);
                    wait_vm0();
                    if (lane < NR && lane != rank)
                        __hip_atomic_store((GAS u64*)(wsl + flag_off(rank, NIN + o)), (u64)call << 32 | inf, __ATOMIC_RELAXED, __HIP_MEMORY_SCOPE_SYSTEM);
                } else {
                    unsigned short* d = P.out + (size_t)(2048 * ri + 128 * a + 64 * h + (lane >> 3)) * NN + 1024 * rj + 64 * cb + (lane & 7) * 8;
#pragma unroll
                    for (int j = 0; j < 8; ++j) st_agent(d + (size_t)j * 8 * NN, (u32x4){x[4 * j], x[4 * j + 1], x[4 * j + 2], x[4 * j + 3]});
                }
            }
            if (g == 0) STAMP(6); else STAMP(14);
            __syncthreads();
        }
    } else if (is_dec) {
        const int src = peer_r, si = src >> 2, sj = src & 3;
        const unsigned char* peer = P.ws[src];
        const int dj = q - 16;
        const int np = (NQ - dj + ND - 1) / ND;
        volatile LAS u32* c_info = (volatile LAS u32*)(sm + LDS_CTL);
        volatile LAS u32* c_full = (volatile LAS u32*)(sm + LDS_CTL + 1024);
        volatile LAS u32* c_done = (volatile LAS u32*)(sm + LDS_CTL + 1088);
        volatile LAS u32* c_piece = (volatile LAS u32*)(sm + LDS_CTL + 1152);
        LAS u32* c_ticket = (LAS u32*)(sm + LDS_CTL + 1216);
        __syncthreads();
        if (tid < 256) c_info[tid] = 0;
        if (tid < 16) { c_full[tid] = 0; c_done[tid] = 0; }
        if (tid == 0) *c_ticket = 0;
        __syncthreads();
        auto src_slot = [&](int idx) -> const unsigned char* {
            if (idx >= NIN) return peer + OFF_OBO + (size_t)(idx - NIN) * PIECE_B;
            int c, bl; bool isw2; in_entry(idx, c, isw2, bl);
            return peer + OFF_OBI + (size_t)(isw2 ? 512 + (16 * rj + bl) * 8 + c : (32 * ri + bl) * 8 + c) * PIECE_B;
        };
        if (wid == 0) {
            const unsigned char* fb = my + OFF_FLAG + (size_t)(src * ND + dj) * 256 * 8;
            const u64 t0 = now();
            bool p0 = lane >= np, p1 = lane + 64 >= np;
            int rounds = 0;
            for (;;) {
                u64 v0 = 0, v1 = 0;
                if (!p0) v0 = __hip_atomic_load((const GAS u64*)(fb + (size_t)lane * 8), __ATOMIC_RELAXED, __HIP_MEMORY_SCOPE_SYSTEM);
                if (!p1) v1 = __hip_atomic_load((const GAS u64*)(fb + (size_t)(lane + 64) * 8), __ATOMIC_RELAXED, __HIP_MEMORY_SCOPE_SYSTEM);
                if (!p0 && (u32)(v0 >> 32) >= call) { c_info[lane] = (u32)v0 | 0x40000000u; p0 = true; }
                if (!p1 && (u32)(v1 >> 32) >= call) { c_info[lane + 64] = (u32)v1 | 0x40000000u; p1 = true; }
                if (__all(p0 && p1)) break;
                if ((++rounds & 7) == 0 && (rfl(__hip_atomic_load((const GAS u32*)tmo, __ATOMIC_RELAXED, __HIP_MEMORY_SCOPE_AGENT)) == call || now() - t0 > DEADLINE)) {
                    __hip_atomic_store((GAS u32*)tmo, call, __ATOMIC_RELAXED, __HIP_MEMORY_SCOPE_AGENT);
                    if (!p0) c_info[lane] = 0x60000000u;
                    if (!p1) c_info[lane + 64] = 0x60000000u;
                    break;
                }
                __builtin_amdgcn_s_sleep(2);
            }
        } else if (wid == 1) {
            int pub = 0;
            auto publish_upto = [&](int e) { for (; pub < e; ++pub) if (lane == 0) c_full[pub % NSLOT] = pub + 1; };
            u64 todo0 = np >= 64 ? ~0ull : (1ull << np) - 1ull, todo1 = np > 64 ? (1ull << (np - 64)) - 1ull : 0ull;
            int n = 0; u32 inf = 0;
            auto pick = [&](int s) -> bool {
                const u32 v0 = c_info[lane], v1 = c_info[lane + 64], dn = c_done[s % NSLOT];
                const u64 a0 = __ballot(v0 != 0u) & todo0, a1 = __ballot(v1 != 0u) & todo1;
                if ((a0 | a1) == 0ull || !(s < NSLOT || rfl(dn) == (u32)(s - NSLOT + 1))) return false;
                if (a0) { const int l = __builtin_ctzll(a0); todo0 &= ~(1ull << l); n = l; inf = (u32)__builtin_amdgcn_readlane((int)v0, l); }
                else { const int l = __builtin_ctzll(a1); todo1 &= ~(1ull << l); n = 64 + l; inf = (u32)__builtin_amdgcn_readlane((int)v1, l); }
                return true;
            };
            bool have = false;
            for (int s = 0; s < np; ++s) {
                const int slot = s % NSLOT;
                if (!have) { wait_vm0(); publish_upto(s); while (!pick(s)) __builtin_amdgcn_s_sleep(1); }
                if (lane == 0) c_piece[slot] = n;
                lds_u8* d = sm + LDS_SLOTS + slot * PIECE_B;
                const unsigned char* s0 = src_slot(dj + n * ND);
                const unsigned char* sp = s0 + lane * 16;
                if (inf & 0x20000000u) { have = false; wait_vm0(); publish_upto(s + 1); continue; }
                if (inf >> 31) {
                    wait_vm0(); publish_upto(s);
#pragma unroll
                    for (int j = 0; j < 8; ++j) __builtin_amdgcn_global_load_lds((const GAS u32*)(sp + j * 1024), (LAS u32*)(d + j * 1024), 16, 0, 17);
                    have = s + 1 < np && pick(s + 1);
                    wait_vm0(); publish_upto(s + 1);
                } else {
                    const unsigned char* sv = ((u32)lane * 16u < ((inf >> 8) & 0x7ffu)) ? sp + 5120 : s0 + 5120;
#pragma unroll
                    for (int j = 0; j < 6; ++j) {
                        asm volatile("s_waitcnt vmcnt(%0)" :: "n"(NFL - 1) : "memory");
                        publish_upto(s + (j - NFL + 1 - 5) / 6);
                        __builtin_amdgcn_global_load_lds((const GAS u32*)(j < 5 ? sp + j * 1024 : sv), (LAS u32*)(d + j * 1024), 16, 0, 17);
                        if (j == 1) have = s + 1 < np && pick(s + 1);
                    }
                }
            }
            wait_vm0(); publish_upto(np);
            STAMP(9);
        } else {
            lds_u8* sc = sm + LDS_DSCR + (wid - 2) * DEC_SCR - 1040;
            int nstamp = 0;
            for (;;) {
                u32 nn = 0;
                if (lane == 0) nn = __hip_atomic_fetch_add(c_ticket, 1u, __ATOMIC_RELAXED, __HIP_MEMORY_SCOPE_WORKGROUP);
                const int sq = (int)rfl(nn);
                if (sq >= np) break;
                while (rfl(c_full[sq % NSLOT]) != (u32)(sq + 1)) __builtin_amdgcn_s_sleep(1);
                asm volatile("" ::: "memory");
                const int n = (int)rfl(c_piece[sq % NSLOT]);
                const u32 inf = rfl(c_info[n]);
                const bool ok = !(inf & 0x20000000u);
                lds_u8* sl = sm + LDS_SLOTS + (sq % NSLOT) * PIECE_B;
                u32 x[32];
                if (!ok) {
#pragma unroll
                    for (int j = 0; j < 32; ++j) x[j] = 0x7fc07fc0u;
                } else if (inf >> 31) {
#pragma unroll
                    for (int j = 0; j < 8; ++j) { const u32x4 v = *lp<u32x4>(sl + j * 1024 + lane * 16); x[4 * j] = v.x; x[4 * j + 1] = v.y; x[4 * j + 2] = v.z; x[4 * j + 3] = v.w; }
                } else {
                    u32 A[16], L1[4];
#pragma unroll
                    for (int j = 0; j < 4; ++j) { const u32x4 v = *lp<u32x4>(sl + j * 1024 + lane * 16); A[4 * j] = v.x; A[4 * j + 1] = v.y; A[4 * j + 2] = v.z; A[4 * j + 3] = v.w; }
                    { const u32x4 v = *lp<u32x4>(sl + 4096 + lane * 16); L1[0] = v.x; L1[1] = v.y; L1[2] = v.z; L1[3] = v.w; }
                    codec_decode(x, A, L1, inf & 255u, (const LAS u32*)(sl + 5120), sc, tab, lane);
                }
                asm volatile("s_waitcnt lgkmcnt(0)" ::: "memory");
                if (lane == 0) c_done[sq % NSLOT] = sq + 1;
                const int idx = dj + n * ND;
                if (idx < NIN) {
                    int c, bl; bool isw2; in_entry(idx, c, isw2, bl);
                    const int step = src * 8 + c;
                    unsigned char* d = (isw2 ? my + OFF_WS + (size_t)(step * 16 + bl) * PIECE_B : my + OFF_XS + (size_t)(step * 32 + bl) * PIECE_B) + lane * 16;
                    if (ok) {
#pragma unroll
                        for (int j = 0; j < 8; ++j) st_agent(d + j * 1024, (u32x4){x[4 * j], x[4 * j + 1], x[4 * j + 2], x[4 * j + 3]});
                        wait_vm0();
                        if (lane == 0) __hip_atomic_fetch_add((GAS u32*)(my + OFF_CNT + (size_t)(isw2 ? 128 + bl * 8 + c : (bl >> 1) * 8 + c) * 128), 1u, __ATOMIC_RELAXED, __HIP_MEMORY_SCOPE_AGENT);
                    }
                } else {
                    const int o = idx - NIN, g = o >> 8, tile = (o & 255) >> 1, h = o & 1;
                    unsigned short* d = P.out + (size_t)(2048 * si + 1024 * g + 128 * (tile >> 4) + 64 * h + (lane >> 3)) * NN + 1024 * sj + 64 * (tile & 15) + (lane & 7) * 8;
#pragma unroll
                    for (int j = 0; j < 8; ++j) st_agent(d + (size_t)j * 8 * NN, (u32x4){x[4 * j], x[4 * j + 1], x[4 * j + 2], x[4 * j + 3]});
                }
            }
            (void)nstamp;
        }
    }
    STAMP(31);
}

extern "C" void kernel_launch(int rank, int nranks, ncclComm_t comm, hipStream_t stream,
                              void* const* d_in, const int* in_sizes, int n_in,
                              void* d_out, int out_size, const int* in_dtypes, int out_dtype,
                              void* const* d_out_all, void* const* d_ws_all, size_t ws_bytes) {
    (void)comm; (void)in_dtypes; (void)out_dtype; (void)d_out_all;
    if (nranks != NR || n_in != 2 || rank < 0 || rank >= NR || in_sizes[0] != MM * KSH || in_sizes[1] != KSH * NN || out_size != MM * NN || ws_bytes < WS_NEED) {
        fprintf(stderr, "kernel_launch: unexpected problem shape\n");
        return;
    }
    static bool first = true;
    if (first) {
        hipError_t e = hipFuncSetAttribute((const void*)gemm_ar_kernel, hipFuncAttributeMaxDynamicSharedMemorySize, LDS_BYTES);
        if (e != hipSuccess) { fprintf(stderr, "hipFuncSetAttribute: %s\n", hipGetErrorString(e)); return; }
        first = false;
    }
    Params P;
    P.x = (const unsigned short*)d_in[0];
    P.w = (const unsigned short*)d_in[1];
    P.out = (unsigned short*)d_out;
    for (int p = 0; p < NR; ++p) P.ws[p] = (unsigned char*)d_ws_all[p];
    P.rank = rank;
    P.pad_ = 0;
    hipLaunchKernelGGL(gemm_ar_kernel, dim3(NWG), dim3(NTH), LDS_BYTES, stream, P);
    hipError_t e = hipGetLastError();
    if (e != hipSuccess) fprintf(stderr, "launch: %s\n", hipGetErrorString(e));
}
```

```cpp
#include <hip/hip_runtime.h>
#include <rccl/rccl.h>
#include <cstdio>
#include <type_traits>

typedef unsigned u32;
typedef unsigned long long u64;
typedef unsigned u32x4 __attribute__((ext_vector_type(4)));
typedef unsigned u32x2 __attribute__((ext_vector_type(2)));
typedef unsigned short u16x2 __attribute__((ext_vector_type(2)));
#define LAS __attribute__((address_space(3)))
typedef LAS unsigned char lds_u8;

constexpr int CD_V = 0;
constexpr int CD_R2 = 1040;
constexpr int CD_R3 = CD_R2 + 64 * 68;
constexpr int CD_R4 = CD_R3 + 64 * 36;
constexpr int CD_BYTES = CD_R4 + 64 * 20;
constexpr int CD_DEC_R2 = 1040, CD_DEC_R3 = CD_DEC_R2 + 64 * 36, CD_DEC_BYTES = CD_DEC_R3 + 64 * 20;
constexpr int CD_TAB_BYTES = 2048;
static_assert(CD_BYTES % 16 == 0 && CD_DEC_BYTES % 16 == 0, "align");

__device__ __forceinline__ u32 perm(u32 hi, u32 lo, u32 sel) { return __builtin_amdgcn_perm(hi, lo, sel); }
__device__ __forceinline__ u32 dot4(u32 a, u32 b, u32 c) { return __builtin_amdgcn_udot4(a, b, c, false); }
__device__ __forceinline__ u32 pk_sub(u32 a, u32 b) { return __builtin_bit_cast(u32, (u16x2)(__builtin_bit_cast(u16x2, a) - __builtin_bit_cast(u16x2, b))); }
__device__ __forceinline__ u32 pk_min(u32 a, u32 b) { return __builtin_bit_cast(u32, __builtin_elementwise_min(__builtin_bit_cast(u16x2, a), __builtin_bit_cast(u16x2, b))); }
__device__ __forceinline__ u32 wave_incl_scan(u32 x) {
    x += (u32)__builtin_amdgcn_update_dpp(0, (int)x, 0x111, 0xf, 0xf, false);
    x += (u32)__builtin_amdgcn_update_dpp(0, (int)x, 0x112, 0xf, 0xf, false);
    x += (u32)__builtin_amdgcn_update_dpp(0, (int)x, 0x114, 0xf, 0xf, false);
    x += (u32)__builtin_amdgcn_update_dpp(0, (int)x, 0x118, 0xf, 0xf, false);
    x += (u32)__builtin_amdgcn_update_dpp(0, (int)x, 0x142, 0xa, 0xf, false);
    x += (u32)__builtin_amdgcn_update_dpp(0, (int)x, 0x143, 0xc, 0xf, false);
    return x;
}
template <class T> __device__ __forceinline__ LAS T* lp(lds_u8* p) { return (LAS T*)p; }
#define CFENCE() asm volatile("" ::: "memory")

__device__ __forceinline__ void codec_build_tables(lds_u8* tab, int i) {
    u32 dep = 0, cmp = 0x0c0c0c0cu; int n = 0;
#pragma unroll
    for (int k = 0; k < 4; ++k) {
        const u32 c = (i >> (2 * k)) & 3;
        if (c == 3) { dep |= (u32)n << (8 * k); cmp = (cmp & ~(0xffu << (8 * n))) | ((u32)k << (8 * n)); ++n; }
        else dep |= (4u + c) << (8 * k);
    }
    lp<u32>(tab)[i] = dep; lp<u32>(tab)[256 + i] = cmp;
}

__device__ __forceinline__ void stream_append(LAS u32* r, u32 pk, u32& p, u32 pnew, u32& acc) {
    const u64 w = (u64)pk << ((p & 3u) * 8u);
    const u32 lo = (u32)w | acc;
    r[p >> 2] = lo;
    acc = ((p ^ pnew) & ~3u) ? (u32)(w >> 32) : lo;
    p = pnew;
}

__device__ __forceinline__ u32 codec_encode(const u32 (&x)[32], u32 (&A)[16], u32 (&L1)[4], lds_u8* sc, const lds_u8* tab, int lane) {
    const LAS u32* CMP = (const LAS u32*)(tab + 1024);
    LAS u32* r2 = (LAS u32*)(sc + CD_R2) + lane * 17; LAS u32* r3 = (LAS u32*)(sc + CD_R3) + lane * 9; LAS u32* r4 = (LAS u32*)(sc + CD_R4) + lane * 5;
    const u32x4 z4 = {0, 0, 0, 0};
#pragma unroll
    for (int k = 0; k < 8; ++k) r2[k] = 0;
#pragma unroll
    for (int k = 0; k < 4; ++k) r3[k] = 0;
    r4[0] = 0; r4[1] = 0;
    *lp<u32x4>(sc + CD_V + lane * 16) = z4; if (lane == 0) *lp<u32x4>(sc + CD_V + 1024) = z4;
    u32 B;
    { const u32 y0 = __builtin_amdgcn_alignbit(x[0], x[0], 31), y1 = __builtin_amdgcn_alignbit(x[1], x[1], 31);
      const u32 s = wave_incl_scan(dot4(perm(y1, y0, 0x07050301u), 0x01010101u, 0u));
      B = ((u32)__builtin_amdgcn_readlane((int)s, 63) + 724u) >> 8; B = B > 255u ? 255u : B; }
    const u32 BK = (B << 8 | 0xffu) * 0x00010001u;
    u32 tb[16], bb[16], pn[16];
    u32 pc = 0;
#pragma unroll
    for (int g = 0; g < 16; ++g) {
        const u32 y0 = __builtin_amdgcn_alignbit(x[2 * g], x[2 * g], 31), y1 = __builtin_amdgcn_alignbit(x[2 * g + 1], x[2 * g + 1], 31);
        A[g] = perm(y1, y0, 0x06040200u);
        const u32 d0 = pk_sub(BK, y0), d1 = pk_sub(BK, y1);
        const u32 m0 = pk_min(d0, 0x07ff07ffu), m1 = pk_min(d1, 0x07ff07ffu);
        const u32 tc = perm(m1, m0, 0x07050301u);
        tb[g] = perm(d1, d0, 0x07050301u) ^ 0x04040404u;
        const u32 c1 = perm(0x03030303u, 0x02010003u, tc), ef = perm(0x01010101u, 0x00000001u, tc);
        bb[g] = dot4(c1, 0x40100401u, 0u);
        pc = dot4(ef, 0x01010101u, pc); pn[g] = pc;
        if ((g & 3) == 0) L1[g >> 2] = bb[g]; else L1[g >> 2] |= bb[g] << (8 * (g & 3));
    }
    u32 sel[16];
#pragma unroll
    for (int g = 0; g < 16; ++g) sel[g] = CMP[bb[g]];
    CFENCE();
    u32 p2 = 0, acc = 0;
#pragma unroll
    for (int g = 0; g < 16; ++g) stream_append(r2, perm(0u, tb[g], sel[g]), p2, pn[g], acc);
    r2[p2 >> 2] = acc;
    CFENCE();
    u32 C2[2], p3 = 0;
    { u32 u[8], c[8], b[8], s2[8];
#pragma unroll
      for (int k = 0; k < 8; ++k) u[k] = r2[k];
#pragma unroll
      for (int k = 0; k < 8; ++k) {
          const u32 big = perm(0u, 0u, ((u[k] & 0xf8f8f8f8u) >> 1) + 0x0c0c0c0cu);
          c[k] = perm(0x03030302u, 0x03030100u, u[k] & 0x07070707u) | (big & 0x03030303u);
          b[k] = dot4(c[k], 0x40100401u, 0u);
          if ((k & 3) == 0) C2[k >> 2] = b[k]; else C2[k >> 2] |= b[k] << (8 * (k & 3));
      }
#pragma unroll
      for (int k = 0; k < 8; ++k) s2[k] = CMP[b[k]];
      CFENCE();
      acc = 0;
#pragma unroll
      for (int k = 0; k < 8; ++k)
          stream_append(r3, perm(0u, u[k] ^ 0x02020202u, s2[k]), p3, dot4(perm(0u, 0x01000000u, c[k]), 0x01010101u, p3), acc);
      r3[p3 >> 2] = acc;
      CFENCE(); }
    u32 C3 = 0, p4 = 0;
    { u32 u[4], c[4], b[4], s3[4];
#pragma unroll
      for (int k = 0; k < 4; ++k) u[k] = r3[k];
#pragma unroll
      for (int k = 0; k < 4; ++k) {
          const u32 big = perm(0u, 0u, ((u[k] & 0xfefefefeu) >> 1) + 0x0c0c0c0cu);
          const u32 n14 = perm(0u, 0u, (((u[k] ^ 0x0e0e0e0eu) & 0x7f7f7f7fu) + 0x0c0c0c0cu) | (u[k] & 0x80808080u));
          c[k] = (u[k] & 0x01010101u & ~big) | (big & (0x02020202u | (n14 & 0x01010101u)));
          b[k] = dot4(c[k], 0x40100401u, 0u);
          C3 |= b[k] << (8 * k);
      }
#pragma unroll
      for (int k = 0; k < 4; ++k) s3[k] = CMP[b[k]];
      CFENCE();
      acc = 0;
#pragma unroll
      for (int k = 0; k < 4; ++k) {
          const u32 pk = perm(0u, u[k] ^ 0x06060606u, s3[k]);
          stream_append(r4, pk, p4, dot4(perm(0u, 0x01000000u, c[k]), 0x01010101u, p4), acc);
      }
      r4[p4 >> 2] = acc;
      CFENCE(); }
    const bool over = p2 > 31u || p3 > 15u || p4 > 8u;
    if (__any(over)) return 0x80000000u;
    const u32 q = p2 | p3 << 11 | p4 << 21, qi = wave_incl_scan(q), qe = qi - q;
    const u32 tot = (u32)__builtin_amdgcn_readlane((int)qi, 63);
    const u32 o3 = ((2 * (tot & 0x7ffu) + 31u) >> 5) * 4u, o4 = o3 + ((2 * ((tot >> 11) & 0x3ffu) + 31u) >> 5) * 4u, vlen = o4 + (tot >> 21);
    if (vlen > 1024u) return 0x80000000u;
    LAS u32* V = (LAS u32*)(sc + CD_V);
    { const u32 bit = 2 * (qe & 0x7ffu), sh = bit & 31u; LAS u32* a = V + (bit >> 5);
      const u64 w = ((u64)C2[1] << 32 | C2[0]) << sh;
      __hip_atomic_fetch_or(a, (u32)w, __ATOMIC_RELAXED, __HIP_MEMORY_SCOPE_WAVEFRONT);
      __hip_atomic_fetch_or(a + 1, (u32)(w >> 32), __ATOMIC_RELAXED, __HIP_MEMORY_SCOPE_WAVEFRONT);
      __hip_atomic_fetch_or(a + 2, (C2[1] >> 1) >> (31u - sh), __ATOMIC_RELAXED, __HIP_MEMORY_SCOPE_WAVEFRONT); }
    { const u32 bit = 8 * o3 + 2 * ((qe >> 11) & 0x3ffu), sh = bit & 31u; LAS u32* a = V + (bit >> 5);
      const u64 w = (u64)C3 << sh;
      __hip_atomic_fetch_or(a, (u32)w, __ATOMIC_RELAXED, __HIP_MEMORY_SCOPE_WAVEFRONT);
      __hip_atomic_fetch_or(a + 1, (u32)(w >> 32), __ATOMIC_RELAXED, __HIP_MEMORY_SCOPE_WAVEFRONT); }
    { const u32 r0 = r4[0], r1 = r4[1];
      const u32 bit = 8 * (o4 + (qe >> 21)), sh = bit & 31u; LAS u32* a = V + (bit >> 5);
      const u64 w = ((u64)r1 << 32 | r0) << sh;
      __hip_atomic_fetch_or(a, (u32)w, __ATOMIC_RELAXED, __HIP_MEMORY_SCOPE_WAVEFRONT);
      __hip_atomic_fetch_or(a + 1, (u32)(w >> 32), __ATOMIC_RELAXED, __HIP_MEMORY_SCOPE_WAVEFRONT);
      __hip_atomic_fetch_or(a + 2, (r1 >> 1) >> (31u - sh), __ATOMIC_RELAXED, __HIP_MEMORY_SCOPE_WAVEFRONT); }
    CFENCE();
    return B | vlen << 8;
}

__device__ __forceinline__ void codec_decode(u32 (&x)[32], const u32 (&A)[16], const u32 (&L1)[4], u32 B, const LAS u32* V, lds_u8* sc, const lds_u8* tab, int lane) {
    const LAS u32* DEP = (const LAS u32*)tab;
    LAS u32* r2 = (LAS u32*)(sc + CD_DEC_R2) + lane * 9; LAS u32* r3 = (LAS u32*)(sc + CD_DEC_R3) + lane * 5;
    const u32 T1 = ((B - 1u) & 255u) | ((B - 2u) & 255u) << 8 | ((B - 3u) & 255u) << 16;
    const u32 T2 = ((B - 4u) & 255u) | ((B - 5u) & 255u) << 8 | (B & 255u) << 16;
    const u32 T3 = ((B - 6u) & 255u) | ((B - 7u) & 255u) << 8 | ((B - 8u) & 255u) << 16;
    u32 e1[4], n2 = 0;
#pragma unroll
    for (int k = 0; k < 4; ++k) { e1[k] = L1[k] & (L1[k] >> 1) & 0x55555555u; n2 += __builtin_popcount(e1[k]); }
    const u32 s2i = wave_incl_scan(n2), s2 = s2i - n2, tot2 = (u32)__builtin_amdgcn_readlane((int)s2i, 63);
    u32 d1[16];
#pragma unroll
    for (int g = 0; g < 16; ++g) d1[g] = DEP[(L1[g >> 2] >> (8 * (g & 3))) & 255u];
    u32 C2[2];
    { const u32 bit = 2 * s2, sh = bit & 31u; const LAS u32* a = V + (bit >> 5);
      const u32 d0 = a[0], dd1 = a[1], d2 = a[2];
      const u64 m = (1ull << (2 * n2)) - 1ull;
      C2[0] = __builtin_amdgcn_alignbit(dd1, d0, sh) & (u32)m; C2[1] = __builtin_amdgcn_alignbit(d2, dd1, sh) & (u32)(m >> 32); }
    u32 e2[2]; e2[0] = C2[0] & (C2[0] >> 1) & 0x55555555u; e2[1] = C2[1] & (C2[1] >> 1) & 0x55555555u;
    const u32 n3 = __builtin_popcount(e2[0]) + __builtin_popcount(e2[1]);
    const u32 s3i = wave_incl_scan(n3), s3 = s3i - n3, tot3 = (u32)__builtin_amdgcn_readlane((int)s3i, 63);
    u32 dp2[8];
#pragma unroll
    for (int k = 0; k < 8; ++k) dp2[k] = DEP[(C2[k >> 2] >> (8 * (k & 3))) & 255u];
    const u32 o3 = ((2 * tot2 + 31u) >> 5) * 4u, o4 = o3 + ((2 * tot3 + 31u) >> 5) * 4u;
    u32 C3;
    { const u32 bit = 8 * o3 + 2 * s3, sh = bit & 31u; const LAS u32* a = V + (bit >> 5);
      C3 = __builtin_amdgcn_alignbit(a[1], a[0], sh) & ((1u << (2 * n3)) - 1u); }
    const u32 e3 = C3 & (C3 >> 1) & 0x55555555u, n4 = __builtin_popcount(e3);
    const u32 s4 = wave_incl_scan(n4) - n4;
    u32 dp3[4];
#pragma unroll
    for (int k = 0; k < 4; ++k) dp3[k] = DEP[(C3 >> (8 * k)) & 255u];
    u64 R4;
    { const u32 byte = o4 + s4, sh = (byte & 3u) * 8u; const LAS u32* a = V + (byte >> 2);
      const u32 d0 = a[0], dd1 = a[1], d2 = a[2];
      u32 t0 = __builtin_amdgcn_alignbit(dd1, d0, sh), t1 = __builtin_amdgcn_alignbit(d2, dd1, sh);
      const u32 B4 = B * 0x01010101u;
      t0 = ((B4 | 0x80808080u) - (t0 & 0x7f7f7f7fu)) ^ ((B4 ^ ~t0) & 0x80808080u);
      t1 = ((B4 | 0x80808080u) - (t1 & 0x7f7f7f7fu)) ^ ((B4 ^ ~t1) & 0x80808080u);
      R4 = (u64)t1 << 32 | t0; }
#pragma unroll
    for (int k = 0; k < 4; ++k) {
        const u32 p = k == 0 ? 0u : (u32)__builtin_popcount(e3 & ((1u << (8 * k)) - 1u));
        r3[k] = perm(T3, (u32)(R4 >> (8 * p)), dp3[k]);
    }
    CFENCE();
    { u32 ov[8];
#pragma unroll
      for (int k = 0; k < 8; ++k) {
          u32 p = (k & 3) == 0 ? 0u : (u32)__builtin_popcount(e2[k >> 2] & ((1u << (8 * (k & 3))) - 1u));
          if (k >= 4) p += __builtin_popcount(e2[0]);
          const LAS u32* a = r3 + (p >> 2);
          ov[k] = perm(T2, __builtin_amdgcn_alignbyte(a[1], a[0], p), dp2[k]);
      }
      CFENCE();
#pragma unroll
      for (int k = 0; k < 8; ++k) r2[k] = ov[k]; }
    CFENCE();
    u32 pb = 0;
#pragma unroll
    for (int g = 0; g < 16; ++g) {
        const u32 p = pb + ((g & 3) == 0 ? 0u : (u32)__builtin_popcount(e1[g >> 2] & ((1u << (8 * (g & 3))) - 1u)));
        const LAS u32* a = r2 + (p >> 2);
        const u32 E = perm(T1, __builtin_amdgcn_alignbyte(a[1], a[0], p), d1[g]);
        const u32 y0 = perm(E, A[g], 0x05010400u), y1 = perm(E, A[g], 0x07030602u);
        x[2 * g] = __builtin_amdgcn_alignbit(y0, y0, 1); x[2 * g + 1] = __builtin_amdgcn_alignbit(y1, y1, 1);
        if ((g & 3) == 3) pb += __builtin_popcount(e1[g >> 2]);
    }
    CFENCE();
}

constexpr int NR = 8;
constexpr int MM = 4096, NN = 4096, KSH = 512;
constexpr int NWG = 256, NTH = 512;
constexpr int PIECE_B = 8192;
#define NPUSH 16
constexpr int GA = 12;
constexpr int CHP = 16 + GA;
constexpr int NINA = 8 * CHP - NPUSH, NIN = NINA + (32 - GA) * 8, NOUT = 512, NQ = NIN + NOUT;
#define ND 8
#define NFLI 8
#define NFL 9
constexpr int NSLOT = 11;
constexpr int NPW = (NQ + ND - 1) / ND;
#define PTHR 0
#define RAWH 0
#define PRI0 2
#define PRID 1
#define RAWT 12
#define PFD 0
constexpr int NST = 6;
constexpr int STAGE_B = 24576;
constexpr size_t OFF_CALL = 0;
constexpr size_t OFF_TMO = 32768;
constexpr size_t OFF_CNT = 65536;
constexpr size_t OFF_PROF = 131072;
constexpr size_t OFF_FLAG = 1u << 20;
constexpr size_t OFF_OBI = 2u << 20;
constexpr size_t OFF_OBO = OFF_OBI + 1024ull * PIECE_B;
constexpr size_t OFF_XS = OFF_OBO + 512ull * PIECE_B;
constexpr size_t OFF_WS = OFF_XS + 64ull * 32 * PIECE_B;
constexpr size_t WS_NEED = OFF_WS + 64ull * 16 * PIECE_B;
static_assert(OFF_FLAG + (size_t)NR * ND * 256 * 8 <= OFF_OBI && NPW <= 256, "flags");
__host__ __device__ constexpr size_t flag_off(int src, int idx) { return OFF_FLAG + ((size_t)(src * ND + idx % ND) * 256 + idx / ND) * 8; }
constexpr int LDS_TAB = 0, LDS_ROLE = 2048;
constexpr int LDS_MISC = LDS_ROLE + NST * STAGE_B;
constexpr int LDS_BYTES = LDS_MISC + 512;
constexpr int LDS_SLOTS = LDS_ROLE, LDS_PRIV = LDS_SLOTS + NSLOT * PIECE_B, LDS_CTL = LDS_PRIV + 6 * CD_BYTES;
static_assert(LDS_CTL + 1280 <= LDS_MISC && NPW <= 128, "decoder layout");
static_assert(LDS_ROLE + 8 * CD_BYTES <= LDS_MISC && ND == 8 && NPUSH <= 32, "phase 0 scratch");
static_assert(16384 + 2 * CD_BYTES <= NST * STAGE_B, "epilogue scratch");
constexpr u64 DEADLINE = 40000000ull;

struct Params {
    const unsigned short* x;
    const unsigned short* w;
    unsigned short* out;
    unsigned char* ws[NR];
    int rank;
    int pad_;
};

typedef short bf16x8 __attribute__((ext_vector_type(8)));
typedef float f32x4 __attribute__((ext_vector_type(4)));
typedef short s16x4 __attribute__((ext_vector_type(4)));
#define GAS __attribute__((address_space(1)))

__device__ __forceinline__ void st_sys(void* p, u32x4 v) { asm volatile("global_store_dwordx4 %0, %1, off sc0 sc1\n\ts_nop 1" :: "v"(p), "v"(v) : "memory"); }
__device__ __forceinline__ void st_agent(void* p, u32x4 v) { asm volatile("global_store_dwordx4 %0, %1, off sc1\n\ts_nop 1" :: "v"(p), "v"(v) : "memory"); }
constexpr int wcnt(int vm, int lgkm) { return (vm & 15) | 0x70 | (lgkm << 8) | ((vm >> 4) << 14); }
__device__ __forceinline__ void wait_vm0() { asm volatile("s_waitcnt vmcnt(0)" ::: "memory"); }
__device__ __forceinline__ u32 rfl(u32 v) { return (u32)__builtin_amdgcn_readfirstlane((int)v); }
__device__ __forceinline__ u64 now() { return __builtin_amdgcn_s_memrealtime(); }

#define STAMP(slot) do { } while (0)

__device__ __forceinline__ void in_entry(int idx, int& c, bool& isw, int& bl) {
    if (idx < NINA) { const int e = idx + NPUSH; c = e / CHP; const int r = e - c * CHP; isw = r < 16; bl = isw ? r : r - 16; }
    else { const int e = idx - NINA; c = (e >> 1) & 7; isw = false; bl = GA + ((e >> 4) << 1) + (e & 1); }
}
__device__ __forceinline__ bool wait_cnt2(const unsigned char* c0, u32 t0v, const unsigned char* c1, u32 t1v, u32 call, u32* tmo) {
    const u64 t0 = now();
    for (;;) {
        const u32 a = __hip_atomic_load((const GAS u32*)c0, __ATOMIC_RELAXED, __HIP_MEMORY_SCOPE_SYSTEM);
        const u32 b = __hip_atomic_load((const GAS u32*)c1, __ATOMIC_RELAXED, __HIP_MEMORY_SCOPE_SYSTEM);
        if (rfl(a) >= t0v && rfl(b) >= t1v) return true;
        if (rfl(__hip_atomic_load((const GAS u32*)tmo, __ATOMIC_RELAXED, __HIP_MEMORY_SCOPE_AGENT)) == call || now() - t0 > DEADLINE) {
            __hip_atomic_store((GAS u32*)tmo, call, __ATOMIC_RELAXED, __HIP_MEMORY_SCOPE_AGENT);
            return false;
        }
        __builtin_amdgcn_s_sleep(2);
    }
}

__device__ __forceinline__ u32 encode_store(const u32 (&x)[32], unsigned char* slot, lds_u8* sc, const lds_u8* tab, int lane) {
    u32 A[16], L1[4];
    const u32 inf = codec_encode(x, A, L1, sc, tab, lane);
    unsigned char* o = slot + lane * 16;
    if (inf >> 31) {
#pragma unroll
        for (int j = 0; j < 8; ++j) st_sys(o + j * 1024, (u32x4){x[4 * j], x[4 * j + 1], x[4 * j + 2], x[4 * j + 3]});
    } else {
#pragma unroll
        for (int j = 0; j < 4; ++j) st_sys(o + j * 1024, (u32x4){A[4 * j], A[4 * j + 1], A[4 * j + 2], A[4 * j + 3]});
        st_sys(o + 4096, (u32x4){L1[0], L1[1], L1[2], L1[3]});
        const u32x4 v = *lp<u32x4>(sc + CD_V + lane * 16);
        if ((u32)lane * 16u < ((inf >> 8) & 0x7ffu)) st_sys(o + 5120, v);
    }
    return inf;
}

__device__ __forceinline__ u32 pk_bf16(float lo, float hi) { u32 r; asm("v_cvt_pk_bf16_f32 %0, %1, %2" : "=v"(r) : "v"(lo), "v"(hi)); return r; }

__global__ __launch_bounds__(NTH, 2) void gemm_ar_kernel(Params P) {
    extern __shared__ __attribute__((aligned(16))) unsigned char smem[];
    lds_u8* sm = (lds_u8*)smem;
    const int tid = threadIdx.x, lane = tid & 63, wid = (int)rfl(tid >> 6);
    const int b = blockIdx.x, xcd = b & 7, q = b >> 3;
    const int rank = P.rank, ri = rank >> 2, rj = rank & 3;
    unsigned char* my = P.ws[rank];
    u32* tmo = (u32*)(my + OFF_TMO);
    const lds_u8* tab = sm + LDS_TAB;
    STAMP(0);
    u32 call_v = 0;
    if (lane == 0) call_v = __hip_atomic_fetch_add((GAS u32*)(my + OFF_CALL + b * 128 + wid * 4), 1u, __ATOMIC_RELAXED, __HIP_MEMORY_SCOPE_AGENT) + 1u;

    unsigned char* wsl = P.ws[lane & 7];

    const bool is_dec = q >= 16 && q < 16 + ND && xcd < NR - 1;
    const int peer_r = (rank + 1 + xcd) & 7;
    const int pi = q * 4 + (wid - 4);
    const bool pusher = wid >= 4 && q < 8 && xcd < NR - 1 && pi < NPUSH;
    const int pid = pusher ? (pi < 16 ? 512 + (16 * (peer_r & 3) + pi) * 8 : (32 * (peer_r >> 2) + pi - 16) * 8) : b * 4 + wid;
    const bool enc = wid < 4 || pusher;
    const bool isw = pid >= 512;
    const int blk = (pid & 511) >> 3, ch = pid & 7;
    u32x4 raw[8];
    auto load_raw = [&]() {
        if (!isw) {
            const unsigned short* s = P.x + (size_t)(blk * 64 + (lane & 15)) * KSH + ch * 64 + (lane >> 4) * 8;
#pragma unroll
            for (int j = 0; j < 8; ++j) raw[j] = *(const u32x4*)(s + (size_t)(j >> 1) * 16 * KSH + (j & 1) * 32);
        } else {
            const unsigned short* s = P.w + (size_t)(ch * 64 + (lane >> 3)) * NN + blk * 64 + (lane & 7) * 8;
#pragma unroll
            for (int j = 0; j < 8; ++j) raw[j] = *(const u32x4*)(s + (size_t)j * 8 * NN);
        }
    };
    const int cls = pusher ? 0 : ((!isw && (blk & 31) >= GA) ? 2 : (ch >= PRI0 ? 1 : 0));
    if (enc && cls == 0) load_raw();
    if (tid < 256) codec_build_tables(sm + LDS_TAB, tid);
    if (is_dec && tid < 320) *lp<u32>(sm + LDS_CTL + tid * 4) = 0;
    __syncthreads();
    const u32 call = rfl(call_v);
    if (enc) {
        if (cls != 0) {
            for (int k = 0; k < cls * PRID - 1; ++k) __builtin_amdgcn_s_sleep(16);
            load_raw();
        }
        lds_u8* sc = sm + (is_dec ? LDS_PRIV : LDS_ROLE) + wid * CD_BYTES;
        u32 x[32];
        if (!isw) {
#pragma unroll
            for (int j = 0; j < 8; ++j) { x[4 * j] = raw[j].x; x[4 * j + 1] = raw[j].y; x[4 * j + 2] = raw[j].z; x[4 * j + 3] = raw[j].w; }
        } else {
#pragma unroll
            for (int j = 0; j < 8; ++j) { const int k = 8 * j + (lane >> 3); *lp<u32x4>(sc + k * 128 + (((lane & 7) ^ (k & 7)) << 4)) = raw[j]; }
            const int g = lane >> 4, qq = (lane & 15) >> 2, pp = lane & 3;
#pragma unroll
            for (int j = 0; j < 8; ++j) {
                const int ni = j >> 1, ks = j & 1;
#pragma unroll
                for (int t = 0; t < 2; ++t) {
                    const int k = 32 * ks + 8 * g + 4 * t + qq, chunk = 2 * ni + (pp >> 1);
                    const s16x4 v = __builtin_amdgcn_ds_read_tr16_b64_v4i16((LAS s16x4*)(sc + k * 128 + ((chunk ^ (k & 7)) << 4) + (pp & 1) * 8));
                    const u32x2 u = __builtin_bit_cast(u32x2, v);
                    x[4 * j + 2 * t] = u.x; x[4 * j + 2 * t + 1] = u.y;
                }
            }
            asm volatile("s_waitcnt lgkmcnt(0)" ::: "memory");
        }
        STAMP(1);
        const int rb = blk & 31, step = rank * 8 + ch;
        const size_t stg = isw ? OFF_WS + (size_t)(step * 16 + (blk & 15)) * PIECE_B : OFF_XS + (size_t)(step * 32 + rb) * PIECE_B;
        const size_t cnt = OFF_CNT + (size_t)(isw ? 128 + (blk & 15) * 8 + ch : (rb >> 1) * 8 + ch) * 128;
        const bool cons = lane < NR && (isw ? (lane & 3) == (blk >> 4) : (lane >> 2) == (blk >> 5));
        const bool mine = isw ? ((blk >> 4) == rj) : ((blk >> 5) == ri);
        if (pusher) {
            unsigned char* d = P.ws[peer_r] + stg + lane * 16;
#pragma unroll
            for (int j = 0; j < 8; ++j) {
                st_sys(d + j * 1024, (u32x4){x[4 * j], x[4 * j + 1], x[4 * j + 2], x[4 * j + 3]});
                if (PTHR > 0 && (j % PTHR) == PTHR - 1) wait_vm0();
            }
            wait_vm0();
            STAMP(2);
            if (lane == peer_r) __hip_atomic_fetch_add((GAS u32*)(wsl + cnt), 1u, __ATOMIC_RELAXED, __HIP_MEMORY_SCOPE_SYSTEM);
        } else if (ch == 0 && (isw ? NPUSH >= 16 : (rb < GA && NPUSH == CHP))) {
            if (mine) {
                unsigned char* d = my + stg + lane * 16;
#pragma unroll
                for (int j = 0; j < 8; ++j) st_sys(d + j * 1024, (u32x4){x[4 * j], x[4 * j + 1], x[4 * j + 2], x[4 * j + 3]});
                wait_vm0();
                if (lane == 0) __hip_atomic_fetch_add((GAS u32*)(my + cnt), 1u, __ATOMIC_RELAXED, __HIP_MEMORY_SCOPE_SYSTEM);
            }
        } else {
            u32 inf;
            if ((isw ? ch * CHP + (blk & 15) : (rb < GA ? ch * CHP + 16 + rb : 1 << 20)) < NPUSH + RAWH) {
                inf = 0x80000000u;
                unsigned char* ob = my + OFF_OBI + (size_t)pid * PIECE_B + lane * 16;
#pragma unroll
                for (int j = 0; j < 8; ++j) st_sys(ob + j * 1024, (u32x4){x[4 * j], x[4 * j + 1], x[4 * j + 2], x[4 * j + 3]});
            } else inf = encode_store(x, my + OFF_OBI + (size_t)pid * PIECE_B, sc, tab, lane);
            wait_vm0();
            STAMP(2);
            const int idx = isw ? ch * CHP + (blk & 15) - NPUSH : (rb < GA ? ch * CHP + 16 + rb - NPUSH : NINA + ((rb - GA) >> 1) * 16 + ch * 2 + (rb & 1));
            if (cons && lane != rank) __hip_atomic_store((GAS u64*)(wsl + flag_off(rank, idx)), (u64)call << 32 | inf, __ATOMIC_RELAXED, __HIP_MEMORY_SCOPE_SYSTEM);
            if (mine) {
                unsigned char* d = my + stg + lane * 16;
#pragma unroll
                for (int j = 0; j < 8; ++j) st_agent(d + j * 1024, (u32x4){x[4 * j], x[4 * j + 1], x[4 * j + 2], x[4 * j + 3]});
                wait_vm0();
                if (lane == 0) __hip_atomic_fetch_add((GAS u32*)(my + cnt), 1u, __ATOMIC_RELAXED, __HIP_MEMORY_SCOPE_AGENT);
            }
        }
        STAMP(3);
    }

    if (q < 16 || q >= 24) {
        __syncthreads();
        lds_u8* ring = sm + LDS_ROLE;
        const int wr = wid >> 1, wc = wid & 1;
        const int cb = 4 * (xcd & 3) + (q & 3);
        bool bad = false;
        const int ntile = q < 8 ? 2 : 1;
        const int q2 = q < 16 ? q : q - 8;
        for (int t = 0; t < ntile; ++t) {
            const int a = t == 1 ? 12 + 2 * (xcd >> 2) + (q >> 2) : (q2 < 12 ? 3 * (xcd >> 2) + (q2 >> 2) : 6 + 3 * (xcd >> 2) + ((q2 - 12) >> 2));
            const int g = t;
            const unsigned char* gx = my + OFF_XS + (size_t)(2 * a) * PIECE_B;
            const unsigned char* gw = my + OFF_WS + (size_t)cb * PIECE_B;
            const bool ldA = wid < 4;
            const unsigned char* gl = (ldA ? gx + wid * 4096 : gw + (wid - 4) * 4096) + lane * 16;
            const size_t lstep = ldA ? (size_t)32 * PIECE_B : (size_t)16 * PIECE_B;
            f32x4 acc[2][2] = {};
            auto issue = [&](int i) {
                const int step = (i & 7) * 8 + (i >> 3);
                lds_u8* d = ring + (i % NST) * STAGE_B + wid * 4096;
                const unsigned char* sa = gl + (size_t)step * lstep;
#pragma unroll
                for (int j = 0; j < 4; ++j) __builtin_amdgcn_global_load_lds((const GAS u32*)(sa + j * 1024), (LAS u32*)(d + j * 1024), 16, 0, 17);
            };
            const unsigned char* cx = my + OFF_CNT + (size_t)(a * 8) * 128; const unsigned char* cw = my + OFF_CNT + (size_t)(128 + cb * 8) * 128;
            volatile LAS u32* rdyw = (volatile LAS u32*)(sm + LDS_MISC + 256);
            int rdy = 0;
            u32 pa = 0, pb = 0; bool pend = false;
            auto poll_issue = [&]() { pa = __hip_atomic_load((const GAS u32*)(cx + rdy * 128), __ATOMIC_RELAXED, __HIP_MEMORY_SCOPE_SYSTEM); pb = __hip_atomic_load((const GAS u32*)(cw + rdy * 128), __ATOMIC_RELAXED, __HIP_MEMORY_SCOPE_SYSTEM); pend = true; };
            auto poll_take = [&]() { if (pend) { pend = false; if (rfl(pa) >= 16u * call && rfl(pb) >= 8u * call) { ++rdy; if (lane == 0) *rdyw = rdy; } } };
            auto need = [&](int c) {
                poll_take();
                while (rdy <= c) {
                    if (!wait_cnt2(cx + rdy * 128, 16u * call, cw + rdy * 128, 8u * call, call, tmo)) bad = true;
                    ++rdy; if (lane == 0) *rdyw = rdy;
                }
            };
            int tp = NST;
            const unsigned char* gt = lane < 32 ? gx + (q & 3) * 4096 + lane * 128 : gw + ((q >> 2) & 3) * 2048 + (lane & 15) * 128;
            const size_t tstep = lane < 32 ? (size_t)32 * PIECE_B : (size_t)16 * PIECE_B;
            auto warm = [&](int i) {
                const int lim = min(min(64, (int)rfl(*rdyw) * 8), i + PFD);
#pragma unroll 1
                for (int k = 0; k < 3 && tp < lim; ++k, ++tp)
                    __builtin_amdgcn_global_load_lds((const GAS u32*)(gt + (size_t)((tp & 7) * 8 + (tp >> 3)) * tstep), (LAS u32*)(sm + LDS_MISC), 4, 0, 17);
            };
            auto rd = [&](int i, bf16x8 (&af)[2][2], bf16x8 (&bfr)[2][2]) {
                lds_u8* st = ring + (i % NST) * STAGE_B;
#pragma unroll
                for (int m = 0; m < 2; ++m)
#pragma unroll
                    for (int ks = 0; ks < 2; ++ks) af[m][ks] = *lp<bf16x8>(st + (wr >> 1) * 8192 + (((wr & 1) * 2 + m) * 2 + ks) * 1024 + lane * 16);
#pragma unroll
                for (int n = 0; n < 2; ++n)
#pragma unroll
                    for (int ks = 0; ks < 2; ++ks) bfr[n][ks] = *lp<bf16x8>(st + 16384 + ((wc * 2 + n) * 2 + ks) * 1024 + lane * 16);
            };
            auto mm = [&](bf16x8 (&af)[2][2], bf16x8 (&bfr)[2][2]) {
#pragma unroll
                for (int ks = 0; ks < 2; ++ks)
#pragma unroll
                    for (int m = 0; m < 2; ++m)
#pragma unroll
                        for (int n = 0; n < 2; ++n) acc[m][n] = __builtin_amdgcn_mfma_f32_16x16x32_bf16(bfr[n][ks], af[m][ks], acc[m][n], 0, 0, 0);
            };
            auto kloop = [&](auto role_c) {
                constexpr int ROLE = decltype(role_c)::value;
                bf16x8 a0[2][2], b0[2][2], a1[2][2], b1[2][2];
                if constexpr (ROLE == 1) { if (lane == 0) *rdyw = 0; need(0); }
                __builtin_amdgcn_s_barrier();
                if (g == 0) STAMP(4); else STAMP(12);
                if constexpr (ROLE == 0) {
#pragma unroll
                    for (int i = 0; i < NST - 1; ++i) issue(i);
                    __builtin_amdgcn_s_waitcnt(wcnt(4 * (NST - 2), 15));
                }
                __builtin_amdgcn_s_barrier();
                rd(0, a0, b0);
                auto pair = [&](int i, auto tail_c) {
                    constexpr int TAIL = decltype(tail_c)::value;
                    { const int nx = i + NST - 1;
                      if constexpr (ROLE == 0) __builtin_amdgcn_s_waitcnt(wcnt(TAIL == 0 || TAIL + 4 < 64 ? 4 * (NST - 3) : 4 * (63 - TAIL - 1 > 0 ? 63 - TAIL - 1 : 0), 0));
                      else __builtin_amdgcn_s_waitcnt(wcnt(63, 0));
                      __builtin_amdgcn_s_barrier();
                      if constexpr (ROLE == 0 && (TAIL == 0 || TAIL + NST - 1 < 64)) issue(nx);
                      rd(i + 1, a1, b1);
                      __builtin_amdgcn_sched_barrier(0);
                      mm(a0, b0);
                      __builtin_amdgcn_sched_barrier(0);
                      if constexpr (ROLE == 1 && TAIL == 0) { if ((i & 2) == 0 && rdy < 8) { poll_take(); if (rdy < 8) poll_issue(); } }
                      if constexpr (ROLE == 2 && TAIL == 0) warm(i); }
                    { const int nx = i + NST;
                      if constexpr (ROLE == 1 && TAIL == 0) { if ((nx & 7) == 0) need(nx >> 3); }
                      if constexpr (ROLE == 0) __builtin_amdgcn_s_waitcnt(wcnt(TAIL == 0 ? 4 * (NST - 3) : 4 * (63 - TAIL - 2 > 0 ? 63 - TAIL - 2 : 0), 0));
                      else __builtin_amdgcn_s_waitcnt(wcnt(63, 0));
                      __builtin_amdgcn_s_barrier();
                      if constexpr (ROLE == 0 && TAIL == 0) issue(nx);
                      if constexpr (TAIL != 62) rd(i + 2, a0, b0);
                      __builtin_amdgcn_sched_barrier(0);
                      mm(a1, b1);
                      __builtin_amdgcn_sched_barrier(0);
                      if constexpr (ROLE == 2 && TAIL == 0) warm(i + 1); }
                };
#pragma unroll 1
                for (int i = 0; i < 58; i += 2) pair(i, std::integral_constant<int, 0>{});
                pair(58, std::integral_constant<int, 58>{});
                pair(60, std::integral_constant<int, 60>{});
                pair(62, std::integral_constant<int, 62>{});
            };
            if (wid < 6) kloop(std::integral_constant<int, 0>{});
            else if (wid == 6) kloop(std::integral_constant<int, 1>{});
            else kloop(std::integral_constant<int, 2>{});
            wait_vm0();
            if (g == 0) STAMP(5); else STAMP(13);
            __syncthreads();
#pragma unroll
            for (int m = 0; m < 2; ++m)
#pragma unroll
                for (int n = 0; n < 2; ++n) {
                    const int row = 32 * wr + 16 * m + (lane & 15), col = 32 * wc + 16 * n + 4 * (lane >> 4);
                    const u32x2 v = {pk_bf16(acc[m][n][0], acc[m][n][1]), pk_bf16(acc[m][n][2], acc[m][n][3])};
                    *lp<u32x2>(ring + row * 128 + (((col >> 3) ^ (row & 7)) << 4) + (col & 4) * 2) = v;
                }
            if (tid == 6 * 64) *lp<u32>(sm + LDS_MISC + 272) = (bad || __hip_atomic_load((const GAS u32*)tmo, __ATOMIC_RELAXED, __HIP_MEMORY_SCOPE_AGENT) == call) ? 1u : 0u;
            __syncthreads();
            if (wid < 4) {
                const int h = wid & 1;
                const bool poison = *lp<u32>(sm + LDS_MISC + 272) != 0u;
                u32 x[32];
#pragma unroll
                for (int j = 0; j < 8; ++j) {
                    const int row = 64 * h + 8 * j + (lane >> 3);
                    u32x4 v = *lp<u32x4>(ring + row * 128 + (((lane & 7) ^ (row & 7)) << 4));
                    if (poison) v = (u32x4){0x7fc07fc0u, 0x7fc07fc0u, 0x7fc07fc0u, 0x7fc07fc0u};
                    x[4 * j] = v.x; x[4 * j + 1] = v.y; x[4 * j + 2] = v.z; x[4 * j + 3] = v.w;
                }
                const int o = (a * 16 + cb) * 2 + h;
                if (wid < 2) {
                    u32 inf;
                    if (o >= NOUT - RAWT) {
                        inf = 0x80000000u;
                        unsigned char* ob = my + OFF_OBO + (size_t)o * PIECE_B + lane * 16;
#pragma unroll
                        for (int j = 0; j < 8; ++j) st_sys(ob + j * 1024, (u32x4){x[4 * j], x[4 * j + 1], x[4 * j + 2], x[4 * j + 3]});
                    } else inf = encode_store(x, my + OFF_OBO + (size_t)o * PIECE_B, ring + 16384 + wid * CD_BYTES, tab, lane);
                    wait_vm0();
                    if (lane < NR && lane != rank)
                        __hip_atomic_store((GAS u64*)(wsl + flag_off(rank, NIN + o)), (u64)call << 32 | inf, __ATOMIC_RELAXED, __HIP_MEMORY_SCOPE_SYSTEM);
                } else {
                    unsigned short* d = P.out + (size_t)(2048 * ri + 128 * a + 64 * h + (lane >> 3)) * NN + 1024 * rj + 64 * cb + (lane & 7) * 8;
#pragma unroll
                    for (int j = 0; j < 8; ++j) st_agent(d + (size_t)j * 8 * NN, (u32x4){x[4 * j], x[4 * j + 1], x[4 * j + 2], x[4 * j + 3]});
                }
            }
            if (g == 0) STAMP(6); else STAMP(14);
            __syncthreads();
        }
    } else if (is_dec) {
        const int src = peer_r, si = src >> 2, sj = src & 3;
        const unsigned char* peer = P.ws[src];
        const int dj = q - 16;
        const int np = (NQ - dj + ND - 1) / ND;
        volatile LAS u32* c_info = (volatile LAS u32*)(sm + LDS_CTL);
        volatile LAS u32* c_full = (volatile LAS u32*)(sm + LDS_CTL + 1024);
        volatile LAS u32* c_done = (volatile LAS u32*)(sm + LDS_CTL + 1088);
        volatile LAS u32* c_piece = (volatile LAS u32*)(sm + LDS_CTL + 1152);
        LAS u32* c_ticket = (LAS u32*)(sm + LDS_CTL + 1216);
        auto src_slot = [&](int idx) -> const unsigned char* {
            if (idx >= NIN) return peer + OFF_OBO + (size_t)(idx - NIN) * PIECE_B;
            int c, bl; bool isw2; in_entry(idx, c, isw2, bl);
            return peer + OFF_OBI + (size_t)(isw2 ? 512 + (16 * rj + bl) * 8 + c : (32 * ri + bl) * 8 + c) * PIECE_B;
        };
        if (wid == 6) {
            const unsigned char* fb = my + OFF_FLAG + (size_t)(src * ND + dj) * 256 * 8;
            const u64 t0 = now();
            bool p0 = lane >= np, p1 = lane + 64 >= np;
            int rounds = 0;
            for (;;) {
                u64 v0 = 0, v1 = 0;
                if (!p0) v0 = __hip_atomic_load((const GAS u64*)(fb + (size_t)lane * 8), __ATOMIC_RELAXED, __HIP_MEMORY_SCOPE_SYSTEM);
                if (!p1) v1 = __hip_atomic_load((const GAS u64*)(fb + (size_t)(lane + 64) * 8), __ATOMIC_RELAXED, __HIP_MEMORY_SCOPE_SYSTEM);
                if (!p0 && (u32)(v0 >> 32) >= call) { c_info[lane] = (u32)v0 | 0x40000000u; p0 = true; }
                if (!p1 && (u32)(v1 >> 32) >= call) { c_info[lane + 64] = (u32)v1 | 0x40000000u; p1 = true; }
                if (__all(p0 && p1)) break;
                if ((++rounds & 7) == 0 && (rfl(__hip_atomic_load((const GAS u32*)tmo, __ATOMIC_RELAXED, __HIP_MEMORY_SCOPE_AGENT)) == call || now() - t0 > DEADLINE)) {
                    __hip_atomic_store((GAS u32*)tmo, call, __ATOMIC_RELAXED, __HIP_MEMORY_SCOPE_AGENT);
                    if (!p0) c_info[lane] = 0x60000000u;
                    if (!p1) c_info[lane + 64] = 0x60000000u;
                    break;
                }
                __builtin_amdgcn_s_sleep(2);
            }
        } else if (wid == 7) {
            int pub = 0;
            auto publish_upto = [&](int e) { for (; pub < e; ++pub) if (lane == 0) c_full[pub % NSLOT] = pub + 1; };
            u64 todo0 = np >= 64 ? ~0ull : (1ull << np) - 1ull, todo1 = np > 64 ? (1ull << (np - 64)) - 1ull : 0ull;
            int n = 0; u32 inf = 0;
            auto pick = [&](int s) -> bool {
                const u32 v0 = c_info[lane], v1 = c_info[lane + 64], dn = c_done[s % NSLOT];
                const u64 a0 = __ballot(v0 != 0u) & todo0, a1 = __ballot(v1 != 0u) & todo1;
                if ((a0 | a1) == 0ull || !(s < NSLOT || rfl(dn) == (u32)(s - NSLOT + 1))) return false;
                if (a0) { const int l = __builtin_ctzll(a0); todo0 &= ~(1ull << l); n = l; inf = (u32)__builtin_amdgcn_readlane((int)v0, l); }
                else { const int l = __builtin_ctzll(a1); todo1 &= ~(1ull << l); n = 64 + l; inf = (u32)__builtin_amdgcn_readlane((int)v1, l); }
                return true;
            };
            bool have = false;
            for (int s = 0; s < np; ++s) {
                const int slot = s % NSLOT;
                if (!have) { wait_vm0(); publish_upto(s); while (!pick(s)) __builtin_amdgcn_s_sleep(1); }
                if (lane == 0) c_piece[slot] = n;
                lds_u8* d = sm + LDS_SLOTS + slot * PIECE_B;
                const unsigned char* s0 = src_slot(dj + n * ND);
                const unsigned char* sp = s0 + lane * 16;
                if (inf & 0x20000000u) { have = false; wait_vm0(); publish_upto(s + 1); continue; }
                if (inf >> 31) {
#pragma unroll
                    for (int j = 0; j < 8; ++j) {
                        asm volatile("s_waitcnt vmcnt(%0)" :: "n"(NFL - 1) : "memory");
                        __builtin_amdgcn_global_load_lds((const GAS u32*)(sp + j * 1024), (LAS u32*)(d + j * 1024), 16, 0, 17);
                    }
                    have = s + 1 < np && pick(s + 1);
                    asm volatile("s_waitcnt vmcnt(8)" ::: "memory"); publish_upto(s);
                    wait_vm0(); publish_upto(s + 1);
                } else {
                    const unsigned char* sv = ((u32)lane * 16u < ((inf >> 8) & 0x7ffu)) ? sp + 5120 : s0 + 5120;
                    auto pull6 = [&](auto nfl_c) {
                        constexpr int N = decltype(nfl_c)::value;
#pragma unroll
                        for (int j = 0; j < 6; ++j) {
                            asm volatile("s_waitcnt vmcnt(%0)" :: "n"(N - 1) : "memory");
                            publish_upto(s + (j - N + 1 - 5) / 6);
                            __builtin_amdgcn_global_load_lds((const GAS u32*)(j < 5 ? sp + j * 1024 : sv), (LAS u32*)(d + j * 1024), 16, 0, 17);
                            if (j == 1) have = s + 1 < np && pick(s + 1);
                        }
                    };
                    const int cur_n = n;
                    if (dj + cur_n * ND < NIN) pull6(std::integral_constant<int, NFLI>{}); else pull6(std::integral_constant<int, NFL>{});
                }
            }
            wait_vm0(); publish_upto(np);
            STAMP(9);
        } else {
            lds_u8* sc = sm + LDS_PRIV + wid * CD_BYTES;
            int nstamp = 0;
            for (;;) {
                u32 nn = 0;
                if (lane == 0) nn = __hip_atomic_fetch_add(c_ticket, 1u, __ATOMIC_RELAXED, __HIP_MEMORY_SCOPE_WORKGROUP);
                const int sq = (int)rfl(nn);
                if (sq >= np) break;
                while (rfl(c_full[sq % NSLOT]) != (u32)(sq + 1)) __builtin_amdgcn_s_sleep(1);
                asm volatile("" ::: "memory");
                const int n = (int)rfl(c_piece[sq % NSLOT]);
                const u32 inf = rfl(c_info[n]);
                const bool ok = !(inf & 0x20000000u);
                lds_u8* sl = sm + LDS_SLOTS + (sq % NSLOT) * PIECE_B;
                u32 x[32];
                if (!ok) {
#pragma unroll
                    for (int j = 0; j < 32; ++j) x[j] = 0x7fc07fc0u;
                } else if (inf >> 31) {
#pragma unroll
                    for (int j = 0; j < 8; ++j) { const u32x4 v = *lp<u32x4>(sl + j * 1024 + lane * 16); x[4 * j] = v.x; x[4 * j + 1] = v.y; x[4 * j + 2] = v.z; x[4 * j + 3] = v.w; }
                } else {
                    u32 A[16], L1[4];
#pragma unroll
                    for (int j = 0; j < 4; ++j) { const u32x4 v = *lp<u32x4>(sl + j * 1024 + lane * 16); A[4 * j] = v.x; A[4 * j + 1] = v.y; A[4 * j + 2] = v.z; A[4 * j + 3] = v.w; }
                    { const u32x4 v = *lp<u32x4>(sl + 4096 + lane * 16); L1[0] = v.x; L1[1] = v.y; L1[2] = v.z; L1[3] = v.w; }
                    codec_decode(x, A, L1, inf & 255u, (const LAS u32*)(sl + 5120), sc, tab, lane);
                }
                asm volatile("s_waitcnt lgkmcnt(0)" ::: "memory");
                if (lane == 0) c_done[sq % NSLOT] = sq + 1;
                const int idx = dj + n * ND;
                if (idx < NIN) {
                    int c, bl; bool isw2; in_entry(idx, c, isw2, bl);
                    const int step = src * 8 + c;
                    unsigned char* d = (isw2 ? my + OFF_WS + (size_t)(step * 16 + bl) * PIECE_B : my + OFF_XS + (size_t)(step * 32 + bl) * PIECE_B) + lane * 16;
                    if (ok) {
#pragma unroll
                        for (int j = 0; j < 8; ++j) st_agent(d + j * 1024, (u32x4){x[4 * j], x[4 * j + 1], x[4 * j + 2], x[4 * j + 3]});
                        wait_vm0();
                        if (lane == 0) __hip_atomic_fetch_add((GAS u32*)(my + OFF_CNT + (size_t)(isw2 ? 128 + bl * 8 + c : (bl >> 1) * 8 + c) * 128), 1u, __ATOMIC_RELAXED, __HIP_MEMORY_SCOPE_AGENT);
                    }
                } else {
                    const int o = idx - NIN, g = o >> 8, tile = (o & 255) >> 1, h = o & 1;
                    unsigned short* d = P.out + (size_t)(2048 * si + 1024 * g + 128 * (tile >> 4) + 64 * h + (lane >> 3)) * NN + 1024 * sj + 64 * (tile & 15) + (lane & 7) * 8;
#pragma unroll
                    for (int j = 0; j < 8; ++j) st_agent(d + (size_t)j * 8 * NN, (u32x4){x[4 * j], x[4 * j + 1], x[4 * j + 2], x[4 * j + 3]});
                }
            }
            (void)nstamp;
        }
    }
    STAMP(31);
}

extern "C" void kernel_launch(int rank, int nranks, ncclComm_t comm, hipStream_t stream,
                              void* const* d_in, const int* in_sizes, int n_in,
                              void* d_out, int out_size, const int* in_dtypes, int out_dtype,
                              void* const* d_out_all, void* const* d_ws_all, size_t ws_bytes) {
    (void)comm; (void)in_dtypes; (void)out_dtype; (void)d_out_all;
    if (nranks != NR || n_in != 2 || rank < 0 || rank >= NR || in_sizes[0] != MM * KSH || in_sizes[1] != KSH * NN || out_size != MM * NN || ws_bytes < WS_NEED) {
        fprintf(stderr, "kernel_launch: unexpected problem shape\n");
        return;
    }
    static bool first = true;
    if (first) {
        hipError_t e = hipFuncSetAttribute((const void*)gemm_ar_kernel, hipFuncAttributeMaxDynamicSharedMemorySize, LDS_BYTES);
        if (e != hipSuccess) { fprintf(stderr, "hipFuncSetAttribute: %s\n", hipGetErrorString(e)); return; }
        first = false;
    }
    Params P;
    P.x = (const unsigned short*)d_in[0];
    P.w = (const unsigned short*)d_in[1];
    P.out = (unsigned short*)d_out;
    for (int p = 0; p < NR; ++p) P.ws[p] = (unsigned char*)d_ws_all[p];
    P.rank = rank;
    P.pad_ = 0;
    hipLaunchKernelGGL(gemm_ar_kernel, dim3(NWG), dim3(NTH), LDS_BYTES, stream, P);
    hipError_t e = hipGetLastError();
    if (e != hipSuccess) fprintf(stderr, "launch: %s\n", hipGetErrorString(e));
}
```

```cpp
#include <hip/hip_runtime.h>
#include <rccl/rccl.h>
#include <cstdio>
#include <type_traits>

typedef unsigned u32;
typedef unsigned long long u64;
typedef unsigned u32x4 __attribute__((ext_vector_type(4)));
typedef unsigned u32x2 __attribute__((ext_vector_type(2)));
typedef unsigned short u16x2 __attribute__((ext_vector_type(2)));
#define LAS __attribute__((address_space(3)))
typedef LAS unsigned char lds_u8;

constexpr int CD_V = 0;
constexpr int CD_R2 = 1040;
constexpr int CD_R3 = CD_R2 + 64 * 68;
constexpr int CD_R4 = CD_R3 + 64 * 36;
constexpr int CD_BYTES = CD_R4 + 64 * 20;
constexpr int CD_DEC_R2 = 1040, CD_DEC_R3 = CD_DEC_R2 + 64 * 36, CD_DEC_BYTES = CD_DEC_R3 + 64 * 20;
constexpr int CD_TAB_BYTES = 2048;
static_assert(CD_BYTES % 16 == 0 && CD_DEC_BYTES % 16 == 0, "align");

__device__ __forceinline__ u32 perm(u32 hi, u32 lo, u32 sel) { return __builtin_amdgcn_perm(hi, lo, sel); }
__device__ __forceinline__ u32 dot4(u32 a, u32 b, u32 c) { return __builtin_amdgcn_udot4(a, b, c, false); }
__device__ __forceinline__ u32 pk_sub(u32 a, u32 b) { return __builtin_bit_cast(u32, (u16x2)(__builtin_bit_cast(u16x2, a) - __builtin_bit_cast(u16x2, b))); }
__device__ __forceinline__ u32 pk_min(u32 a, u32 b) { return __builtin_bit_cast(u32, __builtin_elementwise_min(__builtin_bit_cast(u16x2, a), __builtin_bit_cast(u16x2, b))); }
__device__ __forceinline__ u32 wave_incl_scan(u32 x) {
    x += (u32)__builtin_amdgcn_update_dpp(0, (int)x, 0x111, 0xf, 0xf, false);
    x += (u32)__builtin_amdgcn_update_dpp(0, (int)x, 0x112, 0xf, 0xf, false);
    x += (u32)__builtin_amdgcn_update_dpp(0, (int)x, 0x114, 0xf, 0xf, false);
    x += (u32)__builtin_amdgcn_update_dpp(0, (int)x, 0x118, 0xf, 0xf, false);
    x += (u32)__builtin_amdgcn_update_dpp(0, (int)x, 0x142, 0xa, 0xf, false);
    x += (u32)__builtin_amdgcn_update_dpp(0, (int)x, 0x143, 0xc, 0xf, false);
    return x;
}
template <class T> __device__ __forceinline__ LAS T* lp(lds_u8* p) { return (LAS T*)p; }
#define CFENCE() asm volatile("" ::: "memory")

__device__ __forceinline__ void codec_build_tables(lds_u8* tab, int i) {
    u32 dep = 0, cmp = 0x0c0c0c0cu; int n = 0;
#pragma unroll
    for (int k = 0; k < 4; ++k) {
        const u32 c = (i >> (2 * k)) & 3;
        if (c == 3) { dep |= (u32)n << (8 * k); cmp = (cmp & ~(0xffu << (8 * n))) | ((u32)k << (8 * n)); ++n; }
        else dep |= (4u + c) << (8 * k);
    }
    lp<u32>(tab)[i] = dep; lp<u32>(tab)[256 + i] = cmp;
}

__device__ __forceinline__ void stream_append(LAS u32* r, u32 pk, u32& p, u32 pnew, u32& acc) {
    const u64 w = (u64)pk << ((p & 3u) * 8u);
    const u32 lo = (u32)w | acc;
    r[p >> 2] = lo;
    acc = ((p ^ pnew) & ~3u) ? (u32)(w >> 32) : lo;
    p = pnew;
}

__device__ __forceinline__ u32 codec_encode(const u32 (&x)[32], u32 (&A)[16], u32 (&L1)[4], lds_u8* sc, const lds_u8* tab, int lane) {
    const LAS u32* CMP = (const LAS u32*)(tab + 1024);
    LAS u32* r2 = (LAS u32*)(sc + CD_R2) + lane * 17; LAS u32* r3 = (LAS u32*)(sc + CD_R3) + lane * 9; LAS u32* r4 = (LAS u32*)(sc + CD_R4) + lane * 5;
    const u32x4 z4 = {0, 0, 0, 0};
#pragma unroll
    for (int k = 0; k < 8; ++k) r2[k] = 0;
#pragma unroll
    for (int k = 0; k < 4; ++k) r3[k] = 0;
    r4[0] = 0; r4[1] = 0;
    *lp<u32x4>(sc + CD_V + lane * 16) = z4; if (lane == 0) *lp<u32x4>(sc + CD_V + 1024) = z4;
    u32 B;
    { const u32 y0 = __builtin_amdgcn_alignbit(x[0], x[0], 31), y1 = __builtin_amdgcn_alignbit(x[1], x[1], 31);
      const u32 s = wave_incl_scan(dot4(perm(y1, y0, 0x07050301u), 0x01010101u, 0u));
      B = ((u32)__builtin_amdgcn_readlane((int)s, 63) + 724u) >> 8; B = B > 255u ? 255u : B; }
    const u32 BK = (B << 8 | 0xffu) * 0x00010001u;
    u32 tb[16], bb[16], pn[16];
    u32 pc = 0;
#pragma unroll
    for (int g = 0; g < 16; ++g) {
        const u32 y0 = __builtin_amdgcn_alignbit(x[2 * g], x[2 * g], 31), y1 = __builtin_amdgcn_alignbit(x[2 * g + 1], x[2 * g + 1], 31);
        A[g] = perm(y1, y0, 0x06040200u);
        const u32 d0 = pk_sub(BK, y0), d1 = pk_sub(BK, y1);
        const u32 m0 = pk_min(d0, 0x07ff07ffu), m1 = pk_min(d1, 0x07ff07ffu);
        const u32 tc = perm(m1, m0, 0x07050301u);
        tb[g] = perm(d1, d0, 0x07050301u) ^ 0x04040404u;
        const u32 c1 = perm(0x03030303u, 0x02010003u, tc), ef = perm(0x01010101u, 0x00000001u, tc);
        bb[g] = dot4(c1, 0x40100401u, 0u);
        pc = dot4(ef, 0x01010101u, pc); pn[g] = pc;
        if ((g & 3) == 0) L1[g >> 2] = bb[g]; else L1[g >> 2] |= bb[g] << (8 * (g & 3));
    }
    u32 sel[16];
#pragma unroll
    for (int g = 0; g < 16; ++g) sel[g] = CMP[bb[g]];
    CFENCE();
    u32 p2 = 0, acc = 0;
#pragma unroll
    for (int g = 0; g < 16; ++g) stream_append(r2, perm(0u, tb[g], sel[g]), p2, pn[g], acc);
    r2[p2 >> 2] = acc;
    CFENCE();
    u32 C2[2], p3 = 0;
    { u32 u[8], c[8], b[8], s2[8];
#pragma unroll
      for (int k = 0; k < 8; ++k) u[k] = r2[k];
#pragma unroll
      for (int k = 0; k < 8; ++k) {
          const u32 big = perm(0u, 0u, ((u[k] & 0xf8f8f8f8u) >> 1) + 0x0c0c0c0cu);
          c[k] = perm(0x03030302u, 0x03030100u, u[k] & 0x07070707u) | (big & 0x03030303u);
          b[k] = dot4(c[k], 0x40100401u, 0u);
          if ((k & 3) == 0) C2[k >> 2] = b[k]; else C2[k >> 2] |= b[k] << (8 * (k & 3));
      }
#pragma unroll
      for (int k = 0; k < 8; ++k) s2[k] = CMP[b[k]];
      CFENCE();
      acc = 0;
#pragma unroll
      for (int k = 0; k < 8; ++k)
          stream_append(r3, perm(0u, u[k] ^ 0x02020202u, s2[k]), p3, dot4(perm(0u, 0x01000000u, c[k]), 0x01010101u, p3), acc);
      r3[p3 >> 2] = acc;
      CFENCE(); }
    u32 C3 = 0, p4 = 0;
    { u32 u[4], c[4], b[4], s3[4];
#pragma unroll
      for (int k = 0; k < 4; ++k) u[k] = r3[k];
#pragma unroll
      for (int k = 0; k < 4; ++k) {
          const u32 big = perm(0u, 0u, ((u[k] & 0xfefefefeu) >> 1) + 0x0c0c0c0cu);
          const u32 n14 = perm(0u, 0u, (((u[k] ^ 0x0e0e0e0eu) & 0x7f7f7f7fu) + 0x0c0c0c0cu) | (u[k] & 0x80808080u));
          c[k] = (u[k] & 0x01010101u & ~big) | (big & (0x02020202u | (n14 & 0x01010101u)));
          b[k] = dot4(c[k], 0x40100401u, 0u);
          C3 |= b[k] << (8 * k);
      }
#pragma unroll
      for (int k = 0; k < 4; ++k) s3[k] = CMP[b[k]];
      CFENCE();
      acc = 0;
#pragma unroll
      for (int k = 0; k < 4; ++k) {
          const u32 pk = perm(0u, u[k] ^ 0x06060606u, s3[k]);
          stream_append(r4, pk, p4, dot4(perm(0u, 0x01000000u, c[k]), 0x01010101u, p4), acc);
      }
      r4[p4 >> 2] = acc;
      CFENCE(); }
    const bool over = p2 > 31u || p3 > 15u || p4 > 8u;
    if (__any(over)) return 0x80000000u;
    const u32 q = p2 | p3 << 11 | p4 << 21, qi = wave_incl_scan(q), qe = qi - q;
    const u32 tot = (u32)__builtin_amdgcn_readlane((int)qi, 63);
    const u32 o3 = ((2 * (tot & 0x7ffu) + 31u) >> 5) * 4u, o4 = o3 + ((2 * ((tot >> 11) & 0x3ffu) + 31u) >> 5) * 4u, vlen = o4 + (tot >> 21);
    if (vlen > 1024u) return 0x80000000u;
    LAS u32* V = (LAS u32*)(sc + CD_V);
    { const u32 bit = 2 * (qe & 0x7ffu), sh = bit & 31u; LAS u32* a = V + (bit >> 5);
      const u64 w = ((u64)C2[1] << 32 | C2[0]) << sh;
      __hip_atomic_fetch_or(a, (u32)w, __ATOMIC_RELAXED, __HIP_MEMORY_SCOPE_WAVEFRONT);
      __hip_atomic_fetch_or(a + 1, (u32)(w >> 32), __ATOMIC_RELAXED, __HIP_MEMORY_SCOPE_WAVEFRONT);
      __hip_atomic_fetch_or(a + 2, (C2[1] >> 1) >> (31u - sh), __ATOMIC_RELAXED, __HIP_MEMORY_SCOPE_WAVEFRONT); }
    { const u32 bit = 8 * o3 + 2 * ((qe >> 11) & 0x3ffu), sh = bit & 31u; LAS u32* a = V + (bit >> 5);
      const u64 w = (u64)C3 << sh;
      __hip_atomic_fetch_or(a, (u32)w, __ATOMIC_RELAXED, __HIP_MEMORY_SCOPE_WAVEFRONT);
      __hip_atomic_fetch_or(a + 1, (u32)(w >> 32), __ATOMIC_RELAXED, __HIP_MEMORY_SCOPE_WAVEFRONT); }
    { const u32 r0 = r4[0], r1 = r4[1];
      const u32 bit = 8 * (o4 + (qe >> 21)), sh = bit & 31u; LAS u32* a = V + (bit >> 5);
      const u64 w = ((u64)r1 << 32 | r0) << sh;
      __hip_atomic_fetch_or(a, (u32)w, __ATOMIC_RELAXED, __HIP_MEMORY_SCOPE_WAVEFRONT);
      __hip_atomic_fetch_or(a + 1, (u32)(w >> 32), __ATOMIC_RELAXED, __HIP_MEMORY_SCOPE_WAVEFRONT);
      __hip_atomic_fetch_or(a + 2, (r1 >> 1) >> (31u - sh), __ATOMIC_RELAXED, __HIP_MEMORY_SCOPE_WAVEFRONT); }
    CFENCE();
    return B | vlen << 8;
}

__device__ __forceinline__ void codec_decode(u32 (&x)[32], const u32 (&A)[16], const u32 (&L1)[4], u32 B, const LAS u32* V, lds_u8* sc, const lds_u8* tab, int lane) {
    const LAS u32* DEP = (const LAS u32*)tab;
    LAS u32* r2 = (LAS u32*)(sc + CD_DEC_R2) + lane * 9; LAS u32* r3 = (LAS u32*)(sc + CD_DEC_R3) + lane * 5;
    const u32 T1 = ((B - 1u) & 255u) | ((B - 2u) & 255u) << 8 | ((B - 3u) & 255u) << 16;
    const u32 T2 = ((B - 4u) & 255u) | ((B - 5u) & 255u) << 8 | (B & 255u) << 16;
    const u32 T3 = ((B - 6u) & 255u) | ((B - 7u) & 255u) << 8 | ((B - 8u) & 255u) << 16;
    u32 e1[4], n2 = 0;
#pragma unroll
    for (int k = 0; k < 4; ++k) { e1[k] = L1[k] & (L1[k] >> 1) & 0x55555555u; n2 += __builtin_popcount(e1[k]); }
    const u32 s2i = wave_incl_scan(n2), s2 = s2i - n2, tot2 = (u32)__builtin_amdgcn_readlane((int)s2i, 63);
    u32 d1[16];
#pragma unroll
    for (int g = 0; g < 16; ++g) d1[g] = DEP[(L1[g >> 2] >> (8 * (g & 3))) & 255u];
    u32 C2[2];
    { const u32 bit = 2 * s2, sh = bit & 31u; const LAS u32* a = V + (bit >> 5);
      const u32 d0 = a[0], dd1 = a[1], d2 = a[2];
      const u64 m = (1ull << (2 * n2)) - 1ull;
      C2[0] = __builtin_amdgcn_alignbit(dd1, d0, sh) & (u32)m; C2[1] = __builtin_amdgcn_alignbit(d2, dd1, sh) & (u32)(m >> 32); }
    u32 e2[2]; e2[0] = C2[0] & (C2[0] >> 1) & 0x55555555u; e2[1] = C2[1] & (C2[1] >> 1) & 0x55555555u;
    const u32 n3 = __builtin_popcount(e2[0]) + __builtin_popcount(e2[1]);
    const u32 s3i = wave_incl_scan(n3), s3 = s3i - n3, tot3 = (u32)__builtin_amdgcn_readlane((int)s3i, 63);
    u32 dp2[8];
#pragma unroll
    for (int k = 0; k < 8; ++k) dp2[k] = DEP[(C2[k >> 2] >> (8 * (k & 3))) & 255u];
    const u32 o3 = ((2 * tot2 + 31u) >> 5) * 4u, o4 = o3 + ((2 * tot3 + 31u) >> 5) * 4u;
    u32 C3;
    { const u32 bit = 8 * o3 + 2 * s3, sh = bit & 31u; const LAS u32* a = V + (bit >> 5);
      C3 = __builtin_amdgcn_alignbit(a[1], a[0], sh) & ((1u << (2 * n3)) - 1u); }
    const u32 e3 = C3 & (C3 >> 1) & 0x55555555u, n4 = __builtin_popcount(e3);
    const u32 s4 = wave_incl_scan(n4) - n4;
    u32 dp3[4];
#pragma unroll
    for (int k = 0; k < 4; ++k) dp3[k] = DEP[(C3 >> (8 * k)) & 255u];
    u64 R4;
    { const u32 byte = o4 + s4, sh = (byte & 3u) * 8u; const LAS u32* a = V + (byte >> 2);
      const u32 d0 = a[0], dd1 = a[1], d2 = a[2];
      u32 t0 = __builtin_amdgcn_alignbit(dd1, d0, sh), t1 = __builtin_amdgcn_alignbit(d2, dd1, sh);
      const u32 B4 = B * 0x01010101u;
      t0 = ((B4 | 0x80808080u) - (t0 & 0x7f7f7f7fu)) ^ ((B4 ^ ~t0) & 0x80808080u);
      t1 = ((B4 | 0x80808080u) - (t1 & 0x7f7f7f7fu)) ^ ((B4 ^ ~t1) & 0x80808080u);
      R4 = (u64)t1 << 32 | t0; }
#pragma unroll
    for (int k = 0; k < 4; ++k) {
        const u32 p = k == 0 ? 0u : (u32)__builtin_popcount(e3 & ((1u << (8 * k)) - 1u));
        r3[k] = perm(T3, (u32)(R4 >> (8 * p)), dp3[k]);
    }
    CFENCE();
    { u32 ov[8];
#pragma unroll
      for (int k = 0; k < 8; ++k) {
          u32 p = (k & 3) == 0 ? 0u : (u32)__builtin_popcount(e2[k >> 2] & ((1u << (8 * (k & 3))) - 1u));
          if (k >= 4) p += __builtin_popcount(e2[0]);
          const LAS u32* a = r3 + (p >> 2);
          ov[k] = perm(T2, __builtin_amdgcn_alignbyte(a[1], a[0], p), dp2[k]);
      }
      CFENCE();
#pragma unroll
      for (int k = 0; k < 8; ++k) r2[k] = ov[k]; }
    CFENCE();
    u32 pb = 0;
#pragma unroll
    for (int g = 0; g < 16; ++g) {
        const u32 p = pb + ((g & 3) == 0 ? 0u : (u32)__builtin_popcount(e1[g >> 2] & ((1u << (8 * (g & 3))) - 1u)));
        const LAS u32* a = r2 + (p >> 2);
        const u32 E = perm(T1, __builtin_amdgcn_alignbyte(a[1], a[0], p), d1[g]);
        const u32 y0 = perm(E, A[g], 0x05010400u), y1 = perm(E, A[g], 0x07030602u);
        x[2 * g] = __builtin_amdgcn_alignbit(y0, y0, 1); x[2 * g + 1] = __builtin_amdgcn_alignbit(y1, y1, 1);
        if ((g & 3) == 3) pb += __builtin_popcount(e1[g >> 2]);
    }
    CFENCE();
}

constexpr int NR = 8;
constexpr int MM = 4096, NN = 4096, KSH = 512;
constexpr int NWG = 256, NTH = 512;
constexpr int PIECE_B = 8192;
#define NPUSH 28
constexpr int GA = 12;
constexpr int CHP = 16 + GA;
constexpr int NINA = 8 * CHP - NPUSH, NIN = NINA + (32 - GA) * 8, NOUT = 512, NQ = NIN + NOUT;
#define ND 8
#define NFLE 9
#define KEND 6
#define NFLI 8
#define NFL 9
constexpr int NSLOT = 11;
constexpr int NPW = (NQ + ND - 1) / ND;
#define PTHR 0
#define RAWH 0
#define PRI0 2
#define PRID 1
#define RAWT 12
#define PFD 0
constexpr int NST = 6;
constexpr int STAGE_B = 24576;
constexpr size_t OFF_CALL = 0;
constexpr size_t OFF_TMO = 32768;
constexpr size_t OFF_CNT = 65536;
constexpr size_t OFF_PROF = 131072;
constexpr size_t OFF_FLAG = 1u << 20;
constexpr size_t OFF_OBI = 2u << 20;
constexpr size_t OFF_OBO = OFF_OBI + 1024ull * PIECE_B;
constexpr size_t OFF_XS = OFF_OBO + 512ull * PIECE_B;
constexpr size_t OFF_WS = OFF_XS + 64ull * 32 * PIECE_B;
constexpr size_t WS_NEED = OFF_WS + 64ull * 16 * PIECE_B;
static_assert(OFF_FLAG + (size_t)NR * ND * 256 * 8 <= OFF_OBI && NPW <= 256, "flags");
__host__ __device__ constexpr size_t flag_off(int src, int idx) { return OFF_FLAG + ((size_t)(src * ND + idx % ND) * 256 + idx / ND) * 8; }
constexpr int LDS_TAB = 0, LDS_ROLE = 2048;
constexpr int LDS_MISC = LDS_ROLE + NST * STAGE_B;
constexpr int LDS_BYTES = LDS_MISC + 512;
constexpr int LDS_SLOTS = LDS_ROLE, LDS_PRIV = LDS_SLOTS + NSLOT * PIECE_B, LDS_CTL = LDS_PRIV + 6 * CD_BYTES;
static_assert(LDS_CTL + 1280 <= LDS_MISC && NPW <= 128, "decoder layout");
static_assert(LDS_ROLE + 8 * CD_BYTES <= LDS_MISC && ND == 8 && NPUSH <= 32, "phase 0 scratch");
static_assert(16384 + 2 * CD_BYTES <= NST * STAGE_B, "epilogue scratch");
constexpr u64 DEADLINE = 40000000ull;

struct Params {
    const unsigned short* x;
    const unsigned short* w;
    unsigned short* out;
    unsigned char* ws[NR];
    int rank;
    int pad_;
};

typedef short bf16x8 __attribute__((ext_vector_type(8)));
typedef float f32x4 __attribute__((ext_vector_type(4)));
typedef short s16x4 __attribute__((ext_vector_type(4)));
#define GAS __attribute__((address_space(1)))

__device__ __forceinline__ void st_sys(void* p, u32x4 v) { asm volatile("global_store_dwordx4 %0, %1, off sc0 sc1\n\ts_nop 1" :: "v"(p), "v"(v) : "memory"); }
__device__ __forceinline__ void st_agent(void* p, u32x4 v) { asm volatile("global_store_dwordx4 %0, %1, off sc1\n\ts_nop 1" :: "v"(p), "v"(v) : "memory"); }
constexpr int wcnt(int vm, int lgkm) { return (vm & 15) | 0x70 | (lgkm << 8) | ((vm >> 4) << 14); }
__device__ __forceinline__ void wait_vm0() { asm volatile("s_waitcnt vmcnt(0)" ::: "memory"); }
__device__ __forceinline__ u32 rfl(u32 v) { return (u32)__builtin_amdgcn_readfirstlane((int)v); }
__device__ __forceinline__ u64 now() { return __builtin_amdgcn_s_memrealtime(); }

#define STAMP(slot) do { } while (0)
#define STAMPC(slot) do { } while (0)

__device__ __forceinline__ void in_entry(int idx, int& c, bool& isw, int& bl) {
    if (idx < NINA) { const int e = idx + NPUSH; c = e / CHP; const int r = e - c * CHP; isw = r < 16; bl = isw ? r : r - 16; }
    else { const int e = idx - NINA; c = (e >> 1) & 7; isw = false; bl = GA + ((e >> 4) << 1) + (e & 1); }
}
__device__ __forceinline__ bool wait_cnt2(const unsigned char* c0, u32 t0v, const unsigned char* c1, u32 t1v, u32 call, u32* tmo) {
    const u64 t0 = now();
    for (;;) {
        const u32 a = __hip_atomic_load((const GAS u32*)c0, __ATOMIC_RELAXED, __HIP_MEMORY_SCOPE_SYSTEM);
        const u32 b = __hip_atomic_load((const GAS u32*)c1, __ATOMIC_RELAXED, __HIP_MEMORY_SCOPE_SYSTEM);
        if (rfl(a) >= t0v && rfl(b) >= t1v) return true;
        if (rfl(__hip_atomic_load((const GAS u32*)tmo, __ATOMIC_RELAXED, __HIP_MEMORY_SCOPE_AGENT)) == call || now() - t0 > DEADLINE) {
            __hip_atomic_store((GAS u32*)tmo, call, __ATOMIC_RELAXED, __HIP_MEMORY_SCOPE_AGENT);
            return false;
        }
        __builtin_amdgcn_s_sleep(2);
    }
}

__device__ __forceinline__ void store_piece(const u32 (&x)[32], const u32 (&A)[16], const u32 (&L1)[4], u32 inf, unsigned char* slot, lds_u8* sc, int lane) {
    unsigned char* o = slot + lane * 16;
    if (inf >> 31) {
#pragma unroll
        for (int j = 0; j < 8; ++j) st_sys(o + j * 1024, (u32x4){x[4 * j], x[4 * j + 1], x[4 * j + 2], x[4 * j + 3]});
    } else {
#pragma unroll
        for (int j = 0; j < 4; ++j) st_sys(o + j * 1024, (u32x4){A[4 * j], A[4 * j + 1], A[4 * j + 2], A[4 * j + 3]});
        st_sys(o + 4096, (u32x4){L1[0], L1[1], L1[2], L1[3]});
        const u32x4 v = *lp<u32x4>(sc + CD_V + lane * 16);
        if ((u32)lane * 16u < ((inf >> 8) & 0x7ffu)) st_sys(o + 5120, v);
    }
}

__device__ __forceinline__ u32 pk_bf16(float lo, float hi) { u32 r; asm("v_cvt_pk_bf16_f32 %0, %1, %2" : "=v"(r) : "v"(lo), "v"(hi)); return r; }

__global__ __launch_bounds__(NTH, 2) void gemm_ar_kernel(Params P) {
    extern __shared__ __attribute__((aligned(16))) unsigned char smem[];
    lds_u8* sm = (lds_u8*)smem;
    const int tid = threadIdx.x, lane = tid & 63, wid = (int)rfl(tid >> 6);
    const int b = blockIdx.x, xcd = b & 7, q = b >> 3;
    const int rank = P.rank, ri = rank >> 2, rj = rank & 3;
    unsigned char* my = P.ws[rank];
    u32* tmo = (u32*)(my + OFF_TMO);
    const lds_u8* tab = sm + LDS_TAB;
    STAMP(0);
    u32 call_v = 0;
    if (lane == 0) call_v = __hip_atomic_fetch_add((GAS u32*)(my + OFF_CALL + b * 128 + wid * 4), 1u, __ATOMIC_RELAXED, __HIP_MEMORY_SCOPE_AGENT) + 1u;

    unsigned char* wsl = P.ws[lane & 7];

    const bool is_dec = q >= 16 && q < 16 + ND && xcd < NR - 1;
    const int peer_r = (rank + 1 + xcd) & 7;
    const int pi = q * 4 + (wid - 4);
    const bool pusher = wid >= 4 && q < 8 && xcd < NR - 1 && pi < NPUSH;
    const int pid = pusher ? (pi < 16 ? 512 + (16 * (peer_r & 3) + pi) * 8 : (32 * (peer_r >> 2) + pi - 16) * 8) : b * 4 + wid;
    const bool enc = wid < 4 || pusher;
    const bool isw = pid >= 512;
    const int blk = (pid & 511) >> 3, ch = pid & 7;
    u32x4 raw[8];
    auto load_raw = [&]() {
        if (!isw) {
            const unsigned short* s = P.x + (size_t)(blk * 64 + (lane & 15)) * KSH + ch * 64 + (lane >> 4) * 8;
#pragma unroll
            for (int j = 0; j < 8; ++j) raw[j] = *(const u32x4*)(s + (size_t)(j >> 1) * 16 * KSH + (j & 1) * 32);
        } else {
            const unsigned short* s = P.w + (size_t)(ch * 64 + (lane >> 3)) * NN + blk * 64 + (lane & 7) * 8;
#pragma unroll
            for (int j = 0; j < 8; ++j) raw[j] = *(const u32x4*)(s + (size_t)j * 8 * NN);
        }
    };
    const int cls = pusher ? 0 : ((!isw && (blk & 31) >= GA) ? 2 : (ch >= PRI0 ? 1 : 0));
    if (enc && cls == 0) load_raw();
    if (tid < 256) codec_build_tables(sm + LDS_TAB, tid);
    if (is_dec && tid < 320) *lp<u32>(sm + LDS_CTL + tid * 4) = 0;
    __syncthreads();
    const u32 call = rfl(call_v);
    if (enc) {
        if (cls != 0) {
            for (int k = 0; k < cls * PRID - 1; ++k) __builtin_amdgcn_s_sleep(16);
            load_raw();
        }
        lds_u8* sc = sm + (is_dec ? LDS_PRIV : LDS_ROLE) + wid * CD_BYTES;
        u32 x[32];
        if (!isw) {
#pragma unroll
            for (int j = 0; j < 8; ++j) { x[4 * j] = raw[j].x; x[4 * j + 1] = raw[j].y; x[4 * j + 2] = raw[j].z; x[4 * j + 3] = raw[j].w; }
        } else {
#pragma unroll
            for (int j = 0; j < 8; ++j) { const int k = 8 * j + (lane >> 3); *lp<u32x4>(sc + k * 128 + (((lane & 7) ^ (k & 7)) << 4)) = raw[j]; }
            const int g = lane >> 4, qq = (lane & 15) >> 2, pp = lane & 3;
#pragma unroll
            for (int j = 0; j < 8; ++j) {
                const int ni = j >> 1, ks = j & 1;
#pragma unroll
                for (int t = 0; t < 2; ++t) {
                    const int k = 32 * ks + 8 * g + 4 * t + qq, chunk = 2 * ni + (pp >> 1);
                    const s16x4 v = __builtin_amdgcn_ds_read_tr16_b64_v4i16((LAS s16x4*)(sc + k * 128 + ((chunk ^ (k & 7)) << 4) + (pp & 1) * 8));
                    const u32x2 u = __builtin_bit_cast(u32x2, v);
                    x[4 * j + 2 * t] = u.x; x[4 * j + 2 * t + 1] = u.y;
                }
            }
            asm volatile("s_waitcnt lgkmcnt(0)" ::: "memory");
        }
        STAMP(1); STAMPC(28);
        const int rb = blk & 31, step = rank * 8 + ch;
        const size_t stg = isw ? OFF_WS + (size_t)(step * 16 + (blk & 15)) * PIECE_B : OFF_XS + (size_t)(step * 32 + rb) * PIECE_B;
        const size_t cnt = OFF_CNT + (size_t)(isw ? 128 + (blk & 15) * 8 + ch : (rb >> 1) * 8 + ch) * 128;
        const bool cons = lane < NR && (isw ? (lane & 3) == (blk >> 4) : (lane >> 2) == (blk >> 5));
        const bool mine = isw ? ((blk >> 4) == rj) : ((blk >> 5) == ri);
        if (pusher) {
            unsigned char* d = P.ws[peer_r] + stg + lane * 16;
#pragma unroll
            for (int j = 0; j < 8; ++j) {
                st_sys(d + j * 1024, (u32x4){x[4 * j], x[4 * j + 1], x[4 * j + 2], x[4 * j + 3]});
                if (PTHR > 0 && (j % PTHR) == PTHR - 1) wait_vm0();
            }
            wait_vm0();
            STAMP(2);
            if (lane == peer_r) __hip_atomic_fetch_add((GAS u32*)(wsl + cnt), 1u, __ATOMIC_RELAXED, __HIP_MEMORY_SCOPE_SYSTEM);
        } else if (ch == 0 && (isw ? (blk & 15) < NPUSH : (rb < GA && 16 + rb < NPUSH))) {
            if (mine) {
                unsigned char* d = my + stg + lane * 16;
#pragma unroll
                for (int j = 0; j < 8; ++j) st_sys(d + j * 1024, (u32x4){x[4 * j], x[4 * j + 1], x[4 * j + 2], x[4 * j + 3]});
                wait_vm0();
                if (lane == 0) __hip_atomic_fetch_add((GAS u32*)(my + cnt), 1u, __ATOMIC_RELAXED, __HIP_MEMORY_SCOPE_SYSTEM);
            }
        } else {
            u32 A[16], L1[4];
            u32 inf = 0x80000000u;
            if (!((isw ? ch * CHP + (blk & 15) : (rb < GA ? ch * CHP + 16 + rb : 1 << 20)) < NPUSH + RAWH)) inf = codec_encode(x, A, L1, sc, tab, lane);
            STAMP(7); STAMPC(29);
            store_piece(x, A, L1, inf, my + OFF_OBI + (size_t)pid * PIECE_B, sc, lane);
            wait_vm0();
            STAMP(2);
            const int idx = isw ? ch * CHP + (blk & 15) - NPUSH : (rb < GA ? ch * CHP + 16 + rb - NPUSH : NINA + ((rb - GA) >> 1) * 16 + ch * 2 + (rb & 1));
            if (cons && lane != rank) __hip_atomic_store((GAS u64*)(wsl + flag_off(rank, idx)), (u64)call << 32 | inf, __ATOMIC_RELAXED, __HIP_MEMORY_SCOPE_SYSTEM);
            if (mine) {
                unsigned char* d = my + stg + lane * 16;
#pragma unroll
                for (int j = 0; j < 8; ++j) st_agent(d + j * 1024, (u32x4){x[4 * j], x[4 * j + 1], x[4 * j + 2], x[4 * j + 3]});
                wait_vm0();
                if (lane == 0) __hip_atomic_fetch_add((GAS u32*)(my + cnt), 1u, __ATOMIC_RELAXED, __HIP_MEMORY_SCOPE_AGENT);
            }
        }
        STAMP(3);
    }

    if (q < 16 || q >= 24) {
        __syncthreads();
        lds_u8* ring = sm + LDS_ROLE;
        const int wr = wid >> 1, wc = wid & 1;
        const int cb = 4 * (xcd & 3) + (q & 3);
        bool bad = false;
        const int ntile = q < 8 ? 2 : 1;
        const int q2 = q < 16 ? q : q - 8;
        for (int t = 0; t < ntile; ++t) {
            const int a = t == 1 ? 12 + 2 * (xcd >> 2) + (q >> 2) : (q2 < 12 ? 3 * (xcd >> 2) + (q2 >> 2) : 6 + 3 * (xcd >> 2) + ((q2 - 12) >> 2));
            const int g = t;
            const unsigned char* gx = my + OFF_XS + (size_t)(2 * a) * PIECE_B;
            const unsigned char* gw = my + OFF_WS + (size_t)cb * PIECE_B;
            const bool ldA = wid < 4;
            const unsigned char* gl = (ldA ? gx + wid * 4096 : gw + (wid - 4) * 4096) + lane * 16;
            const size_t lstep = ldA ? (size_t)32 * PIECE_B : (size_t)16 * PIECE_B;
            f32x4 acc[2][2] = {};
            auto issue = [&](int i) {
                const int step = (i & 7) * 8 + (i >> 3);
                lds_u8* d = ring + (i % NST) * STAGE_B + wid * 4096;
                const unsigned char* sa = gl + (size_t)step * lstep;
#pragma unroll
                for (int j = 0; j < 4; ++j) __builtin_amdgcn_global_load_lds((const GAS u32*)(sa + j * 1024), (LAS u32*)(d + j * 1024), 16, 0, 17);
            };
            const unsigned char* cx = my + OFF_CNT + (size_t)(a * 8) * 128; const unsigned char* cw = my + OFF_CNT + (size_t)(128 + cb * 8) * 128;
            volatile LAS u32* rdyw = (volatile LAS u32*)(sm + LDS_MISC + 256);
            int rdy = 0;
            u32 pa = 0, pb = 0; bool pend = false;
            auto poll_issue = [&]() { pa = __hip_atomic_load((const GAS u32*)(cx + rdy * 128), __ATOMIC_RELAXED, __HIP_MEMORY_SCOPE_SYSTEM); pb = __hip_atomic_load((const GAS u32*)(cw + rdy * 128), __ATOMIC_RELAXED, __HIP_MEMORY_SCOPE_SYSTEM); pend = true; };
            auto poll_take = [&]() { if (pend) { pend = false; if (rfl(pa) >= 16u * call && rfl(pb) >= 8u * call) { ++rdy; if (lane == 0) *rdyw = rdy; } } };
            auto need = [&](int c) {
                poll_take();
                while (rdy <= c) {
                    if (!wait_cnt2(cx + rdy * 128, 16u * call, cw + rdy * 128, 8u * call, call, tmo)) bad = true;
                    ++rdy; if (lane == 0) *rdyw = rdy;
                }
            };
            int tp = NST;
            const unsigned char* gt = lane < 32 ? gx + (q & 3) * 4096 + lane * 128 : gw + ((q >> 2) & 3) * 2048 + (lane & 15) * 128;
            const size_t tstep = lane < 32 ? (size_t)32 * PIECE_B : (size_t)16 * PIECE_B;
            auto warm = [&](int i) {
                const int lim = min(min(64, (int)rfl(*rdyw) * 8), i + PFD);
#pragma unroll 1
                for (int k = 0; k < 3 && tp < lim; ++k, ++tp)
                    __builtin_amdgcn_global_load_lds((const GAS u32*)(gt + (size_t)((tp & 7) * 8 + (tp >> 3)) * tstep), (LAS u32*)(sm + LDS_MISC), 4, 0, 17);
            };
            auto rd = [&](int i, bf16x8 (&af)[2][2], bf16x8 (&bfr)[2][2]) {
                lds_u8* st = ring + (i % NST) * STAGE_B;
#pragma unroll
                for (int m = 0; m < 2; ++m)
#pragma unroll
                    for (int ks = 0; ks < 2; ++ks) af[m][ks] = *lp<bf16x8>(st + (wr >> 1) * 8192 + (((wr & 1) * 2 + m) * 2 + ks) * 1024 + lane * 16);
#pragma unroll
                for (int n = 0; n < 2; ++n)
#pragma unroll
                    for (int ks = 0; ks < 2; ++ks) bfr[n][ks] = *lp<bf16x8>(st + 16384 + ((wc * 2 + n) * 2 + ks) * 1024 + lane * 16);
            };
            auto mm = [&](bf16x8 (&af)[2][2], bf16x8 (&bfr)[2][2]) {
#pragma unroll
                for (int ks = 0; ks < 2; ++ks)
#pragma unroll
                    for (int m = 0; m < 2; ++m)
#pragma unroll
                        for (int n = 0; n < 2; ++n) acc[m][n] = __builtin_amdgcn_mfma_f32_16x16x32_bf16(bfr[n][ks], af[m][ks], acc[m][n], 0, 0, 0);
            };
            auto kloop = [&](auto role_c) {
                constexpr int ROLE = decltype(role_c)::value;
                bf16x8 a0[2][2], b0[2][2], a1[2][2], b1[2][2];
                if constexpr (ROLE == 1) { if (lane == 0) *rdyw = 0; need(0); }
                __builtin_amdgcn_s_barrier();
                if (g == 0) STAMP(4); else STAMP(12);
                if constexpr (ROLE == 0) {
#pragma unroll
                    for (int i = 0; i < NST - 1; ++i) issue(i);
                    __builtin_amdgcn_s_waitcnt(wcnt(4 * (NST - 2), 15));
                }
                __builtin_amdgcn_s_barrier();
                rd(0, a0, b0);
                auto pair = [&](int i, auto tail_c) {
                    constexpr int TAIL = decltype(tail_c)::value;
                    { const int nx = i + NST - 1;
                      if constexpr (ROLE == 0) __builtin_amdgcn_s_waitcnt(wcnt(TAIL == 0 || TAIL + 4 < 64 ? 4 * (NST - 3) : 4 * (63 - TAIL - 1 > 0 ? 63 - TAIL - 1 : 0), 0));
                      else __builtin_amdgcn_s_waitcnt(wcnt(63, 0));
                      __builtin_amdgcn_s_barrier();
                      if constexpr (ROLE == 0 && (TAIL == 0 || TAIL + NST - 1 < 64)) issue(nx);
                      rd(i + 1, a1, b1);
                      __builtin_amdgcn_sched_barrier(0);
                      mm(a0, b0);
                      __builtin_amdgcn_sched_barrier(0);
                      if constexpr (ROLE == 1 && TAIL == 0) { if ((i & 2) == 0 && rdy < 8) { poll_take(); if (rdy < 8) poll_issue(); } }
                      if constexpr (ROLE == 2 && TAIL == 0) warm(i); }
                    { const int nx = i + NST;
                      if constexpr (ROLE == 1 && TAIL == 0) { if ((nx & 7) == 0) need(nx >> 3); }
                      if constexpr (ROLE == 0) __builtin_amdgcn_s_waitcnt(wcnt(TAIL == 0 ? 4 * (NST - 3) : 4 * (63 - TAIL - 2 > 0 ? 63 - TAIL - 2 : 0), 0));
                      else __builtin_amdgcn_s_waitcnt(wcnt(63, 0));
                      __builtin_amdgcn_s_barrier();
                      if constexpr (ROLE == 0 && TAIL == 0) issue(nx);
                      if constexpr (TAIL != 62) rd(i + 2, a0, b0);
                      __builtin_amdgcn_sched_barrier(0);
                      mm(a1, b1);
                      __builtin_amdgcn_sched_barrier(0);
                      if constexpr (ROLE == 2 && TAIL == 0) warm(i + 1); }
                };
#pragma unroll 1
                for (int i = 0; i < 58; i += 2) pair(i, std::integral_constant<int, 0>{});
                pair(58, std::integral_constant<int, 58>{});
                pair(60, std::integral_constant<int, 60>{});
                pair(62, std::integral_constant<int, 62>{});
            };
            if (wid < 6) kloop(std::integral_constant<int, 0>{});
            else if (wid == 6) kloop(std::integral_constant<int, 1>{});
            else kloop(std::integral_constant<int, 2>{});
            wait_vm0();
            if (g == 0) STAMP(5); else STAMP(13);
            __syncthreads();
#pragma unroll
            for (int m = 0; m < 2; ++m)
#pragma unroll
                for (int n = 0; n < 2; ++n) {
                    const int row = 32 * wr + 16 * m + (lane & 15), col = 32 * wc + 16 * n + 4 * (lane >> 4);
                    const u32x2 v = {pk_bf16(acc[m][n][0], acc[m][n][1]), pk_bf16(acc[m][n][2], acc[m][n][3])};
                    *lp<u32x2>(ring + row * 128 + (((col >> 3) ^ (row & 7)) << 4) + (col & 4) * 2) = v;
                }
            if (tid == 6 * 64) *lp<u32>(sm + LDS_MISC + 272) = (bad || __hip_atomic_load((const GAS u32*)tmo, __ATOMIC_RELAXED, __HIP_MEMORY_SCOPE_AGENT) == call) ? 1u : 0u;
            __syncthreads();
            if (wid < 4) {
                const int h = wid & 1;
                const bool poison = *lp<u32>(sm + LDS_MISC + 272) != 0u;
                u32 x[32];
#pragma unroll
                for (int j = 0; j < 8; ++j) {
                    const int row = 64 * h + 8 * j + (lane >> 3);
                    u32x4 v = *lp<u32x4>(ring + row * 128 + (((lane & 7) ^ (row & 7)) << 4));
                    if (poison) v = (u32x4){0x7fc07fc0u, 0x7fc07fc0u, 0x7fc07fc0u, 0x7fc07fc0u};
                    x[4 * j] = v.x; x[4 * j + 1] = v.y; x[4 * j + 2] = v.z; x[4 * j + 3] = v.w;
                }
                const int o = (a * 16 + cb) * 2 + h;
                if (wid < 2) {
                    lds_u8* sce = ring + 16384 + wid * CD_BYTES;
                    u32 A[16], L1[4];
                    u32 inf = 0x80000000u;
                    if (o < NOUT - RAWT) inf = codec_encode(x, A, L1, sce, tab, lane);
                    store_piece(x, A, L1, inf, my + OFF_OBO + (size_t)o * PIECE_B, sce, lane);
                    wait_vm0();
                    if (lane < NR && lane != rank)
                        __hip_atomic_store((GAS u64*)(wsl + flag_off(rank, NIN + o)), (u64)call << 32 | inf, __ATOMIC_RELAXED, __HIP_MEMORY_SCOPE_SYSTEM);
                } else {
                    unsigned short* d = P.out + (size_t)(2048 * ri + 128 * a + 64 * h + (lane >> 3)) * NN + 1024 * rj + 64 * cb + (lane & 7) * 8;
#pragma unroll
                    for (int j = 0; j < 8; ++j) st_agent(d + (size_t)j * 8 * NN, (u32x4){x[4 * j], x[4 * j + 1], x[4 * j + 2], x[4 * j + 3]});
                }
            }
            if (g == 0) STAMP(6); else STAMP(14);
            __syncthreads();
        }
    } else if (is_dec) {
        const int src = peer_r, si = src >> 2, sj = src & 3;
        const unsigned char* peer = P.ws[src];
        const int dj = q - 16;
        const int np = (NQ - dj + ND - 1) / ND;
        volatile LAS u32* c_info = (volatile LAS u32*)(sm + LDS_CTL);
        volatile LAS u32* c_full = (volatile LAS u32*)(sm + LDS_CTL + 1024);
        volatile LAS u32* c_done = (volatile LAS u32*)(sm + LDS_CTL + 1088);
        volatile LAS u32* c_piece = (volatile LAS u32*)(sm + LDS_CTL + 1152);
        LAS u32* c_ticket = (LAS u32*)(sm + LDS_CTL + 1216);
        auto src_slot = [&](int idx) -> const unsigned char* {
            if (idx >= NIN) return peer + OFF_OBO + (size_t)(idx - NIN) * PIECE_B;
            int c, bl; bool isw2; in_entry(idx, c, isw2, bl);
            return peer + OFF_OBI + (size_t)(isw2 ? 512 + (16 * rj + bl) * 8 + c : (32 * ri + bl) * 8 + c) * PIECE_B;
        };
        if (wid == 6) {
            const unsigned char* fb = my + OFF_FLAG + (size_t)(src * ND + dj) * 256 * 8;
            const u64 t0 = now();
            bool p0 = lane >= np, p1 = lane + 64 >= np;
            int rounds = 0;
            for (;;) {
                u64 v0 = 0, v1 = 0;
                if (!p0) v0 = __hip_atomic_load((const GAS u64*)(fb + (size_t)lane * 8), __ATOMIC_RELAXED, __HIP_MEMORY_SCOPE_SYSTEM);
                if (!p1) v1 = __hip_atomic_load((const GAS u64*)(fb + (size_t)(lane + 64) * 8), __ATOMIC_RELAXED, __HIP_MEMORY_SCOPE_SYSTEM);
                if (!p0 && (u32)(v0 >> 32) >= call) { c_info[lane] = (u32)v0 | 0x40000000u; p0 = true; }
                if (!p1 && (u32)(v1 >> 32) >= call) { c_info[lane + 64] = (u32)v1 | 0x40000000u; p1 = true; }
                if (__all(p0 && p1)) break;
                if ((++rounds & 7) == 0 && (rfl(__hip_atomic_load((const GAS u32*)tmo, __ATOMIC_RELAXED, __HIP_MEMORY_SCOPE_AGENT)) == call || now() - t0 > DEADLINE)) {
                    __hip_atomic_store((GAS u32*)tmo, call, __ATOMIC_RELAXED, __HIP_MEMORY_SCOPE_AGENT);
                    if (!p0) c_info[lane] = 0x60000000u;
                    if (!p1) c_info[lane + 64] = 0x60000000u;
                    break;
                }
                __builtin_amdgcn_s_sleep(2);
            }
        } else if (wid == 7) {
            int pub = 0;
            auto publish_upto = [&](int e) { for (; pub < e; ++pub) if (lane == 0) c_full[pub % NSLOT] = pub + 1; };
            u64 todo0 = np >= 64 ? ~0ull : (1ull << np) - 1ull, todo1 = np > 64 ? (1ull << (np - 64)) - 1ull : 0ull;
            int n = 0; u32 inf = 0;
            auto pick = [&](int s) -> bool {
                const u32 v0 = c_info[lane], v1 = c_info[lane + 64], dn = c_done[s % NSLOT];
                const u64 a0 = __ballot(v0 != 0u) & todo0, a1 = __ballot(v1 != 0u) & todo1;
                if ((a0 | a1) == 0ull || !(s < NSLOT || rfl(dn) == (u32)(s - NSLOT + 1))) return false;
                if (a0) { const int l = __builtin_ctzll(a0); todo0 &= ~(1ull << l); n = l; inf = (u32)__builtin_amdgcn_readlane((int)v0, l); }
                else { const int l = __builtin_ctzll(a1); todo1 &= ~(1ull << l); n = 64 + l; inf = (u32)__builtin_amdgcn_readlane((int)v1, l); }
                return true;
            };
            bool have = false;
            for (int s = 0; s < np; ++s) {
                const int slot = s % NSLOT;
                if (!have) { wait_vm0(); publish_upto(s); while (!pick(s)) __builtin_amdgcn_s_sleep(1); }
                if (lane == 0) c_piece[slot] = n;
                lds_u8* d = sm + LDS_SLOTS + slot * PIECE_B;
                const unsigned char* s0 = src_slot(dj + n * ND);
                const unsigned char* sp = s0 + lane * 16;
                if (inf & 0x20000000u) { have = false; wait_vm0(); publish_upto(s + 1); continue; }
                const bool israw = (inf >> 31) != 0u;
                const unsigned char* sv = (israw || (u32)lane * 16u < ((inf >> 8) & 0x7ffu)) ? sp + 5120 : s0 + 5120;
                const bool input = dj + n * ND < NIN;
                auto pull = [&](auto nfl_c, auto cnt_c) {
                    constexpr int N = decltype(nfl_c)::value, CNT = decltype(cnt_c)::value;
#pragma unroll
                    for (int j = 0; j < CNT; ++j) {
                        asm volatile("s_waitcnt vmcnt(%0)" :: "n"(N - 1) : "memory");
                        if (j == 0 || (j - N + 1) % 6 == 0) publish_upto(s + (j - N + 1 - 5) / 6);
                        __builtin_amdgcn_global_load_lds((const GAS u32*)(j == 5 ? sv : sp + j * 1024), (LAS u32*)(d + j * 1024), 16, 0, 17);
                        if (j == 1) have = s + 1 < np && pick(s + 1);
                    }
                };
                static_assert(NFL >= 9 && NFLE >= 9 && NFLI >= 7, "the publish rule needs j - N + 1 < 0");
                if (israw) pull(std::integral_constant<int, NFLE>{}, std::integral_constant<int, 8>{});
                else if (input) pull(std::integral_constant<int, NFLI>{}, std::integral_constant<int, 6>{});
                else if (s >= np - KEND) pull(std::integral_constant<int, NFLE>{}, std::integral_constant<int, 6>{});
                else pull(std::integral_constant<int, NFL>{}, std::integral_constant<int, 6>{});
            }
            wait_vm0(); publish_upto(np);
            STAMP(9);
        } else {
            lds_u8* sc = sm + LDS_PRIV + wid * CD_BYTES;
            int nstamp = 0;
            for (;;) {
                u32 nn = 0;
                if (lane == 0) nn = __hip_atomic_fetch_add(c_ticket, 1u, __ATOMIC_RELAXED, __HIP_MEMORY_SCOPE_WORKGROUP);
                const int sq = (int)rfl(nn);
                if (sq >= np) break;
                while (rfl(c_full[sq % NSLOT]) != (u32)(sq + 1)) __builtin_amdgcn_s_sleep(1);
                asm volatile("" ::: "memory");
                const int n = (int)rfl(c_piece[sq % NSLOT]);
                const u32 inf = rfl(c_info[n]);
                const bool ok = !(inf & 0x20000000u);
                lds_u8* sl = sm + LDS_SLOTS + (sq % NSLOT) * PIECE_B;
                u32 x[32];
                if (!ok) {
#pragma unroll
                    for (int j = 0; j < 32; ++j) x[j] = 0x7fc07fc0u;
                } else if (inf >> 31) {
#pragma unroll
                    for (int j = 0; j < 8; ++j) { const u32x4 v = *lp<u32x4>(sl + j * 1024 + lane * 16); x[4 * j] = v.x; x[4 * j + 1] = v.y; x[4 * j + 2] = v.z; x[4 * j + 3] = v.w; }
                } else {
                    u32 A[16], L1[4];
#pragma unroll
                    for (int j = 0; j < 4; ++j) { const u32x4 v = *lp<u32x4>(sl + j * 1024 + lane * 16); A[4 * j] = v.x; A[4 * j + 1] = v.y; A[4 * j + 2] = v.z; A[4 * j + 3] = v.w; }
                    { const u32x4 v = *lp<u32x4>(sl + 4096 + lane * 16); L1[0] = v.x; L1[1] = v.y; L1[2] = v.z; L1[3] = v.w; }
                    codec_decode(x, A, L1, inf & 255u, (const LAS u32*)(sl + 5120), sc, tab, lane);
                }
                asm volatile("s_waitcnt lgkmcnt(0)" ::: "memory");
                if (lane == 0) c_done[sq % NSLOT] = sq + 1;
                const int idx = dj + n * ND;
                if (idx < NIN) {
                    int c, bl; bool isw2; in_entry(idx, c, isw2, bl);
                    const int step = src * 8 + c;
                    unsigned char* d = (isw2 ? my + OFF_WS + (size_t)(step * 16 + bl) * PIECE_B : my + OFF_XS + (size_t)(step * 32 + bl) * PIECE_B) + lane * 16;
                    if (ok) {
#pragma unroll
                        for (int j = 0; j < 8; ++j) st_agent(d + j * 1024, (u32x4){x[4 * j], x[4 * j + 1], x[4 * j + 2], x[4 * j + 3]});
                        wait_vm0();
                        if (lane == 0) __hip_atomic_fetch_add((GAS u32*)(my + OFF_CNT + (size_t)(isw2 ? 128 + bl * 8 + c : (bl >> 1) * 8 + c) * 128), 1u, __ATOMIC_RELAXED, __HIP_MEMORY_SCOPE_AGENT);
                    }
                } else {
                    const int o = idx - NIN, g = o >> 8, tile = (o & 255) >> 1, h = o & 1;
                    unsigned short* d = P.out + (size_t)(2048 * si + 1024 * g + 128 * (tile >> 4) + 64 * h + (lane >> 3)) * NN + 1024 * sj + 64 * (tile & 15) + (lane & 7) * 8;
#pragma unroll
                    for (int j = 0; j < 8; ++j) st_agent(d + (size_t)j * 8 * NN, (u32x4){x[4 * j], x[4 * j + 1], x[4 * j + 2], x[4 * j + 3]});
                }
            }
            (void)nstamp;
        }
    }
    STAMP(31);
}

extern "C" void kernel_launch(int rank, int nranks, ncclComm_t comm, hipStream_t stream,
                              void* const* d_in, const int* in_sizes, int n_in,
                              void* d_out, int out_size, const int* in_dtypes, int out_dtype,
                              void* const* d_out_all, void* const* d_ws_all, size_t ws_bytes) {
    (void)comm; (void)in_dtypes; (void)out_dtype; (void)d_out_all;
    if (nranks != NR || n_in != 2 || rank < 0 || rank >= NR || in_sizes[0] != MM * KSH || in_sizes[1] != KSH * NN || out_size != MM * NN || ws_bytes < WS_NEED) {
        fprintf(stderr, "kernel_launch: unexpected problem shape\n");
        return;
    }
    static bool first = true;
    if (first) {
        hipError_t e = hipFuncSetAttribute((const void*)gemm_ar_kernel, hipFuncAttributeMaxDynamicSharedMemorySize, LDS_BYTES);
        if (e != hipSuccess) { fprintf(stderr, "hipFuncSetAttribute: %s\n", hipGetErrorString(e)); return; }
        first = false;
    }
    Params P;
    P.x = (const unsigned short*)d_in[0];
    P.w = (const unsigned short*)d_in[1];
    P.out = (unsigned short*)d_out;
    for (int p = 0; p < NR; ++p) P.ws[p] = (unsigned char*)d_ws_all[p];
    P.rank = rank;
    P.pad_ = 0;
    hipLaunchKernelGGL(gemm_ar_kernel, dim3(NWG), dim3(NTH), LDS_BYTES, stream, P);
    hipError_t e = hipGetLastError();
    if (e != hipSuccess) fprintf(stderr, "launch: %s\n", hipGetErrorString(e));
}
```

```cpp
#include <hip/hip_runtime.h>
#include <rccl/rccl.h>
#include <cstdio>
#include <type_traits>

typedef unsigned u32;
typedef unsigned long long u64;
typedef unsigned u32x4 __attribute__((ext_vector_type(4)));
typedef unsigned u32x2 __attribute__((ext_vector_type(2)));
typedef unsigned short u16x2 __attribute__((ext_vector_type(2)));
#define LAS __attribute__((address_space(3)))
typedef LAS unsigned char lds_u8;

constexpr int CD_V = 0;
constexpr int CD_R2 = 1040;
constexpr int CD_R3 = CD_R2 + 64 * 68;
constexpr int CD_R4 = CD_R3 + 64 * 36;
constexpr int CD_BYTES = CD_R4 + 64 * 20;
constexpr int CD_DEC_R2 = 1040, CD_DEC_R3 = CD_DEC_R2 + 64 * 36, CD_DEC_BYTES = CD_DEC_R3 + 64 * 20;
constexpr int CD_TAB_BYTES = 2048;
static_assert(CD_BYTES % 16 == 0 && CD_DEC_BYTES % 16 == 0, "align");

__device__ __forceinline__ u32 perm(u32 hi, u32 lo, u32 sel) { return __builtin_amdgcn_perm(hi, lo, sel); }
__device__ __forceinline__ u32 dot4(u32 a, u32 b, u32 c) { return __builtin_amdgcn_udot4(a, b, c, false); }
__device__ __forceinline__ u32 pk_sub(u32 a, u32 b) { return __builtin_bit_cast(u32, (u16x2)(__builtin_bit_cast(u16x2, a) - __builtin_bit_cast(u16x2, b))); }
__device__ __forceinline__ u32 pk_min(u32 a, u32 b) { return __builtin_bit_cast(u32, __builtin_elementwise_min(__builtin_bit_cast(u16x2, a), __builtin_bit_cast(u16x2, b))); }
__device__ __forceinline__ u32 wave_incl_scan(u32 x) {
    x += (u32)__builtin_amdgcn_update_dpp(0, (int)x, 0x111, 0xf, 0xf, false);
    x += (u32)__builtin_amdgcn_update_dpp(0, (int)x, 0x112, 0xf, 0xf, false);
    x += (u32)__builtin_amdgcn_update_dpp(0, (int)x, 0x114, 0xf, 0xf, false);
    x += (u32)__builtin_amdgcn_update_dpp(0, (int)x, 0x118, 0xf, 0xf, false);
    x += (u32)__builtin_amdgcn_update_dpp(0, (int)x, 0x142, 0xa, 0xf, false);
    x += (u32)__builtin_amdgcn_update_dpp(0, (int)x, 0x143, 0xc, 0xf, false);
    return x;
}
template <class T> __device__ __forceinline__ LAS T* lp(lds_u8* p) { return (LAS T*)p; }
#define CFENCE() asm volatile("" ::: "memory")

__device__ __forceinline__ void codec_build_tables(lds_u8* tab, int i) {
    u32 dep = 0, cmp = 0x0c0c0c0cu; int n = 0;
#pragma unroll
    for (int k = 0; k < 4; ++k) {
        const u32 c = (i >> (2 * k)) & 3;
        if (c == 3) { dep |= (u32)n << (8 * k); cmp = (cmp & ~(0xffu << (8 * n))) | ((u32)k << (8 * n)); ++n; }
        else dep |= (4u + c) << (8 * k);
    }
    lp<u32>(tab)[i] = dep; lp<u32>(tab)[256 + i] = cmp;
}

__device__ __forceinline__ void stream_append(LAS u32* r, u32 pk, u32& p, u32 pnew, u32& acc) {
    const u64 w = (u64)pk << ((p & 3u) * 8u);
    const u32 lo = (u32)w | acc;
    r[p >> 2] = lo;
    acc = ((p ^ pnew) & ~3u) ? (u32)(w >> 32) : lo;
    p = pnew;
}

__device__ __forceinline__ u32 codec_encode(const u32 (&x)[32], u32 (&A)[16], u32 (&L1)[4], lds_u8* sc, const lds_u8* tab, int lane) {
    const LAS u32* CMP = (const LAS u32*)(tab + 1024);
    LAS u32* r2 = (LAS u32*)(sc + CD_R2) + lane * 17; LAS u32* r3 = (LAS u32*)(sc + CD_R3) + lane * 9; LAS u32* r4 = (LAS u32*)(sc + CD_R4) + lane * 5;
    const u32x4 z4 = {0, 0, 0, 0};
#pragma unroll
    for (int k = 0; k < 8; ++k) r2[k] = 0;
#pragma unroll
    for (int k = 0; k < 4; ++k) r3[k] = 0;
    r4[0] = 0; r4[1] = 0;
    *lp<u32x4>(sc + CD_V + lane * 16) = z4; if (lane == 0) *lp<u32x4>(sc + CD_V + 1024) = z4;
    u32 B;
    { const u32 y0 = __builtin_amdgcn_alignbit(x[0], x[0], 31), y1 = __builtin_amdgcn_alignbit(x[1], x[1], 31);
      const u32 s = wave_incl_scan(dot4(perm(y1, y0, 0x07050301u), 0x01010101u, 0u));
      B = ((u32)__builtin_amdgcn_readlane((int)s, 63) + 724u) >> 8; B = B > 255u ? 255u : B; }
    const u32 BK = (B << 8 | 0xffu) * 0x00010001u;
    u32 tb[16], bb[16], pn[16];
    u32 pc = 0;
#pragma unroll
    for (int g = 0; g < 16; ++g) {
        const u32 y0 = __builtin_amdgcn_alignbit(x[2 * g], x[2 * g], 31), y1 = __builtin_amdgcn_alignbit(x[2 * g + 1], x[2 * g + 1], 31);
        A[g] = perm(y1, y0, 0x06040200u);
        const u32 d0 = pk_sub(BK, y0), d1 = pk_sub(BK, y1);
        const u32 m0 = pk_min(d0, 0x07ff07ffu), m1 = pk_min(d1, 0x07ff07ffu);
        const u32 tc = perm(m1, m0, 0x07050301u);
        tb[g] = perm(d1, d0, 0x07050301u) ^ 0x04040404u;
        const u32 c1 = perm(0x03030303u, 0x02010003u, tc), ef = perm(0x01010101u, 0x00000001u, tc);
        bb[g] = dot4(c1, 0x40100401u, 0u);
        pc = dot4(ef, 0x01010101u, pc); pn[g] = pc;
        if ((g & 3) == 0) L1[g >> 2] = bb[g]; else L1[g >> 2] |= bb[g] << (8 * (g & 3));
    }
    u32 sel[16];
#pragma unroll
    for (int g = 0; g < 16; ++g) sel[g] = CMP[bb[g]];
    CFENCE();
    u32 p2 = 0, acc = 0;
#pragma unroll
    for (int g = 0; g < 16; ++g) stream_append(r2, perm(0u, tb[g], sel[g]), p2, pn[g], acc);
    r2[p2 >> 2] = acc;
    CFENCE();
    u32 C2[2], p3 = 0;
    { u32 u[8], c[8], b[8], s2[8];
#pragma unroll
      for (int k = 0; k < 8; ++k) u[k] = r2[k];
#pragma unroll
      for (int k = 0; k < 8; ++k) {
          const u32 big = perm(0u, 0u, ((u[k] & 0xf8f8f8f8u) >> 1) + 0x0c0c0c0cu);
          c[k] = perm(0x03030302u, 0x03030100u, u[k] & 0x07070707u) | (big & 0x03030303u);
          b[k] = dot4(c[k], 0x40100401u, 0u);
          if ((k & 3) == 0) C2[k >> 2] = b[k]; else C2[k >> 2] |= b[k] << (8 * (k & 3));
      }
#pragma unroll
      for (int k = 0; k < 8; ++k) s2[k] = CMP[b[k]];
      CFENCE();
      acc = 0;
#pragma unroll
      for (int k = 0; k < 8; ++k)
          stream_append(r3, perm(0u, u[k] ^ 0x02020202u, s2[k]), p3, dot4(perm(0u, 0x01000000u, c[k]), 0x01010101u, p3), acc);
      r3[p3 >> 2] = acc;
      CFENCE(); }
    u32 C3 = 0, p4 = 0;
    { u32 u[4], c[4], b[4], s3[4];
#pragma unroll
      for (int k = 0; k < 4; ++k) u[k] = r3[k];
#pragma unroll
      for (int k = 0; k < 4; ++k) {
          const u32 big = perm(0u, 0u, ((u[k] & 0xfefefefeu) >> 1) + 0x0c0c0c0cu);
          const u32 n14 = perm(0u, 0u, (((u[k] ^ 0x0e0e0e0eu) & 0x7f7f7f7fu) + 0x0c0c0c0cu) | (u[k] & 0x80808080u));
          c[k] = (u[k] & 0x01010101u & ~big) | (big & (0x02020202u | (n14 & 0x01010101u)));
          b[k] = dot4(c[k], 0x40100401u, 0u);
          C3 |= b[k] << (8 * k);
      }
#pragma unroll
      for (int k = 0; k < 4; ++k) s3[k] = CMP[b[k]];
      CFENCE();
      acc = 0;
#pragma unroll
      for (int k = 0; k < 4; ++k) {
          const u32 pk = perm(0u, u[k] ^ 0x06060606u, s3[k]);
          stream_append(r4, pk, p4, dot4(perm(0u, 0x01000000u, c[k]), 0x01010101u, p4), acc);
      }
      r4[p4 >> 2] = acc;
      CFENCE(); }
    const bool over = p2 > 31u || p3 > 15u || p4 > 8u;
    if (__any(over)) return 0x80000000u;
    const u32 q = p2 | p3 << 11 | p4 << 21, qi = wave_incl_scan(q), qe = qi - q;
    const u32 tot = (u32)__builtin_amdgcn_readlane((int)qi, 63);
    const u32 o3 = ((2 * (tot & 0x7ffu) + 31u) >> 5) * 4u, o4 = o3 + ((2 * ((tot >> 11) & 0x3ffu) + 31u) >> 5) * 4u, vlen = o4 + (tot >> 21);
    if (vlen > 1024u) return 0x80000000u;
    LAS u32* V = (LAS u32*)(sc + CD_V);
    { const u32 bit = 2 * (qe & 0x7ffu), sh = bit & 31u; LAS u32* a = V + (bit >> 5);
      const u64 w = ((u64)C2[1] << 32 | C2[0]) << sh;
      __hip_atomic_fetch_or(a, (u32)w, __ATOMIC_RELAXED, __HIP_MEMORY_SCOPE_WAVEFRONT);
      __hip_atomic_fetch_or(a + 1, (u32)(w >> 32), __ATOMIC_RELAXED, __HIP_MEMORY_SCOPE_WAVEFRONT);
      __hip_atomic_fetch_or(a + 2, (C2[1] >> 1) >> (31u - sh), __ATOMIC_RELAXED, __HIP_MEMORY_SCOPE_WAVEFRONT); }
    { const u32 bit = 8 * o3 + 2 * ((qe >> 11) & 0x3ffu), sh = bit & 31u; LAS u32* a = V + (bit >> 5);
      const u64 w = (u64)C3 << sh;
      __hip_atomic_fetch_or(a, (u32)w, __ATOMIC_RELAXED, __HIP_MEMORY_SCOPE_WAVEFRONT);
      __hip_atomic_fetch_or(a + 1, (u32)(w >> 32), __ATOMIC_RELAXED, __HIP_MEMORY_SCOPE_WAVEFRONT); }
    { const u32 r0 = r4[0], r1 = r4[1];
      const u32 bit = 8 * (o4 + (qe >> 21)), sh = bit & 31u; LAS u32* a = V + (bit >> 5);
      const u64 w = ((u64)r1 << 32 | r0) << sh;
      __hip_atomic_fetch_or(a, (u32)w, __ATOMIC_RELAXED, __HIP_MEMORY_SCOPE_WAVEFRONT);
      __hip_atomic_fetch_or(a + 1, (u32)(w >> 32), __ATOMIC_RELAXED, __HIP_MEMORY_SCOPE_WAVEFRONT);
      __hip_atomic_fetch_or(a + 2, (r1 >> 1) >> (31u - sh), __ATOMIC_RELAXED, __HIP_MEMORY_SCOPE_WAVEFRONT); }
    CFENCE();
    return B | vlen << 8;
}

__device__ __forceinline__ void codec_decode(u32 (&x)[32], const u32 (&A)[16], const u32 (&L1)[4], u32 B, const LAS u32* V, lds_u8* sc, const lds_u8* tab, int lane) {
    const LAS u32* DEP = (const LAS u32*)tab;
    LAS u32* r2 = (LAS u32*)(sc + CD_DEC_R2) + lane * 9; LAS u32* r3 = (LAS u32*)(sc + CD_DEC_R3) + lane * 5;
    const u32 T1 = ((B - 1u) & 255u) | ((B - 2u) & 255u) << 8 | ((B - 3u) & 255u) << 16;
    const u32 T2 = ((B - 4u) & 255u) | ((B - 5u) & 255u) << 8 | (B & 255u) << 16;
    const u32 T3 = ((B - 6u) & 255u) | ((B - 7u) & 255u) << 8 | ((B - 8u) & 255u) << 16;
    u32 e1[4], n2 = 0;
#pragma unroll
    for (int k = 0; k < 4; ++k) { e1[k] = L1[k] & (L1[k] >> 1) & 0x55555555u; n2 += __builtin_popcount(e1[k]); }
    const u32 s2i = wave_incl_scan(n2), s2 = s2i - n2, tot2 = (u32)__builtin_amdgcn_readlane((int)s2i, 63);
    u32 d1[16];
#pragma unroll
    for (int g = 0; g < 16; ++g) d1[g] = DEP[(L1[g >> 2] >> (8 * (g & 3))) & 255u];
    u32 C2[2];
    { const u32 bit = 2 * s2, sh = bit & 31u; const LAS u32* a = V + (bit >> 5);
      const u32 d0 = a[0], dd1 = a[1], d2 = a[2];
      const u64 m = (1ull << (2 * n2)) - 1ull;
      C2[0] = __builtin_amdgcn_alignbit(dd1, d0, sh) & (u32)m; C2[1] = __builtin_amdgcn_alignbit(d2, dd1, sh) & (u32)(m >> 32); }
    u32 e2[2]; e2[0] = C2[0] & (C2[0] >> 1) & 0x55555555u; e2[1] = C2[1] & (C2[1] >> 1) & 0x55555555u;
    const u32 n3 = __builtin_popcount(e2[0]) + __builtin_popcount(e2[1]);
    const u32 s3i = wave_incl_scan(n3), s3 = s3i - n3, tot3 = (u32)__builtin_amdgcn_readlane((int)s3i, 63);
    u32 dp2[8];
#pragma unroll
    for (int k = 0; k < 8; ++k) dp2[k] = DEP[(C2[k >> 2] >> (8 * (k & 3))) & 255u];
    const u32 o3 = ((2 * tot2 + 31u) >> 5) * 4u, o4 = o3 + ((2 * tot3 + 31u) >> 5) * 4u;
    u32 C3;
    { const u32 bit = 8 * o3 + 2 * s3, sh = bit & 31u; const LAS u32* a = V + (bit >> 5);
      C3 = __builtin_amdgcn_alignbit(a[1], a[0], sh) & ((1u << (2 * n3)) - 1u); }
    const u32 e3 = C3 & (C3 >> 1) & 0x55555555u, n4 = __builtin_popcount(e3);
    const u32 s4 = wave_incl_scan(n4) - n4;
    u32 dp3[4];
#pragma unroll
    for (int k = 0; k < 4; ++k) dp3[k] = DEP[(C3 >> (8 * k)) & 255u];
    u64 R4;
    { const u32 byte = o4 + s4, sh = (byte & 3u) * 8u; const LAS u32* a = V + (byte >> 2);
      const u32 d0 = a[0], dd1 = a[1], d2 = a[2];
      u32 t0 = __builtin_amdgcn_alignbit(dd1, d0, sh), t1 = __builtin_amdgcn_alignbit(d2, dd1, sh);
      const u32 B4 = B * 0x01010101u;
      t0 = ((B4 | 0x80808080u) - (t0 & 0x7f7f7f7fu)) ^ ((B4 ^ ~t0) & 0x80808080u);
      t1 = ((B4 | 0x80808080u) - (t1 & 0x7f7f7f7fu)) ^ ((B4 ^ ~t1) & 0x80808080u);
      R4 = (u64)t1 << 32 | t0; }
#pragma unroll
    for (int k = 0; k < 4; ++k) {
        const u32 p = k == 0 ? 0u : (u32)__builtin_popcount(e3 & ((1u << (8 * k)) - 1u));
        r3[k] = perm(T3, (u32)(R4 >> (8 * p)), dp3[k]);
    }
    CFENCE();
    { u32 ov[8];
#pragma unroll
      for (int k = 0; k < 8; ++k) {
          u32 p = (k & 3) == 0 ? 0u : (u32)__builtin_popcount(e2[k >> 2] & ((1u << (8 * (k & 3))) - 1u));
          if (k >= 4) p += __builtin_popcount(e2[0]);
          const LAS u32* a = r3 + (p >> 2);
          ov[k] = perm(T2, __builtin_amdgcn_alignbyte(a[1], a[0], p), dp2[k]);
      }
      CFENCE();
#pragma unroll
      for (int k = 0; k < 8; ++k) r2[k] = ov[k]; }
    CFENCE();
    u32 pb = 0;
#pragma unroll
    for (int g = 0; g < 16; ++g) {
        const u32 p = pb + ((g & 3) == 0 ? 0u : (u32)__builtin_popcount(e1[g >> 2] & ((1u << (8 * (g & 3))) - 1u)));
        const LAS u32* a = r2 + (p >> 2);
        const u32 E = perm(T1, __builtin_amdgcn_alignbyte(a[1], a[0], p), d1[g]);
        const u32 y0 = perm(E, A[g], 0x05010400u), y1 = perm(E, A[g], 0x07030602u);
        x[2 * g] = __builtin_amdgcn_alignbit(y0, y0, 1); x[2 * g + 1] = __builtin_amdgcn_alignbit(y1, y1, 1);
        if ((g & 3) == 3) pb += __builtin_popcount(e1[g >> 2]);
    }
    CFENCE();
}

constexpr int NR = 8;
constexpr int MM = 4096, NN = 4096, KSH = 512;
constexpr int NWG = 256, NTH = 512;
constexpr int PIECE_B = 8192;
#define NPUSH 28
constexpr int GA = 12;
constexpr int CHP = 16 + GA;
constexpr int NINA = 8 * CHP - NPUSH, NIN = NINA + (32 - GA) * 8, NOUT = 512, NQ = NIN + NOUT;
#define ND 8
#define NFLI 8
#define NFL 9
constexpr int NSLOT = 11;
constexpr int NPW = (NQ + ND - 1) / ND;
#define PRI0 2
#define PRID 1
#define RAWT 12
constexpr int NST = 6;
constexpr int STAGE_B = 24576;
constexpr size_t OFF_CALL = 0;
constexpr size_t OFF_TMO = 32768;
constexpr size_t OFF_CNT = 65536;
constexpr size_t OFF_PROF = 131072;
constexpr size_t OFF_FLAG = 1u << 20;
constexpr size_t OFF_OBI = 2u << 20;
constexpr size_t OFF_OBO = OFF_OBI + 1024ull * PIECE_B;
constexpr size_t OFF_XS = OFF_OBO + 512ull * PIECE_B;
constexpr size_t OFF_WS = OFF_XS + 64ull * 32 * PIECE_B;
constexpr size_t WS_NEED = OFF_WS + 64ull * 16 * PIECE_B;
static_assert(OFF_FLAG + (size_t)NR * ND * 256 * 8 <= OFF_OBI && NPW <= 256, "flags");
__host__ __device__ constexpr size_t flag_off(int src, int idx) { return OFF_FLAG + ((size_t)(src * ND + idx % ND) * 256 + idx / ND) * 8; }
constexpr int LDS_TAB = 0, LDS_ROLE = 2048;
constexpr int LDS_MISC = LDS_ROLE + NST * STAGE_B;
constexpr int LDS_BYTES = LDS_MISC + 512;
constexpr int LDS_SLOTS = LDS_ROLE, LDS_PRIV = LDS_SLOTS + NSLOT * PIECE_B, LDS_CTL = LDS_PRIV + 6 * CD_BYTES;
static_assert(LDS_CTL + 1280 <= LDS_MISC && NPW <= 128, "decoder layout");
static_assert(LDS_ROLE + 8 * CD_BYTES <= LDS_MISC && ND == 8 && NPUSH <= 32, "phase 0 scratch");
static_assert(16384 + 2 * CD_BYTES <= NST * STAGE_B, "epilogue scratch");
constexpr u64 DEADLINE = 40000000ull;

struct Params {
    const unsigned short* x;
    const unsigned short* w;
    unsigned short* out;
    unsigned char* ws[NR];
    int rank;
    int pad_;
};

typedef short bf16x8 __attribute__((ext_vector_type(8)));
typedef float f32x4 __attribute__((ext_vector_type(4)));
typedef short s16x4 __attribute__((ext_vector_type(4)));
#define GAS __attribute__((address_space(1)))

__device__ __forceinline__ void st_sys(void* p, u32x4 v) { asm volatile("global_store_dwordx4 %0, %1, off sc0 sc1\n\ts_nop 1" :: "v"(p), "v"(v) : "memory"); }
__device__ __forceinline__ void st_agent(void* p, u32x4 v) { asm volatile("global_store_dwordx4 %0, %1, off sc1\n\ts_nop 1" :: "v"(p), "v"(v) : "memory"); }
constexpr int wcnt(int vm, int lgkm) { return (vm & 15) | 0x70 | (lgkm << 8) | ((vm >> 4) << 14); }
__device__ __forceinline__ void wait_vm0() { asm volatile("s_waitcnt vmcnt(0)" ::: "memory"); }
__device__ __forceinline__ u32 rfl(u32 v) { return (u32)__builtin_amdgcn_readfirstlane((int)v); }
__device__ __forceinline__ u64 now() { return __builtin_amdgcn_s_memrealtime(); }

#define STAMP(slot) do { } while (0)
#define STAMPC(slot) do { } while (0)

__device__ __forceinline__ void in_entry(int idx, int& c, bool& isw, int& bl) {
    if (idx < NINA) { const int e = idx + NPUSH; c = e / CHP; const int r = e - c * CHP; isw = r < 16; bl = isw ? r : r - 16; }
    else { const int e = idx - NINA; c = (e >> 1) & 7; isw = false; bl = GA + ((e >> 4) << 1) + (e & 1); }
}
__device__ __forceinline__ bool wait_cnt2(const unsigned char* c0, u32 t0v, const unsigned char* c1, u32 t1v, u32 call, u32* tmo) {
    const u64 t0 = now();
    for (;;) {
        const u32 a = __hip_atomic_load((const GAS u32*)c0, __ATOMIC_RELAXED, __HIP_MEMORY_SCOPE_SYSTEM);
        const u32 b = __hip_atomic_load((const GAS u32*)c1, __ATOMIC_RELAXED, __HIP_MEMORY_SCOPE_SYSTEM);
        if (rfl(a) >= t0v && rfl(b) >= t1v) return true;
        if (rfl(__hip_atomic_load((const GAS u32*)tmo, __ATOMIC_RELAXED, __HIP_MEMORY_SCOPE_AGENT)) == call || now() - t0 > DEADLINE) {
            __hip_atomic_store((GAS u32*)tmo, call, __ATOMIC_RELAXED, __HIP_MEMORY_SCOPE_AGENT);
            return false;
        }
        __builtin_amdgcn_s_sleep(2);
    }
}

__device__ __forceinline__ void store_piece(const u32 (&x)[32], const u32 (&A)[16], const u32 (&L1)[4], u32 inf, unsigned char* slot, lds_u8* sc, int lane) {
    unsigned char* o = slot + lane * 16;
    if (inf >> 31) {
#pragma unroll
        for (int j = 0; j < 8; ++j) st_sys(o + j * 1024, (u32x4){x[4 * j], x[4 * j + 1], x[4 * j + 2], x[4 * j + 3]});
    } else {
#pragma unroll
        for (int j = 0; j < 4; ++j) st_sys(o + j * 1024, (u32x4){A[4 * j], A[4 * j + 1], A[4 * j + 2], A[4 * j + 3]});
        st_sys(o + 4096, (u32x4){L1[0], L1[1], L1[2], L1[3]});
        const u32x4 v = *lp<u32x4>(sc + CD_V + lane * 16);
        if ((u32)lane * 16u < ((inf >> 8) & 0x7ffu)) st_sys(o + 5120, v);
    }
}

__device__ __forceinline__ u32 pk_bf16(float lo, float hi) { u32 r; asm("v_cvt_pk_bf16_f32 %0, %1, %2" : "=v"(r) : "v"(lo), "v"(hi)); return r; }

__global__ __launch_bounds__(NTH, 2) void gemm_ar_kernel(Params P) {
    extern __shared__ __attribute__((aligned(16))) unsigned char smem[];
    lds_u8* sm = (lds_u8*)smem;
    const int tid = threadIdx.x, lane = tid & 63, wid = (int)rfl(tid >> 6);
    const int b = blockIdx.x, xcd = b & 7, q = b >> 3;
    const int rank = P.rank, ri = rank >> 2, rj = rank & 3;
    unsigned char* my = P.ws[rank];
    u32* tmo = (u32*)(my + OFF_TMO);
    const lds_u8* tab = sm + LDS_TAB;
    STAMP(0);
    u32 call_v = 0;
    if (lane == 0) call_v = __hip_atomic_fetch_add((GAS u32*)(my + OFF_CALL + b * 128 + wid * 4), 1u, __ATOMIC_RELAXED, __HIP_MEMORY_SCOPE_AGENT) + 1u;

    unsigned char* wsl = P.ws[lane & 7];

    const bool is_dec = q >= 16 && q < 16 + ND && xcd < NR - 1;
    const int peer_r = (rank + 1 + xcd) & 7;
    const int pi = q * 4 + (wid - 4);
    const bool pusher = wid >= 4 && q < 8 && xcd < NR - 1 && pi < NPUSH;
    const int pid = pusher ? (pi < 16 ? 512 + (16 * (peer_r & 3) + pi) * 8 : (32 * (peer_r >> 2) + pi - 16) * 8) : b * 4 + wid;
    const bool enc = wid < 4 || pusher;
    const bool isw = pid >= 512;
    const int blk = (pid & 511) >> 3, ch = pid & 7;
    u32x4 raw[8];
    auto load_raw = [&]() {
        if (!isw) {
            const unsigned short* s = P.x + (size_t)(blk * 64 + (lane & 15)) * KSH + ch * 64 + (lane >> 4) * 8;
#pragma unroll
            for (int j = 0; j < 8; ++j) raw[j] = *(const u32x4*)(s + (size_t)(j >> 1) * 16 * KSH + (j & 1) * 32);
        } else {
            const unsigned short* s = P.w + (size_t)(ch * 64 + (lane >> 3)) * NN + blk * 64 + (lane & 7) * 8;
#pragma unroll
            for (int j = 0; j < 8; ++j) raw[j] = *(const u32x4*)(s + (size_t)j * 8 * NN);
        }
    };
    const int cls = pusher ? 0 : ((!isw && (blk & 31) >= GA) ? 2 : (ch >= PRI0 ? 1 : 0));
    if (enc && cls == 0) load_raw();
    if (tid < 256) codec_build_tables(sm + LDS_TAB, tid);
    if (is_dec && tid < 320) *lp<u32>(sm + LDS_CTL + tid * 4) = 0;
    __syncthreads();
    const u32 call = rfl(call_v);
    if (enc) {
        if (cls != 0) {
            for (int k = 0; k < cls * PRID - 1; ++k) __builtin_amdgcn_s_sleep(16);
            load_raw();
        }
        lds_u8* sc = sm + (is_dec ? LDS_PRIV : LDS_ROLE) + wid * CD_BYTES;
        u32 x[32];
        if (!isw) {
#pragma unroll
            for (int j = 0; j < 8; ++j) { x[4 * j] = raw[j].x; x[4 * j + 1] = raw[j].y; x[4 * j + 2] = raw[j].z; x[4 * j + 3] = raw[j].w; }
        } else {
#pragma unroll
            for (int j = 0; j < 8; ++j) { const int k = 8 * j + (lane >> 3); *lp<u32x4>(sc + k * 128 + (((lane & 7) ^ (k & 7)) << 4)) = raw[j]; }
            const int g = lane >> 4, qq = (lane & 15) >> 2, pp = lane & 3;
#pragma unroll
            for (int j = 0; j < 8; ++j) {
                const int ni = j >> 1, ks = j & 1;
#pragma unroll
                for (int t = 0; t < 2; ++t) {
                    const int k = 32 * ks + 8 * g + 4 * t + qq, chunk = 2 * ni + (pp >> 1);
                    const s16x4 v = __builtin_amdgcn_ds_read_tr16_b64_v4i16((LAS s16x4*)(sc + k * 128 + ((chunk ^ (k & 7)) << 4) + (pp & 1) * 8));
                    const u32x2 u = __builtin_bit_cast(u32x2, v);
                    x[4 * j + 2 * t] = u.x; x[4 * j + 2 * t + 1] = u.y;
                }
            }
            asm volatile("s_waitcnt lgkmcnt(0)" ::: "memory");
        }
        STAMP(1); STAMPC(28);
        const int rb = blk & 31, step = rank * 8 + ch;
        const size_t stg = isw ? OFF_WS + (size_t)(step * 16 + (blk & 15)) * PIECE_B : OFF_XS + (size_t)(step * 32 + rb) * PIECE_B;
        const size_t cnt = OFF_CNT + (size_t)(isw ? 128 + (blk & 15) * 8 + ch : (rb >> 1) * 8 + ch) * 128;
        const bool cons = lane < NR && (isw ? (lane & 3) == (blk >> 4) : (lane >> 2) == (blk >> 5));
        const bool mine = isw ? ((blk >> 4) == rj) : ((blk >> 5) == ri);
        if (pusher) {
            unsigned char* d = P.ws[peer_r] + stg + lane * 16;
#pragma unroll
            for (int j = 0; j < 8; ++j) st_sys(d + j * 1024, (u32x4){x[4 * j], x[4 * j + 1], x[4 * j + 2], x[4 * j + 3]});
            wait_vm0();
            STAMP(2);
            if (lane == peer_r) __hip_atomic_fetch_add((GAS u32*)(wsl + cnt), 1u, __ATOMIC_RELAXED, __HIP_MEMORY_SCOPE_SYSTEM);
        } else if (ch == 0 && (isw ? (blk & 15) < NPUSH : (rb < GA && 16 + rb < NPUSH))) {
            if (mine) {
                unsigned char* d = my + stg + lane * 16;
#pragma unroll
                for (int j = 0; j < 8; ++j) st_sys(d + j * 1024, (u32x4){x[4 * j], x[4 * j + 1], x[4 * j + 2], x[4 * j + 3]});
                wait_vm0();
                if (lane == 0) __hip_atomic_fetch_add((GAS u32*)(my + cnt), 1u, __ATOMIC_RELAXED, __HIP_MEMORY_SCOPE_SYSTEM);
            }
        } else {
            u32 A[16], L1[4];
            const u32 inf = codec_encode(x, A, L1, sc, tab, lane);
            STAMP(7); STAMPC(29);
            store_piece(x, A, L1, inf, my + OFF_OBI + (size_t)pid * PIECE_B, sc, lane);
            wait_vm0();
            STAMP(2);
            const int idx = isw ? ch * CHP + (blk & 15) - NPUSH : (rb < GA ? ch * CHP + 16 + rb - NPUSH : NINA + ((rb - GA) >> 1) * 16 + ch * 2 + (rb & 1));
            if (cons && lane != rank) __hip_atomic_store((GAS u64*)(wsl + flag_off(rank, idx)), (u64)call << 32 | inf, __ATOMIC_RELAXED, __HIP_MEMORY_SCOPE_SYSTEM);
            if (mine) {
                unsigned char* d = my + stg + lane * 16;
#pragma unroll
                for (int j = 0; j < 8; ++j) st_agent(d + j * 1024, (u32x4){x[4 * j], x[4 * j + 1], x[4 * j + 2], x[4 * j + 3]});
                wait_vm0();
                if (lane == 0) __hip_atomic_fetch_add((GAS u32*)(my + cnt), 1u, __ATOMIC_RELAXED, __HIP_MEMORY_SCOPE_AGENT);
            }
        }
        STAMP(3);
    }

    if (q < 16 || q >= 24) {
        __syncthreads();
        lds_u8* ring = sm + LDS_ROLE;
        const int wr = wid >> 1, wc = wid & 1;
        const int cb = 4 * (xcd & 3) + (q & 3);
        bool bad = false;
        const int ntile = q < 8 ? 2 : 1;
        const int q2 = q < 16 ? q : q - 8;
        for (int t = 0; t < ntile; ++t) {
            const int a = t == 1 ? 12 + 2 * (xcd >> 2) + (q >> 2) : (q2 < 12 ? 3 * (xcd >> 2) + (q2 >> 2) : 6 + 3 * (xcd >> 2) + ((q2 - 12) >> 2));
            const int g = t;
            const unsigned char* gx = my + OFF_XS + (size_t)(2 * a) * PIECE_B;
            const unsigned char* gw = my + OFF_WS + (size_t)cb * PIECE_B;
            const bool ldA = wid < 4;
            const unsigned char* gl = (ldA ? gx + wid * 4096 : gw + (wid - 4) * 4096) + lane * 16;
            const size_t lstep = ldA ? (size_t)32 * PIECE_B : (size_t)16 * PIECE_B;
            f32x4 acc[2][2] = {};
            auto issue = [&](int i) {
                const int step = (i & 7) * 8 + (i >> 3);
                lds_u8* d = ring + (i % NST) * STAGE_B + wid * 4096;
                const unsigned char* sa = gl + (size_t)step * lstep;
#pragma unroll
                for (int j = 0; j < 4; ++j) __builtin_amdgcn_global_load_lds((const GAS u32*)(sa + j * 1024), (LAS u32*)(d + j * 1024), 16, 0, 17);
            };
            const unsigned char* cx = my + OFF_CNT + (size_t)(a * 8) * 128; const unsigned char* cw = my + OFF_CNT + (size_t)(128 + cb * 8) * 128;
            int rdy = 0;
            u32 pa = 0, pb = 0; bool pend = false;
            auto poll_issue = [&]() { pa = __hip_atomic_load((const GAS u32*)(cx + rdy * 128), __ATOMIC_RELAXED, __HIP_MEMORY_SCOPE_SYSTEM); pb = __hip_atomic_load((const GAS u32*)(cw + rdy * 128), __ATOMIC_RELAXED, __HIP_MEMORY_SCOPE_SYSTEM); pend = true; };
            auto poll_take = [&]() { if (pend) { pend = false; if (rfl(pa) >= 16u * call && rfl(pb) >= 8u * call) ++rdy; } };
            auto need = [&](int c) {
                poll_take();
                while (rdy <= c) {
                    if (!wait_cnt2(cx + rdy * 128, 16u * call, cw + rdy * 128, 8u * call, call, tmo)) bad = true;
                    ++rdy;
                }
            };
            auto rd = [&](int i, bf16x8 (&af)[2][2], bf16x8 (&bfr)[2][2]) {
                lds_u8* st = ring + (i % NST) * STAGE_B;
#pragma unroll
                for (int m = 0; m < 2; ++m)
#pragma unroll
                    for (int ks = 0; ks < 2; ++ks) af[m][ks] = *lp<bf16x8>(st + (wr >> 1) * 8192 + (((wr & 1) * 2 + m) * 2 + ks) * 1024 + lane * 16);
#pragma unroll
                for (int n = 0; n < 2; ++n)
#pragma unroll
                    for (int ks = 0; ks < 2; ++ks) bfr[n][ks] = *lp<bf16x8>(st + 16384 + ((wc * 2 + n) * 2 + ks) * 1024 + lane * 16);
            };
            auto mm = [&](bf16x8 (&af)[2][2], bf16x8 (&bfr)[2][2]) {
#pragma unroll
                for (int ks = 0; ks < 2; ++ks)
#pragma unroll
                    for (int m = 0; m < 2; ++m)
#pragma unroll
                        for (int n = 0; n < 2; ++n) acc[m][n] = __builtin_amdgcn_mfma_f32_16x16x32_bf16(bfr[n][ks], af[m][ks], acc[m][n], 0, 0, 0);
            };
            auto kloop = [&](auto role_c) {
                constexpr int ROLE = decltype(role_c)::value;
                bf16x8 a0[2][2], b0[2][2], a1[2][2], b1[2][2];
                if constexpr (ROLE == 1) need(0);
                __builtin_amdgcn_s_barrier();
                if (g == 0) STAMP(4); else STAMP(12);
                if constexpr (ROLE == 0) {
#pragma unroll
                    for (int i = 0; i < NST - 1; ++i) issue(i);
                    __builtin_amdgcn_s_waitcnt(wcnt(4 * (NST - 2), 15));
                }
                __builtin_amdgcn_s_barrier();
                rd(0, a0, b0);
                auto pair = [&](int i, auto tail_c) {
                    constexpr int TAIL = decltype(tail_c)::value;
                    { const int nx = i + NST - 1;
                      if constexpr (ROLE == 0) __builtin_amdgcn_s_waitcnt(wcnt(TAIL == 0 || TAIL + 4 < 64 ? 4 * (NST - 3) : 4 * (63 - TAIL - 1 > 0 ? 63 - TAIL - 1 : 0), 0));
                      else __builtin_amdgcn_s_waitcnt(wcnt(63, 0));
                      __builtin_amdgcn_s_barrier();
                      if constexpr (ROLE == 0 && (TAIL == 0 || TAIL + NST - 1 < 64)) issue(nx);
                      rd(i + 1, a1, b1);
                      __builtin_amdgcn_sched_barrier(0);
                      mm(a0, b0);
                      __builtin_amdgcn_sched_barrier(0);
                      if constexpr (ROLE == 1 && TAIL == 0) { if ((i & 2) == 0 && rdy < 8) { poll_take(); if (rdy < 8) poll_issue(); } }
                    }
                    { const int nx = i + NST;
                      if constexpr (ROLE == 1 && TAIL == 0) { if ((nx & 7) == 0) need(nx >> 3); }
                      if constexpr (ROLE == 0) __builtin_amdgcn_s_waitcnt(wcnt(TAIL == 0 ? 4 * (NST - 3) : 4 * (63 - TAIL - 2 > 0 ? 63 - TAIL - 2 : 0), 0));
                      else __builtin_amdgcn_s_waitcnt(wcnt(63, 0));
                      __builtin_amdgcn_s_barrier();
                      if constexpr (ROLE == 0 && TAIL == 0) issue(nx);
                      if constexpr (TAIL != 62) rd(i + 2, a0, b0);
                      __builtin_amdgcn_sched_barrier(0);
                      mm(a1, b1);
                      __builtin_amdgcn_sched_barrier(0);
                    }
                };
#pragma unroll 1
                for (int i = 0; i < 58; i += 2) pair(i, std::integral_constant<int, 0>{});
                pair(58, std::integral_constant<int, 58>{});
                pair(60, std::integral_constant<int, 60>{});
                pair(62, std::integral_constant<int, 62>{});
            };
            if (wid < 6) kloop(std::integral_constant<int, 0>{});
            else if (wid == 6) kloop(std::integral_constant<int, 1>{});
            else kloop(std::integral_constant<int, 2>{});
            wait_vm0();
            if (g == 0) STAMP(5); else STAMP(13);
            __syncthreads();
#pragma unroll
            for (int m = 0; m < 2; ++m)
#pragma unroll
                for (int n = 0; n < 2; ++n) {
                    const int row = 32 * wr + 16 * m + (lane & 15), col = 32 * wc + 16 * n + 4 * (lane >> 4);
                    const u32x2 v = {pk_bf16(acc[m][n][0], acc[m][n][1]), pk_bf16(acc[m][n][2], acc[m][n][3])};
                    *lp<u32x2>(ring + row * 128 + (((col >> 3) ^ (row & 7)) << 4) + (col & 4) * 2) = v;
                }
            if (tid == 6 * 64) *lp<u32>(sm + LDS_MISC + 272) = (bad || __hip_atomic_load((const GAS u32*)tmo, __ATOMIC_RELAXED, __HIP_MEMORY_SCOPE_AGENT) == call) ? 1u : 0u;
            __syncthreads();
            if (wid < 4) {
                const int h = wid & 1;
                const bool poison = *lp<u32>(sm + LDS_MISC + 272) != 0u;
                u32 x[32];
#pragma unroll
                for (int j = 0; j < 8; ++j) {
                    const int row = 64 * h + 8 * j + (lane >> 3);
                    u32x4 v = *lp<u32x4>(ring + row * 128 + (((lane & 7) ^ (row & 7)) << 4));
                    if (poison) v = (u32x4){0x7fc07fc0u, 0x7fc07fc0u, 0x7fc07fc0u, 0x7fc07fc0u};
                    x[4 * j] = v.x; x[4 * j + 1] = v.y; x[4 * j + 2] = v.z; x[4 * j + 3] = v.w;
                }
                const int o = (a * 16 + cb) * 2 + h;
                if (wid < 2) {
                    lds_u8* sce = ring + 16384 + wid * CD_BYTES;
                    u32 A[16], L1[4];
                    u32 inf = 0x80000000u;
                    if (o < NOUT - RAWT) inf = codec_encode(x, A, L1, sce, tab, lane);
                    store_piece(x, A, L1, inf, my + OFF_OBO + (size_t)o * PIECE_B, sce, lane);
                    wait_vm0();
                    if (lane < NR && lane != rank)
                        __hip_atomic_store((GAS u64*)(wsl + flag_off(rank, NIN + o)), (u64)call << 32 | inf, __ATOMIC_RELAXED, __HIP_MEMORY_SCOPE_SYSTEM);
                } else {
                    unsigned short* d = P.out + (size_t)(2048 * ri + 128 * a + 64 * h + (lane >> 3)) * NN + 1024 * rj + 64 * cb + (lane & 7) * 8;
#pragma unroll
                    for (int j = 0; j < 8; ++j) st_agent(d + (size_t)j * 8 * NN, (u32x4){x[4 * j], x[4 * j + 1], x[4 * j + 2], x[4 * j + 3]});
                }
            }
            if (g == 0) STAMP(6); else STAMP(14);
            __syncthreads();
        }
    } else if (is_dec) {
        const int src = peer_r, si = src >> 2, sj = src & 3;
        const unsigned char* peer = P.ws[src];
        const int dj = q - 16;
        const int np = (NQ - dj + ND - 1) / ND;
        volatile LAS u32* c_info = (volatile LAS u32*)(sm + LDS_CTL);
        volatile LAS u32* c_full = (volatile LAS u32*)(sm + LDS_CTL + 1024);
        volatile LAS u32* c_done = (volatile LAS u32*)(sm + LDS_CTL + 1088);
        volatile LAS u32* c_piece = (volatile LAS u32*)(sm + LDS_CTL + 1152);
        LAS u32* c_ticket = (LAS u32*)(sm + LDS_CTL + 1216);
        auto src_slot = [&](int idx) -> const unsigned char* {
            if (idx >= NIN) return peer + OFF_OBO + (size_t)(idx - NIN) * PIECE_B;
            int c, bl; bool isw2; in_entry(idx, c, isw2, bl);
            return peer + OFF_OBI + (size_t)(isw2 ? 512 + (16 * rj + bl) * 8 + c : (32 * ri + bl) * 8 + c) * PIECE_B;
        };
        if (wid == 6) {
            const unsigned char* fb = my + OFF_FLAG + (size_t)(src * ND + dj) * 256 * 8;
            const u64 t0 = now();
            bool p0 = lane >= np, p1 = lane + 64 >= np;
            int rounds = 0;
            for (;;) {
                u64 v0 = 0, v1 = 0;
                if (!p0) v0 = __hip_atomic_load((const GAS u64*)(fb + (size_t)lane * 8), __ATOMIC_RELAXED, __HIP_MEMORY_SCOPE_SYSTEM);
                if (!p1) v1 = __hip_atomic_load((const GAS u64*)(fb + (size_t)(lane + 64) * 8), __ATOMIC_RELAXED, __HIP_MEMORY_SCOPE_SYSTEM);
                if (!p0 && (u32)(v0 >> 32) >= call) { c_info[lane] = (u32)v0 | 0x40000000u; p0 = true; }
                if (!p1 && (u32)(v1 >> 32) >= call) { c_info[lane + 64] = (u32)v1 | 0x40000000u; p1 = true; }
                if (__all(p0 && p1)) break;
                if ((++rounds & 7) == 0 && (rfl(__hip_atomic_load((const GAS u32*)tmo, __ATOMIC_RELAXED, __HIP_MEMORY_SCOPE_AGENT)) == call || now() - t0 > DEADLINE)) {
                    __hip_atomic_store((GAS u32*)tmo, call, __ATOMIC_RELAXED, __HIP_MEMORY_SCOPE_AGENT);
                    if (!p0) c_info[lane] = 0x60000000u;
                    if (!p1) c_info[lane + 64] = 0x60000000u;
                    break;
                }
                __builtin_amdgcn_s_sleep(2);
            }
        } else if (wid == 7) {
            int pub = 0;
            auto publish_upto = [&](int e) { for (; pub < e; ++pub) if (lane == 0) c_full[pub % NSLOT] = pub + 1; };
            u64 todo0 = np >= 64 ? ~0ull : (1ull << np) - 1ull, todo1 = np > 64 ? (1ull << (np - 64)) - 1ull : 0ull;
            int n = 0; u32 inf = 0;
            auto pick = [&](int s) -> bool {
                const u32 v0 = c_info[lane], v1 = c_info[lane + 64], dn = c_done[s % NSLOT];
                const u64 a0 = __ballot(v0 != 0u) & todo0, a1 = __ballot(v1 != 0u) & todo1;
                if ((a0 | a1) == 0ull || !(s < NSLOT || rfl(dn) == (u32)(s - NSLOT + 1))) return false;
                if (a0) { const int l = __builtin_ctzll(a0); todo0 &= ~(1ull << l); n = l; inf = (u32)__builtin_amdgcn_readlane((int)v0, l); }
                else { const int l = __builtin_ctzll(a1); todo1 &= ~(1ull << l); n = 64 + l; inf = (u32)__builtin_amdgcn_readlane((int)v1, l); }
                return true;
            };
            bool have = false;
            for (int s = 0; s < np; ++s) {
                const int slot = s % NSLOT;
                if (!have) { wait_vm0(); publish_upto(s); while (!pick(s)) __builtin_amdgcn_s_sleep(1); }
                if (lane == 0) c_piece[slot] = n;
                lds_u8* d = sm + LDS_SLOTS + slot * PIECE_B;
                const unsigned char* s0 = src_slot(dj + n * ND);
                const unsigned char* sp = s0 + lane * 16;
                if (inf & 0x20000000u) { have = false; wait_vm0(); publish_upto(s + 1); continue; }
                const bool israw = (inf >> 31) != 0u;
                const unsigned char* sv = (israw || (u32)lane * 16u < ((inf >> 8) & 0x7ffu)) ? sp + 5120 : s0 + 5120;
                const bool input = dj + n * ND < NIN;
                auto pull = [&](auto nfl_c, auto cnt_c) {
                    constexpr int N = decltype(nfl_c)::value, CNT = decltype(cnt_c)::value;
#pragma unroll
                    for (int j = 0; j < CNT; ++j) {
                        asm volatile("s_waitcnt vmcnt(%0)" :: "n"(N - 1) : "memory");
                        if (j == 0 || (j - N + 1) % 6 == 0) publish_upto(s + (j - N + 1 - 5) / 6);
                        __builtin_amdgcn_global_load_lds((const GAS u32*)(j == 5 ? sv : sp + j * 1024), (LAS u32*)(d + j * 1024), 16, 0, 17);
                        if (j == 1) have = s + 1 < np && pick(s + 1);
                    }
                };
                static_assert(NFL >= 9 && NFLI >= 7, "the publish rule needs j - N + 1 < 0");
                if (israw) pull(std::integral_constant<int, NFL>{}, std::integral_constant<int, 8>{});
                else if (input) pull(std::integral_constant<int, NFLI>{}, std::integral_constant<int, 6>{});
                else pull(std::integral_constant<int, NFL>{}, std::integral_constant<int, 6>{});
            }
            wait_vm0(); publish_upto(np);
            STAMP(9);
        } else {
            lds_u8* sc = sm + LDS_PRIV + wid * CD_BYTES;
            int nstamp = 0;
            for (;;) {
                u32 nn = 0;
                if (lane == 0) nn = __hip_atomic_fetch_add(c_ticket, 1u, __ATOMIC_RELAXED, __HIP_MEMORY_SCOPE_WORKGROUP);
                const int sq = (int)rfl(nn);
                if (sq >= np) break;
                while (rfl(c_full[sq % NSLOT]) != (u32)(sq + 1)) __builtin_amdgcn_s_sleep(1);
                asm volatile("" ::: "memory");
                const int n = (int)rfl(c_piece[sq % NSLOT]);
                const u32 inf = rfl(c_info[n]);
                const bool ok = !(inf & 0x20000000u);
                lds_u8* sl = sm + LDS_SLOTS + (sq % NSLOT) * PIECE_B;
                u32 x[32];
                if (!ok) {
#pragma unroll
                    for (int j = 0; j < 32; ++j) x[j] = 0x7fc07fc0u;
                } else if (inf >> 31) {
#pragma unroll
                    for (int j = 0; j < 8; ++j) { const u32x4 v = *lp<u32x4>(sl + j * 1024 + lane * 16); x[4 * j] = v.x; x[4 * j + 1] = v.y; x[4 * j + 2] = v.z; x[4 * j + 3] = v.w; }
                } else {
                    u32 A[16], L1[4];
#pragma unroll
                    for (int j = 0; j < 4; ++j) { const u32x4 v = *lp<u32x4>(sl + j * 1024 + lane * 16); A[4 * j] = v.x; A[4 * j + 1] = v.y; A[4 * j + 2] = v.z; A[4 * j + 3] = v.w; }
                    { const u32x4 v = *lp<u32x4>(sl + 4096 + lane * 16); L1[0] = v.x; L1[1] = v.y; L1[2] = v.z; L1[3] = v.w; }
                    codec_decode(x, A, L1, inf & 255u, (const LAS u32*)(sl + 5120), sc, tab, lane);
                }
                asm volatile("s_waitcnt lgkmcnt(0)" ::: "memory");
                if (lane == 0) c_done[sq % NSLOT] = sq + 1;
                const int idx = dj + n * ND;
                if (idx < NIN) {
                    int c, bl; bool isw2; in_entry(idx, c, isw2, bl);
                    const int step = src * 8 + c;
                    unsigned char* d = (isw2 ? my + OFF_WS + (size_t)(step * 16 + bl) * PIECE_B : my + OFF_XS + (size_t)(step * 32 + bl) * PIECE_B) + lane * 16;
                    if (ok) {
#pragma unroll
                        for (int j = 0; j < 8; ++j) st_agent(d + j * 1024, (u32x4){x[4 * j], x[4 * j + 1], x[4 * j + 2], x[4 * j + 3]});
                        wait_vm0();
                        if (lane == 0) __hip_atomic_fetch_add((GAS u32*)(my + OFF_CNT + (size_t)(isw2 ? 128 + bl * 8 + c : (bl >> 1) * 8 + c) * 128), 1u, __ATOMIC_RELAXED, __HIP_MEMORY_SCOPE_AGENT);
                    }
                } else {
                    const int o = idx - NIN, g = o >> 8, tile = (o & 255) >> 1, h = o & 1;
                    unsigned short* d = P.out + (size_t)(2048 * si + 1024 * g + 128 * (tile >> 4) + 64 * h + (lane >> 3)) * NN + 1024 * sj + 64 * (tile & 15) + (lane & 7) * 8;
#pragma unroll
                    for (int j = 0; j < 8; ++j) st_agent(d + (size_t)j * 8 * NN, (u32x4){x[4 * j], x[4 * j + 1], x[4 * j + 2], x[4 * j + 3]});
                }
            }
            (void)nstamp;
        }
    }
    STAMP(31);
}

extern "C" void kernel_launch(int rank, int nranks, ncclComm_t comm, hipStream_t stream,
                              void* const* d_in, const int* in_sizes, int n_in,
                              void* d_out, int out_size, const int* in_dtypes, int out_dtype,
                              void* const* d_out_all, void* const* d_ws_all, size_t ws_bytes) {
    (void)comm; (void)in_dtypes; (void)out_dtype; (void)d_out_all;
    if (nranks != NR || n_in != 2 || rank < 0 || rank >= NR || in_sizes[0] != MM * KSH || in_sizes[1] != KSH * NN || out_size != MM * NN || ws_bytes < WS_NEED) {
        fprintf(stderr, "kernel_launch: unexpected problem shape\n");
        return;
    }
    static bool first = true;
    if (first) {
        hipError_t e = hipFuncSetAttribute((const void*)gemm_ar_kernel, hipFuncAttributeMaxDynamicSharedMemorySize, LDS_BYTES);
        if (e != hipSuccess) { fprintf(stderr, "hipFuncSetAttribute: %s\n", hipGetErrorString(e)); return; }
        first = false;
    }
    Params P;
    P.x = (const unsigned short*)d_in[0];
    P.w = (const unsigned short*)d_in[1];
    P.out = (unsigned short*)d_out;
    for (int p = 0; p < NR; ++p) P.ws[p] = (unsigned char*)d_ws_all[p];
    P.rank = rank;
    P.pad_ = 0;
    hipLaunchKernelGGL(gemm_ar_kernel, dim3(NWG), dim3(NTH), LDS_BYTES, stream, P);
    hipError_t e = hipGetLastError();
    if (e != hipSuccess) fprintf(stderr, "launch: %s\n", hipGetErrorString(e));
}
```

```cpp
#include <hip/hip_runtime.h>
#include <rccl/rccl.h>
#include <cstdio>
#include <type_traits>

typedef unsigned u32;
typedef unsigned long long u64;
typedef unsigned u32x4 __attribute__((ext_vector_type(4)));
typedef unsigned u32x2 __attribute__((ext_vector_type(2)));
typedef unsigned short u16x2 __attribute__((ext_vector_type(2)));
#define LAS __attribute__((address_space(3)))
typedef LAS unsigned char lds_u8;

constexpr int CD_V = 0;
constexpr int CD_R2 = 1040;
constexpr int CD_R3 = CD_R2 + 64 * 68;
constexpr int CD_R4 = CD_R3 + 64 * 36;
constexpr int CD_BYTES = CD_R4 + 64 * 20;
constexpr int CD_DEC_R2 = 1040, CD_DEC_R3 = CD_DEC_R2 + 64 * 36, CD_DEC_BYTES = CD_DEC_R3 + 64 * 20;
constexpr int CD_TAB_BYTES = 2048;
static_assert(CD_BYTES % 16 == 0 && CD_DEC_BYTES % 16 == 0, "align");

__device__ __forceinline__ u32 perm(u32 hi, u32 lo, u32 sel) { return __builtin_amdgcn_perm(hi, lo, sel); }
__device__ __forceinline__ u32 dot4(u32 a, u32 b, u32 c) { return __builtin_amdgcn_udot4(a, b, c, false); }
__device__ __forceinline__ u32 pk_sub(u32 a, u32 b) { return __builtin_bit_cast(u32, (u16x2)(__builtin_bit_cast(u16x2, a) - __builtin_bit_cast(u16x2, b))); }
__device__ __forceinline__ u32 pk_min(u32 a, u32 b) { return __builtin_bit_cast(u32, __builtin_elementwise_min(__builtin_bit_cast(u16x2, a), __builtin_bit_cast(u16x2, b))); }
__device__ __forceinline__ u32 wave_incl_scan(u32 x) {
    x += (u32)__builtin_amdgcn_update_dpp(0, (int)x, 0x111, 0xf, 0xf, false);
    x += (u32)__builtin_amdgcn_update_dpp(0, (int)x, 0x112, 0xf, 0xf, false);
    x += (u32)__builtin_amdgcn_update_dpp(0, (int)x, 0x114, 0xf, 0xf, false);
    x += (u32)__builtin_amdgcn_update_dpp(0, (int)x, 0x118, 0xf, 0xf, false);
    x += (u32)__builtin_amdgcn_update_dpp(0, (int)x, 0x142, 0xa, 0xf, false);
    x += (u32)__builtin_amdgcn_update_dpp(0, (int)x, 0x143, 0xc, 0xf, false);
    return x;
}
template <class T> __device__ __forceinline__ LAS T* lp(lds_u8* p) { return (LAS T*)p; }
#define CFENCE() asm volatile("" ::: "memory")

__device__ __forceinline__ void codec_build_tables(lds_u8* tab, int i) {
    u32 dep = 0, cmp = 0x0c0c0c0cu; int n = 0;
#pragma unroll
    for (int k = 0; k < 4; ++k) {
        const u32 c = (i >> (2 * k)) & 3;
        if (c == 3) { dep |= (u32)n << (8 * k); cmp = (cmp & ~(0xffu << (8 * n))) | ((u32)k << (8 * n)); ++n; }
        else dep |= (4u + c) << (8 * k);
    }
    lp<u32>(tab)[i] = dep; lp<u32>(tab)[256 + i] = cmp;
}

__device__ __forceinline__ void stream_append(LAS u32* r, u32 pk, u32& p, u32 pnew, u32& acc) {
    const u64 w = (u64)pk << ((p & 3u) * 8u);
    const u32 lo = (u32)w | acc;
    r[p >> 2] = lo;
    acc = ((p ^ pnew) & ~3u) ? (u32)(w >> 32) : lo;
    p = pnew;
}

__device__ __forceinline__ u32 codec_encode(const u32 (&x)[32], u32 (&A)[16], u32 (&L1)[4], lds_u8* sc, const lds_u8* tab, int lane) {
    const LAS u32* CMP = (const LAS u32*)(tab + 1024);
    LAS u32* r2 = (LAS u32*)(sc + CD_R2) + lane * 17; LAS u32* r3 = (LAS u32*)(sc + CD_R3) + lane * 9; LAS u32* r4 = (LAS u32*)(sc + CD_R4) + lane * 5;
    const u32x4 z4 = {0, 0, 0, 0};
#pragma unroll
    for (int k = 0; k < 8; ++k) r2[k] = 0;
#pragma unroll
    for (int k = 0; k < 4; ++k) r3[k] = 0;
    r4[0] = 0; r4[1] = 0;
    *lp<u32x4>(sc + CD_V + lane * 16) = z4; if (lane == 0) *lp<u32x4>(sc + CD_V + 1024) = z4;
    u32 B;
    { const u32 y0 = __builtin_amdgcn_alignbit(x[0], x[0], 31), y1 = __builtin_amdgcn_alignbit(x[1], x[1], 31);
      const u32 s = wave_incl_scan(dot4(perm(y1, y0, 0x07050301u), 0x01010101u, 0u));
      B = ((u32)__builtin_amdgcn_readlane((int)s, 63) + 724u) >> 8; B = B > 255u ? 255u : B; }
    const u32 BK = (B << 8 | 0xffu) * 0x00010001u;
    u32 tb[16], bb[16], pn[16];
    u32 pc = 0;
#pragma unroll
    for (int g = 0; g < 16; ++g) {
        const u32 y0 = __builtin_amdgcn_alignbit(x[2 * g], x[2 * g], 31), y1 = __builtin_amdgcn_alignbit(x[2 * g + 1], x[2 * g + 1], 31);
        A[g] = perm(y1, y0, 0x06040200u);
        const u32 d0 = pk_sub(BK, y0), d1 = pk_sub(BK, y1);
        const u32 m0 = pk_min(d0, 0x07ff07ffu), m1 = pk_min(d1, 0x07ff07ffu);
        const u32 tc = perm(m1, m0, 0x07050301u);
        tb[g] = perm(d1, d0, 0x07050301u) ^ 0x04040404u;
        const u32 c1 = perm(0x03030303u, 0x02010003u, tc), ef = perm(0x01010101u, 0x00000001u, tc);
        bb[g] = dot4(c1, 0x40100401u, 0u);
        pc = dot4(ef, 0x01010101u, pc); pn[g] = pc;
        if ((g & 3) == 0) L1[g >> 2] = bb[g]; else L1[g >> 2] |= bb[g] << (8 * (g & 3));
    }
    u32 sel[16];
#pragma unroll
    for (int g = 0; g < 16; ++g) sel[g] = CMP[bb[g]];
    CFENCE();
    u32 p2 = 0, acc = 0;
#pragma unroll
    for (int g = 0; g < 16; ++g) stream_append(r2, perm(0u, tb[g], sel[g]), p2, pn[g], acc);
    r2[p2 >> 2] = acc;
    CFENCE();
    u32 C2[2], p3 = 0;
    { u32 u[8], c[8], b[8], s2[8];
#pragma unroll
      for (int k = 0; k < 8; ++k) u[k] = r2[k];
#pragma unroll
      for (int k = 0; k < 8; ++k) {
          const u32 big = perm(0u, 0u, ((u[k] & 0xf8f8f8f8u) >> 1) + 0x0c0c0c0cu);
          c[k] = perm(0x03030302u, 0x03030100u, u[k] & 0x07070707u) | (big & 0x03030303u);
          b[k] = dot4(c[k], 0x40100401u, 0u);
          if ((k & 3) == 0) C2[k >> 2] = b[k]; else C2[k >> 2] |= b[k] << (8 * (k & 3));
      }
#pragma unroll
      for (int k = 0; k < 8; ++k) s2[k] = CMP[b[k]];
      CFENCE();
      acc = 0;
#pragma unroll
      for (int k = 0; k < 8; ++k)
          stream_append(r3, perm(0u, u[k] ^ 0x02020202u, s2[k]), p3, dot4(perm(0u, 0x01000000u, c[k]), 0x01010101u, p3), acc);
      r3[p3 >> 2] = acc;
      CFENCE(); }
    u32 C3 = 0, p4 = 0;
    { u32 u[4], c[4], b[4], s3[4];
#pragma unroll
      for (int k = 0; k < 4; ++k) u[k] = r3[k];
#pragma unroll
      for (int k = 0; k < 4; ++k) {
          const u32 big = perm(0u, 0u, ((u[k] & 0xfefefefeu) >> 1) + 0x0c0c0c0cu);
          const u32 n14 = perm(0u, 0u, (((u[k] ^ 0x0e0e0e0eu) & 0x7f7f7f7fu) + 0x0c0c0c0cu) | (u[k] & 0x80808080u));
          c[k] = (u[k] & 0x01010101u & ~big) | (big & (0x02020202u | (n14 & 0x01010101u)));
          b[k] = dot4(c[k], 0x40100401u, 0u);
          C3 |= b[k] << (8 * k);
      }
#pragma unroll
      for (int k = 0; k < 4; ++k) s3[k] = CMP[b[k]];
      CFENCE();
      acc = 0;
#pragma unroll
      for (int k = 0; k < 4; ++k) {
          const u32 pk = perm(0u, u[k] ^ 0x06060606u, s3[k]);
          stream_append(r4, pk, p4, dot4(perm(0u, 0x01000000u, c[k]), 0x01010101u, p4), acc);
      }
      r4[p4 >> 2] = acc;
      CFENCE(); }
    const bool over = p2 > 31u || p3 > 15u || p4 > 8u;
    if (__any(over)) return 0x80000000u;
    const u32 q = p2 | p3 << 11 | p4 << 21, qi = wave_incl_scan(q), qe = qi - q;
    const u32 tot = (u32)__builtin_amdgcn_readlane((int)qi, 63);
    const u32 o3 = ((2 * (tot & 0x7ffu) + 31u) >> 5) * 4u, o4 = o3 + ((2 * ((tot >> 11) & 0x3ffu) + 31u) >> 5) * 4u, vlen = o4 + (tot >> 21);
    if (vlen > 1024u) return 0x80000000u;
    LAS u32* V = (LAS u32*)(sc + CD_V);
    { const u32 bit = 2 * (qe & 0x7ffu), sh = bit & 31u; LAS u32* a = V + (bit >> 5);
      const u64 w = ((u64)C2[1] << 32 | C2[0]) << sh;
      __hip_atomic_fetch_or(a, (u32)w, __ATOMIC_RELAXED, __HIP_MEMORY_SCOPE_WAVEFRONT);
      __hip_atomic_fetch_or(a + 1, (u32)(w >> 32), __ATOMIC_RELAXED, __HIP_MEMORY_SCOPE_WAVEFRONT);
      __hip_atomic_fetch_or(a + 2, (C2[1] >> 1) >> (31u - sh), __ATOMIC_RELAXED, __HIP_MEMORY_SCOPE_WAVEFRONT); }
    { const u32 bit = 8 * o3 + 2 * ((qe >> 11) & 0x3ffu), sh = bit & 31u; LAS u32* a = V + (bit >> 5);
      const u64 w = (u64)C3 << sh;
      __hip_atomic_fetch_or(a, (u32)w, __ATOMIC_RELAXED, __HIP_MEMORY_SCOPE_WAVEFRONT);
      __hip_atomic_fetch_or(a + 1, (u32)(w >> 32), __ATOMIC_RELAXED, __HIP_MEMORY_SCOPE_WAVEFRONT); }
    { const u32 r0 = r4[0], r1 = r4[1];
      const u32 bit = 8 * (o4 + (qe >> 21)), sh = bit & 31u; LAS u32* a = V + (bit >> 5);
      const u64 w = ((u64)r1 << 32 | r0) << sh;
      __hip_atomic_fetch_or(a, (u32)w, __ATOMIC_RELAXED, __HIP_MEMORY_SCOPE_WAVEFRONT);
      __hip_atomic_fetch_or(a + 1, (u32)(w >> 32), __ATOMIC_RELAXED, __HIP_MEMORY_SCOPE_WAVEFRONT);
      __hip_atomic_fetch_or(a + 2, (r1 >> 1) >> (31u - sh), __ATOMIC_RELAXED, __HIP_MEMORY_SCOPE_WAVEFRONT); }
    CFENCE();
    return B | vlen << 8;
}

__device__ __forceinline__ void codec_decode(u32 (&x)[32], const u32 (&A)[16], const u32 (&L1)[4], u32 B, const LAS u32* V, lds_u8* sc, const lds_u8* tab, int lane) {
    const LAS u32* DEP = (const LAS u32*)tab;
    LAS u32* r2 = (LAS u32*)(sc + CD_DEC_R2) + lane * 9; LAS u32* r3 = (LAS u32*)(sc + CD_DEC_R3) + lane * 5;
    const u32 T1 = ((B - 1u) & 255u) | ((B - 2u) & 255u) << 8 | ((B - 3u) & 255u) << 16;
    const u32 T2 = ((B - 4u) & 255u) | ((B - 5u) & 255u) << 8 | (B & 255u) << 16;
    const u32 T3 = ((B - 6u) & 255u) | ((B - 7u) & 255u) << 8 | ((B - 8u) & 255u) << 16;
    u32 e1[4], n2 = 0;
#pragma unroll
    for (int k = 0; k < 4; ++k) { e1[k] = L1[k] & (L1[k] >> 1) & 0x55555555u; n2 += __builtin_popcount(e1[k]); }
    const u32 s2i = wave_incl_scan(n2), s2 = s2i - n2, tot2 = (u32)__builtin_amdgcn_readlane((int)s2i, 63);
    u32 d1[16];
#pragma unroll
    for (int g = 0; g < 16; ++g) d1[g] = DEP[(L1[g >> 2] >> (8 * (g & 3))) & 255u];
    u32 C2[2];
    { const u32 bit = 2 * s2, sh = bit & 31u; const LAS u32* a = V + (bit >> 5);
      const u32 d0 = a[0], dd1 = a[1], d2 = a[2];
      const u64 m = (1ull << (2 * n2)) - 1ull;
      C2[0] = __builtin_amdgcn_alignbit(dd1, d0, sh) & (u32)m; C2[1] = __builtin_amdgcn_alignbit(d2, dd1, sh) & (u32)(m >> 32); }
    u32 e2[2]; e2[0] = C2[0] & (C2[0] >> 1) & 0x55555555u; e2[1] = C2[1] & (C2[1] >> 1) & 0x55555555u;
    const u32 n3 = __builtin_popcount(e2[0]) + __builtin_popcount(e2[1]);
    const u32 s3i = wave_incl_scan(n3), s3 = s3i - n3, tot3 = (u32)__builtin_amdgcn_readlane((int)s3i, 63);
    u32 dp2[8];
#pragma unroll
    for (int k = 0; k < 8; ++k) dp2[k] = DEP[(C2[k >> 2] >> (8 * (k & 3))) & 255u];
    const u32 o3 = ((2 * tot2 + 31u) >> 5) * 4u, o4 = o3 + ((2 * tot3 + 31u) >> 5) * 4u;
    u32 C3;
    { const u32 bit = 8 * o3 + 2 * s3, sh = bit & 31u; const LAS u32* a = V + (bit >> 5);
      C3 = __builtin_amdgcn_alignbit(a[1], a[0], sh) & ((1u << (2 * n3)) - 1u); }
    const u32 e3 = C3 & (C3 >> 1) & 0x55555555u, n4 = __builtin_popcount(e3);
    const u32 s4 = wave_incl_scan(n4) - n4;
    u32 dp3[4];
#pragma unroll
    for (int k = 0; k < 4; ++k) dp3[k] = DEP[(C3 >> (8 * k)) & 255u];
    u64 R4;
    { const u32 byte = o4 + s4, sh = (byte & 3u) * 8u; const LAS u32* a = V + (byte >> 2);
      const u32 d0 = a[0], dd1 = a[1], d2 = a[2];
      u32 t0 = __builtin_amdgcn_alignbit(dd1, d0, sh), t1 = __builtin_amdgcn_alignbit(d2, dd1, sh);
      const u32 B4 = B * 0x01010101u;
      t0 = ((B4 | 0x80808080u) - (t0 & 0x7f7f7f7fu)) ^ ((B4 ^ ~t0) & 0x80808080u);
      t1 = ((B4 | 0x80808080u) - (t1 & 0x7f7f7f7fu)) ^ ((B4 ^ ~t1) & 0x80808080u);
      R4 = (u64)t1 << 32 | t0; }
#pragma unroll
    for (int k = 0; k < 4; ++k) {
        const u32 p = k == 0 ? 0u : (u32)__builtin_popcount(e3 & ((1u << (8 * k)) - 1u));
        r3[k] = perm(T3, (u32)(R4 >> (8 * p)), dp3[k]);
    }
    CFENCE();
    { u32 ov[8];
#pragma unroll
      for (int k = 0; k < 8; ++k) {
          u32 p = (k & 3) == 0 ? 0u : (u32)__builtin_popcount(e2[k >> 2] & ((1u << (8 * (k & 3))) - 1u));
          if (k >= 4) p += __builtin_popcount(e2[0]);
          const LAS u32* a = r3 + (p >> 2);
          ov[k] = perm(T2, __builtin_amdgcn_alignbyte(a[1], a[0], p), dp2[k]);
      }
      CFENCE();
#pragma unroll
      for (int k = 0; k < 8; ++k) r2[k] = ov[k]; }
    CFENCE();
    u32 pb = 0;
#pragma unroll
    for (int g = 0; g < 16; ++g) {
        const u32 p = pb + ((g & 3) == 0 ? 0u : (u32)__builtin_popcount(e1[g >> 2] & ((1u << (8 * (g & 3))) - 1u)));
        const LAS u32* a = r2 + (p >> 2);
        const u32 E = perm(T1, __builtin_amdgcn_alignbyte(a[1], a[0], p), d1[g]);
        const u32 y0 = perm(E, A[g], 0x05010400u), y1 = perm(E, A[g], 0x07030602u);
        x[2 * g] = __builtin_amdgcn_alignbit(y0, y0, 1); x[2 * g + 1] = __builtin_amdgcn_alignbit(y1, y1, 1);
        if ((g & 3) == 3) pb += __builtin_popcount(e1[g >> 2]);
    }
    CFENCE();
}

constexpr int NR = 8;
constexpr int MM = 4096, NN = 4096, KSH = 512;
constexpr int NWG = 256, NTH = 512;
constexpr int PIECE_B = 8192;
#define NPUSH 24
constexpr int GA = 12;
constexpr int CHP = 16 + GA;
constexpr int NINA = 8 * CHP - NPUSH, NIN = NINA + (32 - GA) * 8, NOUT = 512, NQ = NIN + NOUT;
#define ND 8
#define NFLI 8
#define NFL 9
constexpr int NSLOT = 11;
constexpr int NPW = (NQ + ND - 1) / ND;
#define PRI0 2
#define PRID 2
#define RAWT 16
constexpr int NST = 6;
constexpr int STAGE_B = 24576;
constexpr size_t OFF_CALL = 0;
constexpr size_t OFF_TMO = 32768;
constexpr size_t OFF_CNT = 65536;
constexpr size_t OFF_FLAG = 1u << 20;
constexpr size_t OFF_OBI = 2u << 20;
constexpr size_t OFF_OBO = OFF_OBI + 1024ull * PIECE_B;
constexpr size_t OFF_XS = OFF_OBO + 512ull * PIECE_B;
constexpr size_t OFF_WS = OFF_XS + 64ull * 32 * PIECE_B;
constexpr size_t WS_NEED = OFF_WS + 64ull * 16 * PIECE_B;
static_assert(OFF_FLAG + (size_t)NR * ND * 256 * 8 <= OFF_OBI && NPW <= 256, "flags");
__host__ __device__ constexpr size_t flag_off(int src, int idx) { return OFF_FLAG + ((size_t)(src * ND + idx % ND) * 256 + idx / ND) * 8; }
constexpr int LDS_TAB = 0, LDS_ROLE = 2048;
constexpr int LDS_MISC = LDS_ROLE + NST * STAGE_B;
constexpr int LDS_BYTES = LDS_MISC + 512;
constexpr int LDS_SLOTS = LDS_ROLE, LDS_PRIV = LDS_SLOTS + NSLOT * PIECE_B, LDS_CTL = LDS_PRIV + 6 * CD_BYTES;
static_assert(LDS_CTL + 1280 <= LDS_MISC && NPW <= 128, "decoder layout");
static_assert(LDS_ROLE + 8 * CD_BYTES <= LDS_MISC && ND == 8 && NPUSH <= 32, "phase 0 scratch");
static_assert(16384 + 2 * CD_BYTES <= NST * STAGE_B, "epilogue scratch");
constexpr u64 DEADLINE = 40000000ull;

struct Params {
    const unsigned short* x;
    const unsigned short* w;
    unsigned short* out;
    unsigned char* ws[NR];
    int rank;
    int pad_;
};

typedef short bf16x8 __attribute__((ext_vector_type(8)));
typedef float f32x4 __attribute__((ext_vector_type(4)));
typedef short s16x4 __attribute__((ext_vector_type(4)));
#define GAS __attribute__((address_space(1)))

__device__ __forceinline__ void st_sys(void* p, u32x4 v) { asm volatile("global_store_dwordx4 %0, %1, off sc0 sc1\n\ts_nop 1" :: "v"(p), "v"(v) : "memory"); }
__device__ __forceinline__ void st_agent(void* p, u32x4 v) { asm volatile("global_store_dwordx4 %0, %1, off sc1\n\ts_nop 1" :: "v"(p), "v"(v) : "memory"); }
constexpr int wcnt(int vm, int lgkm) { return (vm & 15) | 0x70 | (lgkm << 8) | ((vm >> 4) << 14); }
__device__ __forceinline__ void wait_vm0() { asm volatile("s_waitcnt vmcnt(0)" ::: "memory"); }
__device__ __forceinline__ u32 rfl(u32 v) { return (u32)__builtin_amdgcn_readfirstlane((int)v); }
__device__ __forceinline__ u64 now() { return __builtin_amdgcn_s_memrealtime(); }

__device__ __forceinline__ void in_entry(int idx, int& c, bool& isw, int& bl) {
    if (idx < NINA) { const int e = idx + NPUSH; c = e / CHP; const int r = e - c * CHP; isw = r < 16; bl = isw ? r : r - 16; }
    else { const int e = idx - NINA; c = (e >> 1) & 7; isw = false; bl = GA + ((e >> 4) << 1) + (e & 1); }
}
__device__ __forceinline__ bool wait_cnt2(const unsigned char* c0, u32 t0v, const unsigned char* c1, u32 t1v, u32 call, u32* tmo) {
    const u64 t0 = now();
    for (;;) {
        const u32 a = __hip_atomic_load((const GAS u32*)c0, __ATOMIC_RELAXED, __HIP_MEMORY_SCOPE_SYSTEM);
        const u32 b = __hip_atomic_load((const GAS u32*)c1, __ATOMIC_RELAXED, __HIP_MEMORY_SCOPE_SYSTEM);
        if (rfl(a) >= t0v && rfl(b) >= t1v) return true;
        if (rfl(__hip_atomic_load((const GAS u32*)tmo, __ATOMIC_RELAXED, __HIP_MEMORY_SCOPE_AGENT)) == call || now() - t0 > DEADLINE) {
            __hip_atomic_store((GAS u32*)tmo, call, __ATOMIC_RELAXED, __HIP_MEMORY_SCOPE_AGENT);
            return false;
        }
        __builtin_amdgcn_s_sleep(2);
    }
}

__device__ __forceinline__ void store_piece(const u32 (&x)[32], const u32 (&A)[16], const u32 (&L1)[4], u32 inf, unsigned char* slot, lds_u8* sc, int lane) {
    unsigned char* o = slot + lane * 16;
    if (inf >> 31) {
#pragma unroll
        for (int j = 0; j < 8; ++j) st_sys(o + j * 1024, (u32x4){x[4 * j], x[4 * j + 1], x[4 * j + 2], x[4 * j + 3]});
    } else {
#pragma unroll
        for (int j = 0; j < 4; ++j) st_sys(o + j * 1024, (u32x4){A[4 * j], A[4 * j + 1], A[4 * j + 2], A[4 * j + 3]});
        st_sys(o + 4096, (u32x4){L1[0], L1[1], L1[2], L1[3]});
        const u32x4 v = *lp<u32x4>(sc + CD_V + lane * 16);
        if ((u32)lane * 16u < ((inf >> 8) & 0x7ffu)) st_sys(o + 5120, v);
    }
}

__device__ __forceinline__ u32 pk_bf16(float lo, float hi) { u32 r; asm("v_cvt_pk_bf16_f32 %0, %1, %2" : "=v"(r) : "v"(lo), "v"(hi)); return r; }

__global__ __launch_bounds__(NTH, 2) void gemm_ar_kernel(Params P) {
    extern __shared__ __attribute__((aligned(16))) unsigned char smem[];
    lds_u8* sm = (lds_u8*)smem;
    const int tid = threadIdx.x, lane = tid & 63, wid = (int)rfl(tid >> 6);
    const int b = blockIdx.x, xcd = b & 7, q = b >> 3;
    const GAS unsigned short* px = (const GAS unsigned short*)P.x; const GAS unsigned short* pw = (const GAS unsigned short*)P.w;
    int rank = P.rank;
    asm volatile("" : "+s"(px), "+s"(pw), "+s"(rank));
    const int ri = rank >> 2, rj = rank & 3;
    unsigned char* my = P.ws[rank];
    u32* tmo = (u32*)(my + OFF_TMO);
    const lds_u8* tab = sm + LDS_TAB;
    const bool is_dec = q >= 16 && q < 16 + ND && xcd < NR - 1;
    const int peer_r = (rank + 1 + xcd) & 7;
    const int pi = q * 4 + (wid - 4);
    const bool pusher = wid >= 4 && q < 8 && xcd < NR - 1 && pi < NPUSH;
    const int pid = pusher ? (pi < 16 ? 512 + (16 * (peer_r & 3) + pi) * 8 : (32 * (peer_r >> 2) + pi - 16) * 8) : b * 4 + wid;
    const bool enc = wid < 4 || pusher;
    const bool isw = pid >= 512;
    const int blk = (pid & 511) >> 3, ch = pid & 7;
    u32x4 raw[8];
    auto load_raw = [&]() {
        if (!isw) {
            const GAS unsigned short* s = px + (size_t)(blk * 64 + (lane & 15)) * KSH + ch * 64 + (lane >> 4) * 8;
#pragma unroll
            for (int j = 0; j < 8; ++j) raw[j] = *(const GAS u32x4*)(s + (size_t)(j >> 1) * 16 * KSH + (j & 1) * 32);
        } else {
            const GAS unsigned short* s = pw + (size_t)(ch * 64 + (lane >> 3)) * NN + blk * 64 + (lane & 7) * 8;
#pragma unroll
            for (int j = 0; j < 8; ++j) raw[j] = *(const GAS u32x4*)(s + (size_t)j * 8 * NN);
        }
    };
    const int cls = pusher ? 0 : ((!isw && (blk & 31) >= GA) ? 2 : (ch >= PRI0 ? 1 : 0));
    if (enc && cls == 0) load_raw();
    u32 call_v;
    { u64 ex; unsigned char* ca = my + OFF_CALL + b * 128 + wid * 4; const u32 one = 1u;
      asm volatile("s_mov_b64 %1, exec\n\ts_mov_b64 exec, 1\n\tglobal_atomic_add %0, %2, %3, off sc0\n\ts_mov_b64 exec, %1" : "=&v"(call_v), "=&s"(ex) : "v"(ca), "v"(one) : "memory"); }
    __builtin_amdgcn_sched_barrier(0);
    unsigned char* wsl = P.ws[lane & 7];
    if (tid < 256) codec_build_tables(sm + LDS_TAB, tid);
    if (is_dec && tid < 320) *lp<u32>(sm + LDS_CTL + tid * 4) = 0;
    __syncthreads();
    asm volatile("s_waitcnt vmcnt(0)" : "+v"(call_v) :: "memory");
    const u32 call = rfl(call_v) + 1u;
    if (enc) {
        if (cls != 0) {
            for (int k = 0; k < cls * PRID - 1; ++k) __builtin_amdgcn_s_sleep(16);
            load_raw();
        }
        lds_u8* sc = sm + (is_dec ? LDS_PRIV : LDS_ROLE) + wid * CD_BYTES;
        u32 x[32];
        if (!isw) {
#pragma unroll
            for (int j = 0; j < 8; ++j) { x[4 * j] = raw[j].x; x[4 * j + 1] = raw[j].y; x[4 * j + 2] = raw[j].z; x[4 * j + 3] = raw[j].w; }
        } else {
#pragma unroll
            for (int j = 0; j < 8; ++j) { const int k = 8 * j + (lane >> 3); *lp<u32x4>(sc + k * 128 + (((lane & 7) ^ (k & 7)) << 4)) = raw[j]; }
            const int g = lane >> 4, qq = (lane & 15) >> 2, pp = lane & 3;
#pragma unroll
            for (int j = 0; j < 8; ++j) {
                const int ni = j >> 1, ks = j & 1;
#pragma unroll
                for (int t = 0; t < 2; ++t) {
                    const int k = 32 * ks + 8 * g + 4 * t + qq, chunk = 2 * ni + (pp >> 1);
                    const s16x4 v = __builtin_amdgcn_ds_read_tr16_b64_v4i16((LAS s16x4*)(sc + k * 128 + ((chunk ^ (k & 7)) << 4) + (pp & 1) * 8));
                    const u32x2 u = __builtin_bit_cast(u32x2, v);
                    x[4 * j + 2 * t] = u.x; x[4 * j + 2 * t + 1] = u.y;
                }
            }
            asm volatile("s_waitcnt lgkmcnt(0)" ::: "memory");
        }
        const int rb = blk & 31, step = rank * 8 + ch;
        const size_t stg = isw ? OFF_WS + (size_t)(step * 16 + (blk & 15)) * PIECE_B : OFF_XS + (size_t)(step * 32 + rb) * PIECE_B;
        const size_t cnt = OFF_CNT + (size_t)(isw ? 128 + (blk & 15) * 8 + ch : (rb >> 1) * 8 + ch) * 128;
        const bool cons = lane < NR && (isw ? (lane & 3) == (blk >> 4) : (lane >> 2) == (blk >> 5));
        const bool mine = isw ? ((blk >> 4) == rj) : ((blk >> 5) == ri);
        if (pusher) {
            unsigned char* d = P.ws[peer_r] + stg + lane * 16;
#pragma unroll
            for (int j = 0; j < 8; ++j) st_sys(d + j * 1024, (u32x4){x[4 * j], x[4 * j + 1], x[4 * j + 2], x[4 * j + 3]});
            wait_vm0();
            if (lane == peer_r) __hip_atomic_fetch_add((GAS u32*)(wsl + cnt), 1u, __ATOMIC_RELAXED, __HIP_MEMORY_SCOPE_SYSTEM);
        } else if (ch == 0 && (isw ? (blk & 15) < NPUSH : (rb < GA && 16 + rb < NPUSH))) {
            if (mine) {
                unsigned char* d = my + stg + lane * 16;
#pragma unroll
                for (int j = 0; j < 8; ++j) st_sys(d + j * 1024, (u32x4){x[4 * j], x[4 * j + 1], x[4 * j + 2], x[4 * j + 3]});
                wait_vm0();
                if (lane == 0) __hip_atomic_fetch_add((GAS u32*)(my + cnt), 1u, __ATOMIC_RELAXED, __HIP_MEMORY_SCOPE_SYSTEM);
            }
        } else {
            u32 A[16], L1[4];
            const u32 inf = codec_encode(x, A, L1, sc, tab, lane);
            store_piece(x, A, L1, inf, my + OFF_OBI + (size_t)pid * PIECE_B, sc, lane);
            wait_vm0();
            const int idx = isw ? ch * CHP + (blk & 15) - NPUSH : (rb < GA ? ch * CHP + 16 + rb - NPUSH : NINA + ((rb - GA) >> 1) * 16 + ch * 2 + (rb & 1));
            if (cons && lane != rank) __hip_atomic_store((GAS u64*)(wsl + flag_off(rank, idx)), (u64)call << 32 | inf, __ATOMIC_RELAXED, __HIP_MEMORY_SCOPE_SYSTEM);
            if (mine) {
                unsigned char* d = my + stg + lane * 16;
#pragma unroll
                for (int j = 0; j < 8; ++j) st_agent(d + j * 1024, (u32x4){x[4 * j], x[4 * j + 1], x[4 * j + 2], x[4 * j + 3]});
                wait_vm0();
                if (lane == 0) __hip_atomic_fetch_add((GAS u32*)(my + cnt), 1u, __ATOMIC_RELAXED, __HIP_MEMORY_SCOPE_AGENT);
            }
        }
    }

    if (q < 16 || q >= 24) {
        __syncthreads();
        lds_u8* ring = sm + LDS_ROLE;
        const int wr = wid >> 1, wc = wid & 1;
        const int cb = 4 * (xcd & 3) + (q & 3);
        bool bad = false;
        const int ntile = q < 8 ? 2 : 1;
        const int q2 = q < 16 ? q : q - 8;
        for (int t = 0; t < ntile; ++t) {
            const int a = t == 1 ? 12 + 2 * (xcd >> 2) + (q >> 2) : (q2 < 12 ? 3 * (xcd >> 2) + (q2 >> 2) : 6 + 3 * (xcd >> 2) + ((q2 - 12) >> 2));
            const unsigned char* gx = my + OFF_XS + (size_t)(2 * a) * PIECE_B;
            const unsigned char* gw = my + OFF_WS + (size_t)cb * PIECE_B;
            const bool ldA = wid < 4;
            const unsigned char* gl = (ldA ? gx + wid * 4096 : gw + (wid - 4) * 4096) + lane * 16;
            const size_t lstep = ldA ? (size_t)32 * PIECE_B : (size_t)16 * PIECE_B;
            f32x4 acc[2][2] = {};
            auto issue = [&](int i) {
                const int step = (i & 7) * 8 + (i >> 3);
                lds_u8* d = ring + (i % NST) * STAGE_B + wid * 4096;
                const unsigned char* sa = gl + (size_t)step * lstep;
#pragma unroll
                for (int j = 0; j < 4; ++j) __builtin_amdgcn_global_load_lds((const GAS u32*)(sa + j * 1024), (LAS u32*)(d + j * 1024), 16, 0, 17);
            };
            const unsigned char* cx = my + OFF_CNT + (size_t)(a * 8) * 128; const unsigned char* cw = my + OFF_CNT + (size_t)(128 + cb * 8) * 128;
            int rdy = 0;
            u32 pa = 0, pb = 0; bool pend = false;
            auto poll_issue = [&]() { pa = __hip_atomic_load((const GAS u32*)(cx + rdy * 128), __ATOMIC_RELAXED, __HIP_MEMORY_SCOPE_SYSTEM); pb = __hip_atomic_load((const GAS u32*)(cw + rdy * 128), __ATOMIC_RELAXED, __HIP_MEMORY_SCOPE_SYSTEM); pend = true; };
            auto poll_take = [&]() { if (pend) { pend = false; if (rfl(pa) >= 16u * call && rfl(pb) >= 8u * call) ++rdy; } };
            auto need = [&](int c) {
                poll_take();
                while (rdy <= c) {
                    if (!wait_cnt2(cx + rdy * 128, 16u * call, cw + rdy * 128, 8u * call, call, tmo)) bad = true;
                    ++rdy;
                }
            };
            auto rd = [&](int i, bf16x8 (&af)[2][2], bf16x8 (&bfr)[2][2]) {
                lds_u8* st = ring + (i % NST) * STAGE_B;
#pragma unroll
                for (int m = 0; m < 2; ++m)
#pragma unroll
                    for (int ks = 0; ks < 2; ++ks) af[m][ks] = *lp<bf16x8>(st + (wr >> 1) * 8192 + (((wr & 1) * 2 + m) * 2 + ks) * 1024 + lane * 16);
#pragma unroll
                for (int n = 0; n < 2; ++n)
#pragma unroll
                    for (int ks = 0; ks < 2; ++ks) bfr[n][ks] = *lp<bf16x8>(st + 16384 + ((wc * 2 + n) * 2 + ks) * 1024 + lane * 16);
            };
            auto mm = [&](bf16x8 (&af)[2][2], bf16x8 (&bfr)[2][2]) {
#pragma unroll
                for (int ks = 0; ks < 2; ++ks)
#pragma unroll
                    for (int m = 0; m < 2; ++m)
#pragma unroll
                        for (int n = 0; n < 2; ++n) acc[m][n] = __builtin_amdgcn_mfma_f32_16x16x32_bf16(bfr[n][ks], af[m][ks], acc[m][n], 0, 0, 0);
            };
            auto kloop = [&](auto role_c) {
                constexpr int ROLE = decltype(role_c)::value;
                bf16x8 a0[2][2], b0[2][2], a1[2][2], b1[2][2];
                if constexpr (ROLE == 1) need(0);
                __builtin_amdgcn_s_barrier();
                if constexpr (ROLE == 0) {
#pragma unroll
                    for (int i = 0; i < NST - 1; ++i) issue(i);
                    __builtin_amdgcn_s_waitcnt(wcnt(4 * (NST - 2), 15));
                }
                __builtin_amdgcn_s_barrier();
                rd(0, a0, b0);
                auto pair = [&](int i, auto tail_c) {
                    constexpr int TAIL = decltype(tail_c)::value;
                    { const int nx = i + NST - 1;
                      if constexpr (ROLE == 0) __builtin_amdgcn_s_waitcnt(wcnt(TAIL == 0 || TAIL + 4 < 64 ? 4 * (NST - 3) : 4 * (63 - TAIL - 1 > 0 ? 63 - TAIL - 1 : 0), 0));
                      else __builtin_amdgcn_s_waitcnt(wcnt(63, 0));
                      __builtin_amdgcn_s_barrier();
                      if constexpr (ROLE == 0 && (TAIL == 0 || TAIL + NST - 1 < 64)) issue(nx);
                      rd(i + 1, a1, b1);
                      __builtin_amdgcn_sched_barrier(0);
                      mm(a0, b0);
                      __builtin_amdgcn_sched_barrier(0);
                      if constexpr (ROLE == 1 && TAIL == 0) { if ((i & 2) == 0 && rdy < 8) { poll_take(); if (rdy < 8) poll_issue(); } }
                    }
                    { const int nx = i + NST;
                      if constexpr (ROLE == 1 && TAIL == 0) { if ((nx & 7) == 0) need(nx >> 3); }
                      if constexpr (ROLE == 0) __builtin_amdgcn_s_waitcnt(wcnt(TAIL == 0 ? 4 * (NST - 3) : 4 * (63 - TAIL - 2 > 0 ? 63 - TAIL - 2 : 0), 0));
                      else __builtin_amdgcn_s_waitcnt(wcnt(63, 0));
                      __builtin_amdgcn_s_barrier();
                      if constexpr (ROLE == 0 && TAIL == 0) issue(nx);
                      if constexpr (TAIL != 62) rd(i + 2, a0, b0);
                      __builtin_amdgcn_sched_barrier(0);
                      mm(a1, b1);
                      __builtin_amdgcn_sched_barrier(0);
                    }
                };
#pragma unroll 1
                for (int i = 0; i < 58; i += 2) pair(i, std::integral_constant<int, 0>{});
                pair(58, std::integral_constant<int, 58>{});
                pair(60, std::integral_constant<int, 60>{});
                pair(62, std::integral_constant<int, 62>{});
            };
            if (wid < 6) kloop(std::integral_constant<int, 0>{});
            else if (wid == 6) kloop(std::integral_constant<int, 1>{});
            else kloop(std::integral_constant<int, 2>{});
            wait_vm0();
            __syncthreads();
#pragma unroll
            for (int m = 0; m < 2; ++m)
#pragma unroll
                for (int n = 0; n < 2; ++n) {
                    const int row = 32 * wr + 16 * m + (lane & 15), col = 32 * wc + 16 * n + 4 * (lane >> 4);
                    const u32x2 v = {pk_bf16(acc[m][n][0], acc[m][n][1]), pk_bf16(acc[m][n][2], acc[m][n][3])};
                    *lp<u32x2>(ring + row * 128 + (((col >> 3) ^ (row & 7)) << 4) + (col & 4) * 2) = v;
                }
            if (tid == 6 * 64) *lp<u32>(sm + LDS_MISC + 272) = (bad || __hip_atomic_load((const GAS u32*)tmo, __ATOMIC_RELAXED, __HIP_MEMORY_SCOPE_AGENT) == call) ? 1u : 0u;
            __syncthreads();
            if (wid < 4) {
                const int h = wid & 1;
                const bool poison = *lp<u32>(sm + LDS_MISC + 272) != 0u;
                u32 x[32];
#pragma unroll
                for (int j = 0; j < 8; ++j) {
                    const int row = 64 * h + 8 * j + (lane >> 3);
                    u32x4 v = *lp<u32x4>(ring + row * 128 + (((lane & 7) ^ (row & 7)) << 4));
                    if (poison) v = (u32x4){0x7fc07fc0u, 0x7fc07fc0u, 0x7fc07fc0u, 0x7fc07fc0u};
                    x[4 * j] = v.x; x[4 * j + 1] = v.y; x[4 * j + 2] = v.z; x[4 * j + 3] = v.w;
                }
                const int o = (a * 16 + cb) * 2 + h;
                if (wid < 2) {
                    lds_u8* sce = ring + 16384 + wid * CD_BYTES;
                    u32 A[16], L1[4];
                    u32 inf = 0x80000000u;
                    if (o < NOUT - RAWT) inf = codec_encode(x, A, L1, sce, tab, lane);
                    store_piece(x, A, L1, inf, my + OFF_OBO + (size_t)o * PIECE_B, sce, lane);
                    wait_vm0();
                    if (lane < NR && lane != rank)
                        __hip_atomic_store((GAS u64*)(wsl + flag_off(rank, NIN + o)), (u64)call << 32 | inf, __ATOMIC_RELAXED, __HIP_MEMORY_SCOPE_SYSTEM);
                } else {
                    unsigned short* d = P.out + (size_t)(2048 * ri + 128 * a + 64 * h + (lane >> 3)) * NN + 1024 * rj + 64 * cb + (lane & 7) * 8;
#pragma unroll
                    for (int j = 0; j < 8; ++j) st_agent(d + (size_t)j * 8 * NN, (u32x4){x[4 * j], x[4 * j + 1], x[4 * j + 2], x[4 * j + 3]});
                }
            }
            __syncthreads();
        }
    } else if (is_dec) {
        const int src = peer_r, si = src >> 2, sj = src & 3;
        const unsigned char* peer = P.ws[src];
        const int dj = q - 16;
        const int np = (NQ - dj + ND - 1) / ND;
        volatile LAS u32* c_info = (volatile LAS u32*)(sm + LDS_CTL);
        volatile LAS u32* c_full = (volatile LAS u32*)(sm + LDS_CTL + 1024);
        volatile LAS u32* c_done = (volatile LAS u32*)(sm + LDS_CTL + 1088);
        volatile LAS u32* c_piece = (volatile LAS u32*)(sm + LDS_CTL + 1152);
        LAS u32* c_ticket = (LAS u32*)(sm + LDS_CTL + 1216);
        auto src_slot = [&](int idx) -> const unsigned char* {
            if (idx >= NIN) return peer + OFF_OBO + (size_t)(idx - NIN) * PIECE_B;
            int c, bl; bool isw2; in_entry(idx, c, isw2, bl);
            return peer + OFF_OBI + (size_t)(isw2 ? 512 + (16 * rj + bl) * 8 + c : (32 * ri + bl) * 8 + c) * PIECE_B;
        };
        if (wid == 6) {
            const unsigned char* fb = my + OFF_FLAG + (size_t)(src * ND + dj) * 256 * 8;
            const u64 t0 = now();
            bool p0 = lane >= np, p1 = lane + 64 >= np;
            int rounds = 0;
            for (;;) {
                u64 v0 = 0, v1 = 0;
                if (!p0) v0 = __hip_atomic_load((const GAS u64*)(fb + (size_t)lane * 8), __ATOMIC_RELAXED, __HIP_MEMORY_SCOPE_SYSTEM);
                if (!p1) v1 = __hip_atomic_load((const GAS u64*)(fb + (size_t)(lane + 64) * 8), __ATOMIC_RELAXED, __HIP_MEMORY_SCOPE_SYSTEM);
                if (!p0 && (u32)(v0 >> 32) >= call) { c_info[lane] = (u32)v0 | 0x40000000u; p0 = true; }
                if (!p1 && (u32)(v1 >> 32) >= call) { c_info[lane + 64] = (u32)v1 | 0x40000000u; p1 = true; }
                if (__all(p0 && p1)) break;
                if ((++rounds & 7) == 0 && (rfl(__hip_atomic_load((const GAS u32*)tmo, __ATOMIC_RELAXED, __HIP_MEMORY_SCOPE_AGENT)) == call || now() - t0 > DEADLINE)) {
                    __hip_atomic_store((GAS u32*)tmo, call, __ATOMIC_RELAXED, __HIP_MEMORY_SCOPE_AGENT);
                    if (!p0) c_info[lane] = 0x60000000u;
                    if (!p1) c_info[lane + 64] = 0x60000000u;
                    break;
                }
                __builtin_amdgcn_s_sleep(2);
            }
        } else if (wid == 7) {
            int pub = 0;
            auto publish_upto = [&](int e) { for (; pub < e; ++pub) if (lane == 0) c_full[pub % NSLOT] = pub + 1; };
            u64 todo0 = np >= 64 ? ~0ull : (1ull << np) - 1ull, todo1 = np > 64 ? (1ull << (np - 64)) - 1ull : 0ull;
            int n = 0; u32 inf = 0;
            auto pick = [&](int s) -> bool {
                const u32 v0 = c_info[lane], v1 = c_info[lane + 64], dn = c_done[s % NSLOT];
                const u64 a0 = __ballot(v0 != 0u) & todo0, a1 = __ballot(v1 != 0u) & todo1;
                if ((a0 | a1) == 0ull || !(s < NSLOT || rfl(dn) == (u32)(s - NSLOT + 1))) return false;
                if (a0) { const int l = __builtin_ctzll(a0); todo0 &= ~(1ull << l); n = l; inf = (u32)__builtin_amdgcn_readlane((int)v0, l); }
                else { const int l = __builtin_ctzll(a1); todo1 &= ~(1ull << l); n = 64 + l; inf = (u32)__builtin_amdgcn_readlane((int)v1, l); }
                return true;
            };
            bool have = false;
            for (int s = 0; s < np; ++s) {
                const int slot = s % NSLOT;
                if (!have) { wait_vm0(); publish_upto(s); while (!pick(s)) __builtin_amdgcn_s_sleep(1); }
                if (lane == 0) c_piece[slot] = n;
                lds_u8* d = sm + LDS_SLOTS + slot * PIECE_B;
                const unsigned char* s0 = src_slot(dj + n * ND);
                const unsigned char* sp = s0 + lane * 16;
                if (inf & 0x20000000u) { have = false; wait_vm0(); publish_upto(s + 1); continue; }
                const bool israw = (inf >> 31) != 0u;
                const unsigned char* sv = (israw || (u32)lane * 16u < ((inf >> 8) & 0x7ffu)) ? sp + 5120 : s0 + 5120;
                const bool input = dj + n * ND < NIN;
                auto pull = [&](auto nfl_c, auto cnt_c) {
                    constexpr int N = decltype(nfl_c)::value, CNT = decltype(cnt_c)::value;
#pragma unroll
                    for (int j = 0; j < CNT; ++j) {
                        asm volatile("s_waitcnt vmcnt(%0)" :: "n"(N - 1) : "memory");
                        if (j == 0 || (j - N + 1) % 6 == 0) publish_upto(s + (j - N + 1 - 5) / 6);
                        __builtin_amdgcn_global_load_lds((const GAS u32*)(j == 5 ? sv : sp + j * 1024), (LAS u32*)(d + j * 1024), 16, 0, 17);
                        if (j == 1) have = s + 1 < np && pick(s + 1);
                    }
                };
                static_assert(NFL >= 9 && NFLI >= 7, "the publish rule needs j - N + 1 < 0");
                if (israw) pull(std::integral_constant<int, NFL>{}, std::integral_constant<int, 8>{});
                else if (input) pull(std::integral_constant<int, NFLI>{}, std::integral_constant<int, 6>{});
                else pull(std::integral_constant<int, NFL>{}, std::integral_constant<int, 6>{});
            }
            wait_vm0(); publish_upto(np);
        } else {
            lds_u8* sc = sm + LDS_PRIV + wid * CD_BYTES;
            for (;;) {
                u32 nn = 0;
                if (lane == 0) nn = __hip_atomic_fetch_add(c_ticket, 1u, __ATOMIC_RELAXED, __HIP_MEMORY_SCOPE_WORKGROUP);
                const int sq = (int)rfl(nn);
                if (sq >= np) break;
                while (rfl(c_full[sq % NSLOT]) != (u32)(sq + 1)) __builtin_amdgcn_s_sleep(1);
                asm volatile("" ::: "memory");
                const int n = (int)rfl(c_piece[sq % NSLOT]);
                const u32 inf = rfl(c_info[n]);
                const bool ok = !(inf & 0x20000000u);
                lds_u8* sl = sm + LDS_SLOTS + (sq % NSLOT) * PIECE_B;
                u32 x[32];
                if (!ok) {
#pragma unroll
                    for (int j = 0; j < 32; ++j) x[j] = 0x7fc07fc0u;
                } else if (inf >> 31) {
#pragma unroll
                    for (int j = 0; j < 8; ++j) { const u32x4 v = *lp<u32x4>(sl + j * 1024 + lane * 16); x[4 * j] = v.x; x[4 * j + 1] = v.y; x[4 * j + 2] = v.z; x[4 * j + 3] = v.w; }
                } else {
                    u32 A[16], L1[4];
#pragma unroll
                    for (int j = 0; j < 4; ++j) { const u32x4 v = *lp<u32x4>(sl + j * 1024 + lane * 16); A[4 * j] = v.x; A[4 * j + 1] = v.y; A[4 * j + 2] = v.z; A[4 * j + 3] = v.w; }
                    { const u32x4 v = *lp<u32x4>(sl + 4096 + lane * 16); L1[0] = v.x; L1[1] = v.y; L1[2] = v.z; L1[3] = v.w; }
                    codec_decode(x, A, L1, inf & 255u, (const LAS u32*)(sl + 5120), sc, tab, lane);
                }
                asm volatile("s_waitcnt lgkmcnt(0)" ::: "memory");
                if (lane == 0) c_done[sq % NSLOT] = sq + 1;
                const int idx = dj + n * ND;
                if (idx < NIN) {
                    int c, bl; bool isw2; in_entry(idx, c, isw2, bl);
                    const int step = src * 8 + c;
                    unsigned char* d = (isw2 ? my + OFF_WS + (size_t)(step * 16 + bl) * PIECE_B : my + OFF_XS + (size_t)(step * 32 + bl) * PIECE_B) + lane * 16;
                    if (ok) {
#pragma unroll
                        for (int j = 0; j < 8; ++j) st_agent(d + j * 1024, (u32x4){x[4 * j], x[4 * j + 1], x[4 * j + 2], x[4 * j + 3]});
                        wait_vm0();
                        if (lane == 0) __hip_atomic_fetch_add((GAS u32*)(my + OFF_CNT + (size_t)(isw2 ? 128 + bl * 8 + c : (bl >> 1) * 8 + c) * 128), 1u, __ATOMIC_RELAXED, __HIP_MEMORY_SCOPE_AGENT);
                    }
                } else {
                    const int o = idx - NIN, g = o >> 8, tile = (o & 255) >> 1, h = o & 1;
                    unsigned short* d = P.out + (size_t)(2048 * si + 1024 * g + 128 * (tile >> 4) + 64 * h + (lane >> 3)) * NN + 1024 * sj + 64 * (tile & 15) + (lane & 7) * 8;
#pragma unroll
                    for (int j = 0; j < 8; ++j) st_agent(d + (size_t)j * 8 * NN, (u32x4){x[4 * j], x[4 * j + 1], x[4 * j + 2], x[4 * j + 3]});
                }
            }
        }
    }
}

extern "C" void kernel_launch(int rank, int nranks, ncclComm_t comm, hipStream_t stream,
                              void* const* d_in, const int* in_sizes, int n_in,
                              void* d_out, int out_size, const int* in_dtypes, int out_dtype,
                              void* const* d_out_all, void* const* d_ws_all, size_t ws_bytes) {
    (void)comm; (void)in_dtypes; (void)out_dtype; (void)d_out_all;
    if (nranks != NR || n_in != 2 || rank < 0 || rank >= NR || in_sizes[0] != MM * KSH || in_sizes[1] != KSH * NN || out_size != MM * NN || ws_bytes < WS_NEED) {
        fprintf(stderr, "kernel_launch: unexpected problem shape\n");
        return;
    }
    static bool first = true;
    if (first) {
        hipError_t e = hipFuncSetAttribute((const void*)gemm_ar_kernel, hipFuncAttributeMaxDynamicSharedMemorySize, LDS_BYTES);
        if (e != hipSuccess) { fprintf(stderr, "hipFuncSetAttribute: %s\n", hipGetErrorString(e)); return; }
        first = false;
    }
    Params P;
    P.x = (const unsigned short*)d_in[0];
    P.w = (const unsigned short*)d_in[1];
    P.out = (unsigned short*)d_out;
    for (int p = 0; p < NR; ++p) P.ws[p] = (unsigned char*)d_ws_all[p];
    P.rank = rank;
    P.pad_ = 0;
    hipLaunchKernelGGL(gemm_ar_kernel, dim3(NWG), dim3(NTH), LDS_BYTES, stream, P);
    hipError_t e = hipGetLastError();
    if (e != hipSuccess) fprintf(stderr, "launch: %s\n", hipGetErrorString(e));
}
```
